```python
import numpy as np

kernel_name = "gpu_ag_gemm_m4096_k4096_n16384_bf16_reluqe4m3"


ABSMAX_RTOL = 0.09

COMPUTE_DTYPE = "bfloat16"
EPILOGUE = "relu_quant_e4m3"
IN_QUANT = None
OUT_QUANT = "e4m3"
IN_DTYPES = {"x": "bfloat16", "w_mat": "bfloat16"}
WORLD_SIZE = 8

def _to_bf16(x):
    u = np.ascontiguousarray(x, dtype=np.float32).view(np.uint32)
    return ((u + 0x7FFF + ((u >> 16) & 1)) & 0xFFFF0000).view(np.float32)

def setup_inputs():

    x = _to_bf16(np.random.randn(4096, 4096).astype(np.float32))
    w_mat = _to_bf16((np.random.randn(4096, 16384) / np.sqrt(4096)).astype(np.float32))
    return {"x": x, "w_mat": w_mat}

def _snap_fp8(x, mant_bits, max_abs, min_normal):
    a = np.abs(x.astype(np.float64))
    _m, _e = np.frexp(a)
    q = np.where(a >= min_normal, np.ldexp(1.0, _e - 1 - mant_bits),
                 min_normal * 2.0 ** -mant_bits)
    snapped = np.minimum(np.round(a / q) * q, max_abs)
    return (np.sign(x) * snapped).astype(np.float32)
def _quant_dequant(y, amax):
    scale = amax / 448.0
    q = _snap_fp8(y / scale, 3, 448.0, 2.0 ** -6)
    return (q * scale).astype(np.float32)
def _epilogue(y):
    y = np.maximum(y, 0.0)
    return _quant_dequant(y, np.abs(y).max())

def reference(x, w_mat):
    y = (x @ w_mat).astype(np.float32)
    return _epilogue(y)
```

```cpp
#include <hip/hip_runtime.h>
#include <rccl/rccl.h>
#include <cstdio>

#define HIP_CHECK(expr) do { hipError_t _e = (expr); if (_e != hipSuccess) { \
    fprintf(stderr, "HIP error %s at %s:%d\n", hipGetErrorString(_e), __FILE__, __LINE__); \
    return; } } while (0)

namespace agk {

typedef unsigned u32;
typedef unsigned long long u64;
typedef u32 u32x4 __attribute__((ext_vector_type(4)));
typedef float f32x4 __attribute__((ext_vector_type(4)));
typedef __bf16 bf16x8 __attribute__((ext_vector_type(8)));
typedef short s16x4 __attribute__((ext_vector_type(4)));
typedef unsigned short u16x2 __attribute__((ext_vector_type(2)));
#define LAS __attribute__((address_space(3)))

constexpr int NR = 8;
constexpr int KD = 4096;
constexpr int NL = 2048;
constexpr int GRID = 256, THREADS = 704;
constexpr int NGEMM = 208, NCOMM = 48;
constexpr int STAGE = 52 * 1024;
constexpr int LDS_MISC = 3 * STAGE;
constexpr int LDS_BYTES = LDS_MISC + 1024;

constexpr u32 OFF_SEQ = 0;
constexpr u32 OFF_TMO = 16384;
constexpr u32 OFF_CNT = 20480;
constexpr u32 OFF_LMAX = 24576;
constexpr u32 OFF_GMAX = 28672;
constexpr u32 OFF_LOC = 32768;
constexpr u32 OFF_TAG = 40960;
constexpr u32 OFF_GO = 49152;
constexpr u32 OFF_DBG = 128u << 10;
constexpr u32 OFF_HDR = 512u << 10;
constexpr u32 OFF_PK = 1u << 20;
constexpr u32 OFF_XD = 4u << 20;
constexpr u32 WS_USED = 36u << 20;

constexpr u64 DEADLINE_TICKS = 40000000ull;
constexpr int AUX_SYS = 17, AUX_AGT = 16;

struct Params {
    const void* x;
    const void* w;
    void* out;
    char* ws[NR];
    int rank;
    int pad;
};

#define WAIT_V(n) asm volatile("s_waitcnt vmcnt(" #n ")" ::: "memory")
#define WAIT_L(n) asm volatile("s_waitcnt lgkmcnt(" #n ")" ::: "memory")
#define BAR() do { asm volatile("" ::: "memory"); __builtin_amdgcn_s_barrier(); asm volatile("" ::: "memory"); } while (0)
#define SCHED() __builtin_amdgcn_sched_barrier(0)

__device__ __forceinline__ u32 rfl(u32 v) { return __builtin_amdgcn_readfirstlane(v); }
__device__ __forceinline__ __amdgpu_buffer_rsrc_t mk_rsrc(const void* p, u32 bytes) {
    return __builtin_amdgcn_make_buffer_rsrc((void*)p, 0, (int)bytes, 0x00020000);
}
__device__ __forceinline__ u32 umax(u32 a, u32 b) { return a > b ? a : b; }
__device__ __forceinline__ u32 wave_max_u32(u32 v) {
    v = umax(v, (u32)__builtin_amdgcn_update_dpp(0, (int)v, 0x111, 0xf, 0xf, true));
    v = umax(v, (u32)__builtin_amdgcn_update_dpp(0, (int)v, 0x112, 0xf, 0xf, true));
    v = umax(v, (u32)__builtin_amdgcn_update_dpp(0, (int)v, 0x114, 0xf, 0xf, true));
    v = umax(v, (u32)__builtin_amdgcn_update_dpp(0, (int)v, 0x118, 0xf, 0xf, true));
    return umax(umax((u32)__builtin_amdgcn_readlane((int)v, 15), (u32)__builtin_amdgcn_readlane((int)v, 31)),
                umax((u32)__builtin_amdgcn_readlane((int)v, 47), (u32)__builtin_amdgcn_readlane((int)v, 63)));
}
__device__ __forceinline__ bool give_up(u32* tmo, u32 call, u64 t0) {
    const u64 dt = __builtin_amdgcn_s_memrealtime() - t0;
    if (dt < 20000ull) return false;
    const u32 t = rfl(__hip_atomic_load(tmo, __ATOMIC_RELAXED, __HIP_MEMORY_SCOPE_AGENT));
    if (t == call) return true;
    if (dt > DEADLINE_TICKS) {
        __hip_atomic_store(tmo, call, __ATOMIC_RELAXED, __HIP_MEMORY_SCOPE_AGENT);
        return true;
    }
    return false;
}

#define POLL_SLEEP 2
#define STAMP(i) do {} while (0)
#define STAMP_OUT() do {} while (0)

__global__ __launch_bounds__(THREADS, 3) void ag_gemm(Params P) {
    extern __shared__ __attribute__((aligned(16))) char smem_[];
    LAS char* smem = (LAS char*)smem_;
    const int lane = threadIdx.x & 63;
    const int wid = (int)rfl(threadIdx.x >> 6);
    const int b = blockIdx.x;
    const int rank = P.rank & 7;
    char* ws = P.ws[rank];
    u32* tmo = (u32*)(ws + OFF_TMO);
    STAMP(0);
    const __amdgpu_buffer_rsrc_t rsW = mk_rsrc(ws, WS_USED);
    const int xcd = b & 7, idx = b >> 3;
    u32* seq = (u32*)(ws + OFF_SEQ) + b * 16 + wid;

    if (idx >= 26) {
        if (wid >= 8) return;
        const u32 call = rfl(__hip_atomic_load(seq, __ATOMIC_RELAXED, __HIP_MEMORY_SCOPE_AGENT)) + 1u;
        if (lane == 0) __hip_atomic_store(seq, call, __ATOMIC_RELAXED, __HIP_MEMORY_SCOPE_AGENT);
        const int ci = (idx - 26) * 8 + xcd;
        if (wid < 7) {
            const int src = (rank + 1 + wid) & 7;
            const __amdgpu_buffer_rsrc_t rsR = mk_rsrc(P.ws[src], WS_USED);
            bool dead = false;
            {
                const u64 t0 = __builtin_amdgcn_s_memrealtime();
                while (rfl(__hip_atomic_load((u32*)(ws + OFF_GO + (src * 48 + ci) * 128), __ATOMIC_RELAXED, __HIP_MEMORY_SCOPE_SYSTEM)) < call) {
                    if (give_up(tmo, call, t0)) { dead = true; break; }
                    __builtin_amdgcn_s_sleep(1);
                }
            }
            asm volatile("" ::: "memory");
            STAMP(2);
            if (!dead) {
                u32x4 l0, l1, nb; u32 hw;
                u32 tags;
                {
                    const u32 o = OFF_PK + (u32)ci * 3072u + lane * 16u;
                    l0 = __builtin_amdgcn_raw_buffer_load_b128(rsR, o, 0, AUX_SYS);
                    l1 = __builtin_amdgcn_raw_buffer_load_b128(rsR, o, 1024, AUX_SYS);
                    nb = __builtin_amdgcn_raw_buffer_load_b128(rsR, o, 2048, AUX_SYS);
                    hw = __builtin_amdgcn_raw_buffer_load_b32(rsR, OFF_HDR + (u32)ci * 32u + (lane & 7) * 4u, 0, AUX_SYS);
                    const u32 tg = __builtin_amdgcn_raw_buffer_load_b32(rsR, OFF_TAG + (u32)ci * 128u + (lane & 31) * 4u, 0, AUX_SYS);
                    tags = (u32)__builtin_amdgcn_ballot_w64(tg >= call);
                }
                int j = 0;
                for (int v = ci; v < 1024; v += NCOMM) {
                    const int kp = v >> 5, rg = v & 31;
                    const u32 h0 = rfl(hw), cnt = h0 >> 8;
                    const u32 xdb = OFF_XD + (u32)(kp * 4) * 262144u + (u32)(src * 32 + rg) * 1024u;
                    u32x4 o[4];
                    if (cnt != 255u) {
                        const u32 low4 = (h0 & 0xFFu) * 0x01010101u;
#pragma unroll
                        for (int i = 0; i < 4; ++i) {
                            const u32 n = nb[i];
                            const u32 ha = (n & 0x87878787u) + low4, hb = (__builtin_amdgcn_alignbit(n, n, 4) & 0x87878787u) + low4;
                            const u32 la = i < 2 ? l0[(i & 1) * 2] : l1[(i & 1) * 2], lb = i < 2 ? l0[(i & 1) * 2 + 1] : l1[(i & 1) * 2 + 1];
                            o[i][0] = __builtin_amdgcn_perm(ha, la, 0x05010400u);
                            o[i][1] = __builtin_amdgcn_perm(ha, la, 0x07030602u);
                            o[i][2] = __builtin_amdgcn_perm(hb, lb, 0x05010400u);
                            o[i][3] = __builtin_amdgcn_perm(hb, lb, 0x07030602u);
                        }
                    } else {
#pragma unroll
                        for (int i = 0; i < 4; ++i) o[i] = __builtin_amdgcn_raw_buffer_load_b128(rsR, xdb + lane * 16u, i * 262144, AUX_SYS);
                    }
#pragma unroll
                    for (int i = 0; i < 4; ++i) __builtin_amdgcn_raw_buffer_store_b128(o[i], rsW, xdb + lane * 16u, i * 262144, AUX_AGT);
                    if (cnt != 0u && cnt != 255u) {
                        if (lane >= 1 && (u32)lane <= cnt) {
                            const u32 pos = hw >> 16;
                            __builtin_amdgcn_raw_buffer_store_b16((unsigned short)(hw & 0xFFFFu), rsW, xdb + (pos >> 9) * 262144u + (pos & 511u) * 2u, 0, AUX_AGT);
                        }
                    }
                    const int vn = v + NCOMM;
                    ++j;
                    if (vn < 1024) {
                        if (!((tags >> j) & 1u)) {
                            const u64 t0 = __builtin_amdgcn_s_memrealtime();
                            for (;;) {
                                const u32 tg = __builtin_amdgcn_raw_buffer_load_b32(rsR, OFF_TAG + (u32)ci * 128u + (lane & 31) * 4u, 0, AUX_SYS);
                                tags = (u32)__builtin_amdgcn_ballot_w64(tg >= call);
                                if ((tags >> j) & 1u) break;
                                if (give_up(tmo, call, t0)) { tags = ~0u; break; }
                                __builtin_amdgcn_s_sleep(1);
                            }
                        }
                        asm volatile("" ::: "memory");
                        const u32 o2 = OFF_PK + (u32)vn * 3072u + lane * 16u;
                        l0 = __builtin_amdgcn_raw_buffer_load_b128(rsR, o2, 0, AUX_SYS);
                        l1 = __builtin_amdgcn_raw_buffer_load_b128(rsR, o2, 1024, AUX_SYS);
                        nb = __builtin_amdgcn_raw_buffer_load_b128(rsR, o2, 2048, AUX_SYS);
                        hw = __builtin_amdgcn_raw_buffer_load_b32(rsR, OFF_HDR + (u32)vn * 32u + (lane & 7) * 4u, 0, AUX_SYS);
                        WAIT_V(4);
                    } else {
                        WAIT_V(0);
                    }
                    if (lane == 0) __hip_atomic_fetch_add((u32*)(ws + OFF_CNT) + src * 64 + kp, 1u, __ATOMIC_RELAXED, __HIP_MEMORY_SCOPE_AGENT);
                }
            }
            STAMP(3);
        } else if (ci == 1) {
            const u64 t0 = __builtin_amdgcn_s_memrealtime();
            for (;;) {
                bool ok = true;
#pragma unroll
                for (int i = 0; i < 4; ++i) {
                    const u32x4 f = __builtin_amdgcn_raw_buffer_load_b128(rsW, OFF_LOC + lane * 16, i * 1024, AUX_AGT);
                    ok = ok && f[0] >= call && f[1] >= call && f[2] >= call && f[3] >= call;
                }
                if (__builtin_amdgcn_ballot_w64(ok) == ~0ull) break;
                if (give_up(tmo, call, t0)) break;
                __builtin_amdgcn_s_sleep(4);
            }
            if (lane < 32) __hip_atomic_store((u32*)(ws + OFF_CNT) + rank * 64 + lane, call * 32u, __ATOMIC_RELAXED, __HIP_MEMORY_SCOPE_AGENT);
        } else if (ci == 0) {
            u64* dst = (u64*)(P.ws[lane & 7] + OFF_GMAX + rank * 128);
            const u64 t0 = __builtin_amdgcn_s_memrealtime();
            u32 m = 0;
            for (;;) {
                const u32x4 a = __builtin_amdgcn_raw_buffer_load_b128(rsW, OFF_LMAX + lane * 16, 0, AUX_AGT);
                const u32x4 c = __builtin_amdgcn_raw_buffer_load_b128(rsW, OFF_LMAX + 1024 + lane * 16, 0, AUX_AGT);
                const bool ok = a[1] == call && a[3] == call && (lane >= 40 || (c[1] == call && c[3] == call));
                m = a[0] > a[2] ? a[0] : a[2];
                if (lane < 40) { m = m > c[0] ? m : c[0]; m = m > c[2] ? m : c[2]; }
                if (__builtin_amdgcn_ballot_w64(ok) == ~0ull) break;
                if (give_up(tmo, call, t0)) break;
                __builtin_amdgcn_s_sleep(1);
            }
            STAMP(2);
            m = wave_max_u32(m);
            if (lane < NR) __hip_atomic_store(dst, ((u64)call << 32) | m, __ATOMIC_RELAXED, __HIP_MEMORY_SCOPE_SYSTEM);
            STAMP(3);
        }
        STAMP_OUT();
        return;
    }

    const int T = xcd * 26 + idx;
    const int ct = (T / 104) * 4 + (T % 104 & 3), rt = (T % 104) >> 2;
    const int rg0 = rt * 10 < 246 ? rt * 10 : 246;
    const u32 lds0 = (u32)(size_t)smem;
#define DMA(ldsaddr, voff, rs, soff, mods) asm volatile("s_mov_b32 m0, %0\n\ts_nop 0\n\tbuffer_load_dwordx4 %1, %2, %3 offen" mods " lds" \
        :: "s"(ldsaddr), "v"(voff), "s"(rs), "s"(soff) : "memory")
#define DMA_SAME_M0(voff, rs, soff, mods) asm volatile("buffer_load_dwordx4 %0, %1, %2 offen" mods " lds" :: "v"(voff), "s"(rs), "s"(soff) : "memory")

    if (wid >= 9) {
        __builtin_amdgcn_s_setprio(3);
        const int h = wid - 9;
        const __amdgpu_buffer_rsrc_t rsB = mk_rsrc(P.w, (u32)KD * NL * 2u);
        u32 vB[16];
#pragma unroll
        for (int i = 0; i < 16; ++i) {
            const int row = i * 2 + (lane >> 5);
            vB[i] = (u32)((h * 32 + row) * (NL * 2) + ct * 512 + (((lane & 31) ^ ((row & 3) | (row & 8))) << 4));
        }
        const u32 lB = lds0 + 20480 + h * 16384;
#define LOAD_W(kt, so) do { _Pragma("unroll") for (int i = 0; i < 16; ++i) DMA(lB + (so) + i * 1024, vB[i], rsB, (kt) * 262144, ""); } while (0)
        LOAD_W(0, 0);
        LOAD_W(1, STAGE);
        WAIT_V(16);
        BAR();
        int so = 2 * STAGE;
        for (int t = 0; t < 62; ++t) {
            LOAD_W(t + 2, so);
            so = so == 2 * STAGE ? 0 : so + STAGE;
            WAIT_V(16);
            BAR();
        }
        WAIT_V(0);
        BAR();
        BAR();
        return;
    }

    if (wid == 8) {
        __builtin_amdgcn_s_setprio(3);
        const u32 call = rfl(__hip_atomic_load(seq, __ATOMIC_RELAXED, __HIP_MEMORY_SCOPE_AGENT)) + 1u;
        if (lane == 0) __hip_atomic_store(seq, call, __ATOMIC_RELAXED, __HIP_MEMORY_SCOPE_AGENT);
        const u32 target = call * 32u;
        const u32* cp = (const u32*)(ws + OFF_CNT) + (lane < 32 ? rg0 >> 5 : (rg0 + 9) >> 5) * 64 + (lane & 31);
        const u32 vA = (u32)lane * 16u;
        const u32 gA = OFF_XD + (u32)rg0 * 1024u;
        int ready = 0;
        auto digest = [&](u32 c) {
            const u64 ok = __builtin_amdgcn_ballot_w64(c >= target);
            const u32 both = (u32)ok & (u32)(ok >> 32);
            ready = both == ~0u ? 64 : 2 * __builtin_ctz(~both);
        };
        auto need = [&](int kt) {
            if (kt < ready) return;
            const u64 t0 = __builtin_amdgcn_s_memrealtime();
            for (;;) {
                digest(__hip_atomic_load(cp, __ATOMIC_RELAXED, __HIP_MEMORY_SCOPE_AGENT));
                if (kt < ready) break;
                if (give_up(tmo, call, t0)) { ready = 64; break; }
                __builtin_amdgcn_s_sleep(POLL_SLEEP);
            }
        };
#define LOAD_XH(g, l) do { \
            DMA((l), vA, rsW, (g), " sc1"); DMA_SAME_M0(vA, rsW, (g), " offset:1024 sc1"); DMA_SAME_M0(vA, rsW, (g), " offset:2048 sc1"); DMA_SAME_M0(vA, rsW, (g), " offset:3072 sc1"); \
            DMA((l) + 4096, vA, rsW, (g) + 4096, " sc1"); DMA_SAME_M0(vA, rsW, (g) + 4096, " offset:1024 sc1"); DMA_SAME_M0(vA, rsW, (g) + 4096, " offset:2048 sc1"); DMA_SAME_M0(vA, rsW, (g) + 4096, " offset:3072 sc1"); \
            DMA((l) + 8192, vA, rsW, (g) + 8192, " sc1"); DMA_SAME_M0(vA, rsW, (g) + 8192, " offset:1024 sc1"); } while (0)
#define LOAD_X(kt, so) do { LOAD_XH(gA + (kt) * 524288, lds0 + (so)); LOAD_XH(gA + (kt) * 524288 + 262144, lds0 + (so) + 10240); } while (0)
        need(0);
        STAMP(2);
        LOAD_X(0, 0);
        LOAD_X(1, STAGE);
        WAIT_V(20);
        BAR();
        STAMP(3);
        int xi = 2, so = 2 * STAGE;
        for (int t = 0; t < 63; ++t) {
            if (xi == t + 1) {
                need(xi);
                LOAD_X(xi, so);
                ++xi; so = so == 2 * STAGE ? 0 : so + STAGE;
            }
            if (xi < 64 && xi >= ready) digest(__hip_atomic_load(cp, __ATOMIC_RELAXED, __HIP_MEMORY_SCOPE_AGENT));
            if (xi < 64 && xi < ready) {
                LOAD_X(xi, so);
                ++xi; so = so == 2 * STAGE ? 0 : so + STAGE;
                WAIT_V(20);
            } else {
                WAIT_V(0);
            }
            BAR();
        }
        BAR();
        STAMP(4);
        STAMP_OUT();
        return;
    }

    const int wm = wid >> 2, wn = wid & 3;
    u32 call;
    {
        const int unit = b + 208 * wid;
        const bool enc = wid < 5 && unit < 1024;
        const int rg = unit & 31, kp = unit >> 5;
        u32x4 d[4];
        if (enc) {
            const __amdgpu_buffer_rsrc_t rsX = mk_rsrc(P.x, 512u * KD * 2u);
            const u32 xo = (u32)((rg * 16 + (lane & 15)) * KD + kp * 128 + (lane >> 4) * 8) * 2u;
#pragma unroll
            for (int i = 0; i < 4; ++i) d[i] = __builtin_amdgcn_raw_buffer_load_b128(rsX, xo + i * 64, 0, 0);
        }
        const u32 sq = __hip_atomic_load(seq, __ATOMIC_RELAXED, __HIP_MEMORY_SCOPE_AGENT);
        u32* go = (u32*)(P.ws[lane & 7] + OFF_GO + (rank * 48 + (unit < 48 ? unit : 0)) * 128);
        if (enc) {
            const u32 xdo = OFF_XD + (u32)(kp * 4) * 262144u + (u32)(rank * 32 + rg) * 1024u + lane * 16u;
            u16x2 mx = {0, 0};
#pragma unroll
            for (int i = 0; i < 4; ++i)
#pragma unroll
                for (int e = 0; e < 4; ++e) mx = __builtin_elementwise_max(mx, __builtin_bit_cast(u16x2, d[i][e] & 0x7FFF7FFFu));
            u32 base = wave_max_u32(mx.x > mx.y ? mx.x : mx.y) >> 8;
            STAMP(7);
            base = base < 7u ? 7u : base;
            const u32 base4 = base * 0x01010101u;
            u32x4 lo[2], nb;
            u32 exc = 0;
#pragma unroll
            for (int i = 0; i < 4; ++i) {
                const u32 la = __builtin_amdgcn_perm(d[i][1], d[i][0], 0x06040200u), lb = __builtin_amdgcn_perm(d[i][3], d[i][2], 0x06040200u);
                const u32 ha = __builtin_amdgcn_perm(d[i][1], d[i][0], 0x07050301u), hb = __builtin_amdgcn_perm(d[i][3], d[i][2], 0x07050301u);
                const u32 ca = base4 - (ha & 0x7F7F7F7Fu), cb = base4 - (hb & 0x7F7F7F7Fu);
                exc |= ca | cb;
                const u32 ta = (ha & 0x80808080u) | (ca & 0x07070707u), tb = (hb & 0x80808080u) | (cb & 0x07070707u);
                lo[i >> 1][(i & 1) * 2] = la; lo[i >> 1][(i & 1) * 2 + 1] = lb;
                nb[i] = (ta | __builtin_amdgcn_alignbit(tb, tb, 28)) ^ 0x77777777u;
            }
            const u32 pko = OFF_PK + (u32)unit * 3072u + lane * 16u;
            __builtin_amdgcn_raw_buffer_store_b128(lo[0], rsW, pko, 0, AUX_SYS);
            __builtin_amdgcn_raw_buffer_store_b128(lo[1], rsW, pko, 1024, AUX_SYS);
            __builtin_amdgcn_raw_buffer_store_b128(nb, rsW, pko, 2048, AUX_SYS);
            u32 hw = lane == 0 ? base - 7u : 0u;
            if (__builtin_amdgcn_ballot_w64((exc & 0x78787878u) != 0u) != 0ull) {
                u32 n = 0;
#pragma unroll
                for (int i = 0; i < 4; ++i)
#pragma unroll
                    for (int e = 0; e < 8; ++e) {
                        const u32 v = (d[i][e >> 1] >> ((e & 1) * 16)) & 0xFFFFu;
                        u64 m = __builtin_amdgcn_ballot_w64(((v >> 8) & 0x7Fu) + 7u < base);
                        while (m != 0ull && n < 8u) {
                            const int l = __builtin_ctzll(m);
                            m &= m - 1;
                            ++n;
                            const u32 ent = ((u32)(i * 512 + l * 8 + e) << 16) | (u32)__builtin_amdgcn_readlane((int)v, l);
                            hw = (u32)lane == n ? ent : hw;
                        }
                    }
                if (lane == 0) hw |= (n > 7u ? 255u : n) << 8;
                if (n > 7u) {
#pragma unroll
                    for (int i = 0; i < 4; ++i) __builtin_amdgcn_raw_buffer_store_b128(d[i], rsW, xdo, i * 262144, AUX_SYS);
                }
            }
            if (lane < 8) __builtin_amdgcn_raw_buffer_store_b32(hw, rsW, OFF_HDR + (u32)unit * 32u + lane * 4u, 0, AUX_SYS);
            STAMP(8);
            WAIT_V(0);
            STAMP(9);
            call = rfl(sq) + 1u;
            if (unit < 48) { if (lane < NR && lane != rank) __hip_atomic_store(go, call, __ATOMIC_RELAXED, __HIP_MEMORY_SCOPE_SYSTEM); }
            else if (lane == 0) __hip_atomic_store((u32*)(ws + OFF_TAG) + (unit % 48) * 32 + unit / 48, call, __ATOMIC_RELAXED, __HIP_MEMORY_SCOPE_SYSTEM);
            STAMP(1);
#pragma unroll
            for (int i = 0; i < 4; ++i) __builtin_amdgcn_raw_buffer_store_b128(d[i], rsW, xdo, i * 262144, AUX_SYS);
            WAIT_V(0);
            if (lane == 0) __hip_atomic_store((u32*)(ws + OFF_LOC) + unit, call, __ATOMIC_RELAXED, __HIP_MEMORY_SCOPE_AGENT);
        } else {
            call = rfl(sq) + 1u;
        }
        if (lane == 0) __hip_atomic_store(seq, call, __ATOMIC_RELAXED, __HIP_MEMORY_SCOPE_AGENT);
    }

    u32 pa = lds0 + wm * 5120 + lane * 16, pb[4];
    {
        const int g = lane >> 4, q = (lane >> 2) & 3, p = lane & 3;
#pragma unroll
        for (int j = 0; j < 4; ++j) {
            const int jj = j ^ (p & 1);
            pb[j] = lds0 + 20480 + (8 * g + q) * 512 + (((wn * 8 + 2 * p + (jj >> 1)) ^ (q | ((g & 1) << 3))) << 4) + (jj & 1) * 8;
        }
    }
    f32x4 acc[5][4];
#pragma unroll
    for (int m = 0; m < 5; ++m)
#pragma unroll
        for (int j = 0; j < 4; ++j) acc[m][j] = (f32x4){0.f, 0.f, 0.f, 0.f};

    s16x4 f0b[8], f1b[8];
    bf16x8 f0a[5], f1a[5];
#define RD(i, fa, fb, h) do { \
        if ((i) < 8) fb[(i) & 7] = __builtin_amdgcn_ds_read_tr16_b64_v4i16((LAS s16x4*)(size_t)(pb[((i) & 7) >> 1] + (h) * 16384 + ((i) & 1) * 2048)); \
        else fa[(i) < 8 ? 0 : (i) - 8] = *(LAS bf16x8*)(size_t)(pa + (h) * 10240 + ((i) - 8) * 1024); } while (0)
#define MM(i, fa, fb) acc[(i) >> 2][(i) & 3] = __builtin_amdgcn_mfma_f32_16x16x32_bf16( \
        __builtin_bit_cast(bf16x8, __builtin_shufflevector(fb[((i) & 3) * 2], fb[((i) & 3) * 2 + 1], 0, 1, 2, 3, 4, 5, 6, 7)), fa[(i) >> 2], acc[(i) >> 2][(i) & 3], 0, 0, 0)
#define VADD(v, s) asm volatile("v_add_u32 %0, %1, %0" : "+v"(v) : "s"(s))
    BAR();
    STAMP(3);
#pragma unroll
    for (int i = 0; i < 13; ++i) RD(i, f0a, f0b, 0);

    int ph = 0;
    for (int t = 0; t < 64; ++t) {
        SCHED();
        int dlt;
#pragma unroll
        for (int i = 0; i < 20; ++i) {
            MM(i, f0a, f0b);
            SCHED();
            if (i < 13) RD(i, f1a, f1b, 1);
            if (i == 13) { dlt = ph == 2 ? -2 * STAGE : STAGE; asm volatile("" : "+s"(dlt)); }
            if (i == 14) { ph = ph == 2 ? 0 : ph + 1; asm volatile("" : "+s"(ph)); VADD(pa, dlt); }
            if (i >= 15 && i < 19) VADD(pb[i - 15], dlt);
            SCHED();
        }
        WAIT_L(0);
        BAR();
        SCHED();
#pragma unroll
        for (int i = 0; i < 20; ++i) {
            MM(i, f1a, f1b);
            SCHED();
            if (i < 13) RD(i, f0a, f0b, 0);
            SCHED();
        }
    }
    STAMP(4);

    float mx = 0.f;
#pragma unroll
    for (int m = 0; m < 5; ++m)
#pragma unroll
        for (int j = 0; j < 4; ++j)
#pragma unroll
            for (int r = 0; r < 4; ++r) mx = fmaxf(mx, acc[m][j][r]);
    LAS u32* misc = (LAS u32*)(smem + LDS_MISC + 512);
    {
        const u32 wmx = wave_max_u32(__float_as_uint(mx));
        if (lane == 0) misc[wid] = wmx;
    }
    WAIT_L(0);
    BAR();
    if (wid == 0) {
        u32 m = misc[lane & 7];
        m = wave_max_u32(m);
        if (lane == 0) __hip_atomic_store((u64*)(ws + OFF_LMAX) + T, ((u64)call << 32) | m, __ATOMIC_RELAXED, __HIP_MEMORY_SCOPE_AGENT);
        const u64 t0 = __builtin_amdgcn_s_memrealtime();
        u32 g = 0;
        for (;;) {
            const u64 v = __hip_atomic_load((u64*)(ws + OFF_GMAX + (lane & 7) * 128), __ATOMIC_RELAXED, __HIP_MEMORY_SCOPE_SYSTEM);
            g = (u32)v;
            if (__builtin_amdgcn_ballot_w64((u32)(v >> 32) == call) == ~0ull) break;
            if (give_up(tmo, call, t0)) break;
            __builtin_amdgcn_s_sleep(1);
        }
        g = wave_max_u32(g);
        const u32 bad = rfl(__hip_atomic_load(tmo, __ATOMIC_RELAXED, __HIP_MEMORY_SCOPE_AGENT)) == call;
        if (lane == 0) { misc[8] = g; misc[9] = bad; }
    }
    WAIT_L(0);
    BAR();
    STAMP(5);
    const float amax = __uint_as_float(misc[8]);
    const bool bad = misc[9] != 0u;
    const float inv = 1.0f / (amax / 448.0f);
    {
        const int g = lane >> 4;
        char* op = (char*)P.out + (size_t)(rg0 * 16 + wm * 80 + (lane & 15)) * NL + ct * 256 + wn * 64 + g * 16;
#pragma unroll
        for (int m = 0; m < 5; ++m) {
            u32 q[4];
#pragma unroll
            for (int j = 0; j < 4; ++j) {
                int t = __builtin_amdgcn_cvt_pk_fp8_f32(fmaxf(acc[m][j][0], 0.f) * inv, fmaxf(acc[m][j][1], 0.f) * inv, 0, false);
                t = __builtin_amdgcn_cvt_pk_fp8_f32(fmaxf(acc[m][j][2], 0.f) * inv, fmaxf(acc[m][j][3], 0.f) * inv, t, true);
                q[j] = (u32)t;
            }
            const bool sw = g & 1;
            u32x4 o;
            o[0] = sw ? q[1] : q[0]; o[1] = sw ? q[0] : q[1]; o[2] = sw ? q[3] : q[2]; o[3] = sw ? q[2] : q[3];
            if (bad) o = (u32x4){0x7F7F7F7Fu, 0x7F7F7F7Fu, 0x7F7F7F7Fu, 0x7F7F7F7Fu};
            *(u32x4*)(op + (size_t)m * 16 * NL) = o;
        }
    }
    STAMP(6);
    STAMP_OUT();
}

}

extern "C" void kernel_launch(int rank, int nranks, ncclComm_t comm,
                              hipStream_t stream,
                              void* const* d_in, const int* in_sizes, int n_in,
                              void* d_out, int out_size,
                              const int* in_dtypes, int out_dtype,
                              void* const* d_out_all,
                              void* const* d_ws_all, size_t ws_bytes) {
    using namespace agk;
    if (nranks != NR || (unsigned)rank >= (unsigned)NR || ws_bytes < WS_USED || n_in < 2 ||
        in_sizes[0] != 512 * KD || in_sizes[1] != KD * NL || out_size != 4096 * NL) {
        fprintf(stderr, "unexpected problem shape\n");
        return;
    }
    static bool attr_done = false;
    if (!attr_done) {
        HIP_CHECK(hipFuncSetAttribute((const void*)ag_gemm, hipFuncAttributeMaxDynamicSharedMemorySize, LDS_BYTES));
        attr_done = true;
    }
    Params P;
    P.x = d_in[0]; P.w = d_in[1]; P.out = d_out;
    for (int p = 0; p < NR; ++p) P.ws[p] = (char*)d_ws_all[p];
    P.rank = rank; P.pad = 0;
    hipLaunchKernelGGL(ag_gemm, dim3(GRID), dim3(THREADS), LDS_BYTES, stream, P);
    HIP_CHECK(hipGetLastError());
}
```

```cpp
#include <hip/hip_runtime.h>
#include <rccl/rccl.h>
#include <cstdio>

#define HIP_CHECK(expr) do { hipError_t _e = (expr); if (_e != hipSuccess) { \
    fprintf(stderr, "HIP error %s at %s:%d\n", hipGetErrorString(_e), __FILE__, __LINE__); \
    return; } } while (0)

namespace agk {

typedef unsigned u32;
typedef unsigned long long u64;
typedef u32 u32x4 __attribute__((ext_vector_type(4)));
typedef float f32x4 __attribute__((ext_vector_type(4)));
typedef __bf16 bf16x8 __attribute__((ext_vector_type(8)));
typedef short s16x4 __attribute__((ext_vector_type(4)));
typedef unsigned short u16x2 __attribute__((ext_vector_type(2)));
#define LAS __attribute__((address_space(3)))

constexpr int NR = 8;
constexpr int KD = 4096;
constexpr int NL = 2048;
constexpr int GRID = 256, THREADS = 768;
#define NPX 8
#define CPX 2
#define NPC (NPX * CPX)
#define PWAVES 3
#define DWAVES 12
constexpr int NCOMM = 48;
constexpr int NPUSH = NPC * PWAVES;
constexpr int NDEC = (NCOMM - NPC) * DWAVES;
constexpr int NDECU = NDEC / 8 * 8;
__host__ __device__ constexpr int ncomm_of(int x) { return NPX == 8 ? 6 : x < NPX ? CPX : (NCOMM - NPC + (7 - x)) / (NPX == 8 ? 1 : 8 - NPX); }
__host__ __device__ constexpr int ncomm_before(int x) { int n = 0; for (int i = 0; i < x; ++i) n += ncomm_of(i); return n; }
static_assert(ncomm_before(8) == NCOMM, "mover split");
constexpr int STAGE = 52 * 1024;
constexpr int LDS_MISC = 3 * STAGE;
constexpr int LDS_BYTES = LDS_MISC + 1024;

constexpr u32 OFF_SEQ = 0;
constexpr u32 OFF_TMO = 16384;
constexpr u32 OFF_LMAX = 24576;
constexpr u32 OFF_GMAX = 28672;
constexpr u32 OFF_TKT = 32768;
constexpr u32 OFF_DBG = 128u << 10;
constexpr u32 OFF_HDR = 512u << 10;
constexpr u32 OFF_DONE = 1u << 20;
constexpr u32 OFF_XD = 4u << 20;
constexpr u32 OFF_PK = 36u << 20;
constexpr u32 WS_USED = 68u << 20;

constexpr u64 DEADLINE_TICKS = 40000000ull;
constexpr int AUX_SYS = 17, AUX_AGT = 16;

struct Params {
    const void* x;
    const void* w;
    void* out;
    char* ws[NR];
    int rank;
    int pad;
};

#define WAIT_V(n) asm volatile("s_waitcnt vmcnt(" #n ")" ::: "memory")
#define WAIT_L(n) asm volatile("s_waitcnt lgkmcnt(" #n ")" ::: "memory")
#define BAR() do { asm volatile("" ::: "memory"); __builtin_amdgcn_s_barrier(); asm volatile("" ::: "memory"); } while (0)
#define SCHED() __builtin_amdgcn_sched_barrier(0)

__device__ __forceinline__ u32 rfl(u32 v) { return __builtin_amdgcn_readfirstlane(v); }
__device__ __forceinline__ __amdgpu_buffer_rsrc_t mk_rsrc(const void* p, u32 bytes) {
    return __builtin_amdgcn_make_buffer_rsrc((void*)p, 0, (int)bytes, 0x00020000);
}
__device__ __forceinline__ u32 umax(u32 a, u32 b) { return a > b ? a : b; }
__device__ __forceinline__ u32 wave_max_u32(u32 v) {
    v = umax(v, (u32)__builtin_amdgcn_update_dpp(0, (int)v, 0x111, 0xf, 0xf, true));
    v = umax(v, (u32)__builtin_amdgcn_update_dpp(0, (int)v, 0x112, 0xf, 0xf, true));
    v = umax(v, (u32)__builtin_amdgcn_update_dpp(0, (int)v, 0x114, 0xf, 0xf, true));
    v = umax(v, (u32)__builtin_amdgcn_update_dpp(0, (int)v, 0x118, 0xf, 0xf, true));
    return umax(umax((u32)__builtin_amdgcn_readlane((int)v, 15), (u32)__builtin_amdgcn_readlane((int)v, 31)),
                umax((u32)__builtin_amdgcn_readlane((int)v, 47), (u32)__builtin_amdgcn_readlane((int)v, 63)));
}
__device__ __forceinline__ bool give_up(u32* tmo, u32 call, u64 t0) {
    const u64 dt = __builtin_amdgcn_s_memrealtime() - t0;
    if (dt < 20000ull) return false;
    const u32 t = rfl(__hip_atomic_load(tmo, __ATOMIC_RELAXED, __HIP_MEMORY_SCOPE_AGENT));
    if (t == call) return true;
    if (dt > DEADLINE_TICKS) {
        __hip_atomic_store(tmo, call, __ATOMIC_RELAXED, __HIP_MEMORY_SCOPE_AGENT);
        return true;
    }
    return false;
}

#define DEC_SLEEP 2
#define XPRIO 3
#define WPRIO 2
#define POLL_SLEEP 2
#define STAMP(i) do {} while (0)
#define TB() do {} while (0)
#define TE(i) do {} while (0)
#define TFIX(i) do {} while (0)
#define STAMP_OUT() do {} while (0)

#define XL0 ((u32)((lane >> 3) * (KD * 2) + (((lane & 7) ^ (lane >> 4)) << 4)))
#define XL1 ((u32)((8 + (lane >> 3)) * (KD * 2) + (((lane & 7) ^ (4 + (lane >> 4))) << 4)))
#define XPIECE(i) ((u32)(((i) >> 1) * 524288 + ((i) & 1) * 1024))

__global__ __launch_bounds__(THREADS, 3) void ag_gemm(Params P) {
    extern __shared__ __attribute__((aligned(16))) char smem_[];
    LAS char* smem = (LAS char*)smem_;
    const int lane = threadIdx.x & 63;
    const int wid = (int)rfl(threadIdx.x >> 6);
    const int b = blockIdx.x;
    const int rank = P.rank & 7;
    char* ws = P.ws[rank];
    u32* tmo = (u32*)(ws + OFF_TMO);
    STAMP(0);
    const __amdgpu_buffer_rsrc_t rsW = mk_rsrc(ws, WS_USED);
    const int xcd = b & 7, idx = b >> 3;
    u32* seq = (u32*)(ws + OFF_SEQ) + b * 16 + wid;

    int ncx = 0, ncb = 0;
#pragma unroll
    for (int i = 0; i < 8; ++i) if (xcd == i) { ncx = ncomm_of(i); ncb = ncomm_before(i); }
    if (idx >= 32 - ncx) {
        const int ci = NPX == 8 ? (idx - 26) * 8 + xcd : ncb + idx - (32 - ncx);
        if (ci < NPC) {
            if (wid >= PWAVES) return;
            const int pi = wid * NPC + ci;
            const __amdgpu_buffer_rsrc_t rsX = mk_rsrc(P.x, 512u * KD * 2u);
            const u32 xl0 = XL0, xl1 = XL1;
            const u32 sq = __hip_atomic_load(seq, __ATOMIC_RELAXED, __HIP_MEMORY_SCOPE_AGENT);
            u64* hp = (u64*)(P.ws[(rank + 1 + (lane >> 3)) & 7] + OFF_HDR + rank * 65536) + (lane & 7);
            __amdgpu_buffer_rsrc_t rsP[7];
#pragma unroll
            for (int p = 0; p < 7; ++p) rsP[p] = mk_rsrc(P.ws[(rank + 1 + p) & 7] + OFF_PK + rank * (4u << 20), 4u << 20);
            u32* tkt = (u32*)(ws + OFF_TKT);
            u32 call = 0, tbase = 0;
            int done = -1, cur = -1, nxt = pi;
            u32x4 pk[4];
            u32 hw = 0, hw_done = 0;
            bool raw = false;
            for (;;) {
                u32 tk;
                asm volatile("s_cmp_lt_u32 %4, 1024\n\ts_cbranch_scc0 1f\n\ts_mov_b64 exec, 1\n\tglobal_atomic_add %0, %1, %2, %3 sc0\n\ts_mov_b64 exec, -1\n1:"
                             : "=&v"(tk) : "v"(0u), "v"(1u), "s"(tkt), "s"(cur) : "memory", "scc");
                u32x4 d[4];
                asm volatile("s_cmp_lt_u32 %8, 1024\n\ts_cbranch_scc0 1f\n\t"
                             "buffer_load_dwordx4 %0, %4, %6, %7 offen\n\tbuffer_load_dwordx4 %1, %5, %6, %7 offen\n\t"
                             "buffer_load_dwordx4 %2, %4, %6, %7 offen offset:128\n\tbuffer_load_dwordx4 %3, %5, %6, %7 offen offset:128\n1:"
                             : "=&v"(d[0]), "=&v"(d[1]), "=&v"(d[2]), "=&v"(d[3])
                             : "v"(xl0), "v"(xl1), "s"(rsX), "s"(((nxt & 31) * 16 * KD + (nxt >> 5) * 128) * 2), "s"(nxt) : "memory", "scc");
                if (done >= 0 && lane < 56) __hip_atomic_store(hp + done * 8, ((u64)call << 32) | hw_done, __ATOMIC_RELAXED, __HIP_MEMORY_SCOPE_SYSTEM);
                if (cur >= 1024) break;
                if (cur >= 0) {
#pragma unroll
                    for (int p = 0; p < 7; ++p)
#pragma unroll
                        for (int i = 0; i < 3; ++i) __builtin_amdgcn_raw_buffer_store_b128(pk[i], rsP[p], lane * 16u + i * 1024u, cur * 4096, AUX_SYS);
                    if (raw) {
#pragma unroll
                        for (int p = 0; p < 7; ++p) __builtin_amdgcn_raw_buffer_store_b128(pk[3], rsP[p], lane * 16u + 3072u, cur * 4096, AUX_SYS);
                    }
                }
                done = cur; hw_done = hw;
                asm volatile("s_waitcnt vmcnt(21)\n\ts_cmp_lt_i32 %4, 0\n\ts_cbranch_scc0 1f\n\ts_waitcnt vmcnt(0)\n1:"
                             : "+v"(d[0]), "+v"(d[1]), "+v"(d[2]), "+v"(d[3]) : "s"(cur) : "memory", "scc");
                if (nxt < 1024) {
                    u16x2 mx = {0, 0};
#pragma unroll
                    for (int i = 0; i < 4; ++i)
#pragma unroll
                        for (int e = 0; e < 4; ++e) mx = __builtin_elementwise_max(mx, __builtin_bit_cast(u16x2, d[i][e] & 0x7FFF7FFFu));
                    u32 base = wave_max_u32(mx.x > mx.y ? mx.x : mx.y) >> 8;
                    base = base < 7u ? 7u : base;
                    const u32 base4 = base * 0x01010101u;
                    u32 exc = 0;
#pragma unroll
                    for (int i = 0; i < 4; ++i) {
                        const u32 la = __builtin_amdgcn_perm(d[i][1], d[i][0], 0x06040200u), lb = __builtin_amdgcn_perm(d[i][3], d[i][2], 0x06040200u);
                        const u32 ha = __builtin_amdgcn_perm(d[i][1], d[i][0], 0x07050301u), hb = __builtin_amdgcn_perm(d[i][3], d[i][2], 0x07050301u);
                        const u32 ca = base4 - (ha & 0x7F7F7F7Fu), cb = base4 - (hb & 0x7F7F7F7Fu);
                        exc |= ca | cb;
                        const u32 ta = (ha & 0x80808080u) | (ca & 0x07070707u), tb = (hb & 0x80808080u) | (cb & 0x07070707u);
                        pk[i >> 1][(i & 1) * 2] = la; pk[i >> 1][(i & 1) * 2 + 1] = lb;
                        pk[2][i] = (ta | __builtin_amdgcn_alignbit(tb, tb, 28)) ^ 0x77777777u;
                    }
                    hw = (lane & 7) == 0 ? base - 7u : ~0u;
                    raw = false;
                    u64 m = __builtin_amdgcn_ballot_w64((exc & 0x78787878u) != 0u);
                    if (m != 0ull) {
                        u32 n = 0;
                        const u32 lim = (base - 7u) << 8;
                        do {
                            const int l = __builtin_ctzll(m);
                            m &= m - 1;
#pragma unroll
                            for (int i = 0; i < 16; ++i) {
                                const u32 w = (u32)__builtin_amdgcn_readlane((int)d[i >> 2][i & 3], l);
                                const u32 pos = (u32)((i >> 2) * 512 + l * 8 + (i & 3) * 2);
                                if ((w & 0x7FFFu) < lim) { ++n; hw = (u32)(lane & 7) == n ? (pos << 16) | (w & 0xFFFFu) : hw; }
                                if (((w >> 16) & 0x7FFFu) < lim) { ++n; hw = (u32)(lane & 7) == n ? ((pos + 1u) << 16) | (w >> 16) : hw; }
                            }
                        } while (m != 0ull && n < 8u);
                        if (n > 7u) {
                            raw = true;
                            hw = (lane & 7) == 0 ? 256u : ~0u;
#pragma unroll
                            for (int i = 0; i < 4; ++i) pk[i] = d[i];
                        }
                    }
                }
                asm volatile("s_waitcnt vmcnt(0)" : "+v"(tk) :: "memory");
                const bool first = cur < 0;
                cur = nxt;
                if (first) {
                    call = rfl(sq) + 1u;
                    tbase = (call - 1u) * 1024u - 2u * NPUSH;
                    if (lane == 0) __hip_atomic_store(seq, call, __ATOMIC_RELAXED, __HIP_MEMORY_SCOPE_AGENT);
                    nxt = pi + NPUSH;
                    STAMP(2);
                } else {
                    nxt = (int)(rfl(tk) - tbase);
                }
            }
            STAMP(3);
            STAMP_OUT();
            return;
        }
        if (wid >= DWAVES) return;
        const u32 call = rfl(__hip_atomic_load(seq, __ATOMIC_RELAXED, __HIP_MEMORY_SCOPE_AGENT)) + 1u;
        if (lane == 0) __hip_atomic_store(seq, call, __ATOMIC_RELAXED, __HIP_MEMORY_SCOPE_AGENT);
        {
            u32* pend = nullptr;
            const __amdgpu_buffer_rsrc_t rsX = mk_rsrc(P.x, 512u * KD * 2u);
            const u32 xl0 = XL0, xl1 = XL1;
            int rot = 0;
            const int dw = wid * (NCOMM - NPC) + ci - NPC;
            for (int q = dw < NDECU ? dw : 8 * 1024; q < 8 * 1024; q += NDECU, ++rot) {
                const int v = q >> 3, src = (q + rot) & 7;
                const u32 xdb = OFF_XD + (u32)(v >> 5) * 1048576u + (u32)(src * 64 + (v & 31) * 2) * 1024u;
                if (src == rank) {
                    u32x4 d[4];
#pragma unroll
                    for (int i = 0; i < 4; ++i)
                        d[i] = __builtin_amdgcn_raw_buffer_load_b128(rsX, ((i & 1) ? xl1 : xl0) + (i >> 1) * 128, ((v & 31) * 16 * KD + (v >> 5) * 128) * 2, 0);
                    asm volatile("s_waitcnt vmcnt(0)" : "+v"(d[0]), "+v"(d[1]), "+v"(d[2]), "+v"(d[3]) :: "memory");
                    if (pend && lane == 0) __hip_atomic_store(pend, call, __ATOMIC_RELAXED, __HIP_MEMORY_SCOPE_AGENT);
#pragma unroll
                    for (int i = 0; i < 4; ++i) __builtin_amdgcn_raw_buffer_store_b128(d[i], rsW, lane * 16u, xdb + XPIECE(i), AUX_AGT);
                    asm volatile("" ::: "memory");
                    pend = (u32*)(ws + OFF_DONE + ((v >> 5) * 256 + src * 32 + (v & 31)) * 128);
                    continue;
                }
                const u64* hp = (const u64*)(ws + OFF_HDR + src * 65536) + v * 8 + (lane & 7);
                u32 hw, h0;
                bool dead = false;
                {
                    const u64 t0 = __builtin_amdgcn_s_memrealtime();
                    for (;;) {
                        const u64 g = __hip_atomic_load(hp, __ATOMIC_RELAXED, __HIP_MEMORY_SCOPE_SYSTEM);
                        hw = (u32)g; h0 = rfl(hw);
                        u32 ok = (u32)__builtin_amdgcn_ballot_w64((u32)(g >> 32) == call);
                        asm volatile("s_waitcnt vmcnt(0)" : "+s"(ok), "+s"(h0), "+v"(hw) :: "memory");
                        if (pend) {
                            if (lane == 0) __hip_atomic_store(pend, call, __ATOMIC_RELAXED, __HIP_MEMORY_SCOPE_AGENT);
                            pend = nullptr;
                        }
                        if (ok == ~0u) break;
                        if (give_up(tmo, call, t0)) { dead = true; break; }
                        __builtin_amdgcn_s_sleep(DEC_SLEEP);
                    }
                }
                if (dead) break;
                asm volatile("" ::: "memory");
                const u32 pk = OFF_PK + (u32)src * (4u << 20) + (u32)v * 4096u;
                u32x4 in[4];
#pragma unroll
                for (int i = 0; i < 3; ++i) in[i] = __builtin_amdgcn_raw_buffer_load_b128(rsW, lane * 16u + i * 1024u, pk, AUX_SYS);
                if (h0 < 256u) {
                    const u32 low4 = h0 * 0x01010101u;
                    const u32x4 l0 = in[0], l1 = in[1], nb = in[2];
#pragma unroll
                    for (int i = 0; i < 4; ++i) {
                        const u32 n = nb[i];
                        const u32 ha = (n & 0x87878787u) + low4, hb = (__builtin_amdgcn_alignbit(n, n, 4) & 0x87878787u) + low4;
                        const u32 la = i < 2 ? l0[(i & 1) * 2] : l1[(i & 1) * 2], lb = i < 2 ? l0[(i & 1) * 2 + 1] : l1[(i & 1) * 2 + 1];
                        in[i][0] = __builtin_amdgcn_perm(ha, la, 0x05010400u);
                        in[i][1] = __builtin_amdgcn_perm(ha, la, 0x07030602u);
                        in[i][2] = __builtin_amdgcn_perm(hb, lb, 0x05010400u);
                        in[i][3] = __builtin_amdgcn_perm(hb, lb, 0x07030602u);
                    }
                } else {
                    in[3] = __builtin_amdgcn_raw_buffer_load_b128(rsW, lane * 16u + 3072u, pk, AUX_SYS);
                    asm volatile("s_waitcnt vmcnt(0)" : "+v"(in[0]), "+v"(in[1]), "+v"(in[2]), "+v"(in[3]) :: "memory");
                }
#pragma unroll
                for (int i = 0; i < 4; ++i) __builtin_amdgcn_raw_buffer_store_b128(in[i], rsW, lane * 16u, xdb + XPIECE(i), AUX_AGT);
                {
                    asm volatile("" ::: "memory");
                    if (lane >= 1 && lane < 8 && hw != ~0u) {
                        const u32 pos = hw >> 16;
                        __builtin_amdgcn_raw_buffer_store_b16((unsigned short)(hw & 0xFFFFu), rsW, (pos >> 10) * 524288u + (pos & 1023u) * 2u, xdb, AUX_AGT);
                    }
                }
                asm volatile("" ::: "memory");
                pend = (u32*)(ws + OFF_DONE + ((v >> 5) * 256 + src * 32 + (v & 31)) * 128);
            }
            WAIT_V(0);
            if (pend && lane == 0) __hip_atomic_store(pend, call, __ATOMIC_RELAXED, __HIP_MEMORY_SCOPE_AGENT);
            STAMP(3);
        }
        if (ci == 47 && wid == 0) {
            u64* dst = (u64*)(P.ws[lane & 7] + OFF_GMAX + rank * 128);
            const u64 t0 = __builtin_amdgcn_s_memrealtime();
            u32 m = 0;
            for (;;) {
                const u32x4 a = __builtin_amdgcn_raw_buffer_load_b128(rsW, OFF_LMAX + lane * 16, 0, AUX_AGT);
                const u32x4 c = __builtin_amdgcn_raw_buffer_load_b128(rsW, OFF_LMAX + 1024 + lane * 16, 0, AUX_AGT);
                const bool ok = a[1] == call && a[3] == call && (lane >= 40 || (c[1] == call && c[3] == call));
                m = a[0] > a[2] ? a[0] : a[2];
                if (lane < 40) { m = m > c[0] ? m : c[0]; m = m > c[2] ? m : c[2]; }
                if (__builtin_amdgcn_ballot_w64(ok) == ~0ull) break;
                if (give_up(tmo, call, t0)) break;
                __builtin_amdgcn_s_sleep(1);
            }
            STAMP(7);
            m = wave_max_u32(m);
            if (lane < NR) __hip_atomic_store(dst, ((u64)call << 32) | m, __ATOMIC_RELAXED, __HIP_MEMORY_SCOPE_SYSTEM);
            STAMP(8);
        }
        STAMP_OUT();
        return;
    }

    const int T = xcd * 32 - ncb + idx;
    const int ct = (T / 104) * 4 + (T % 104 & 3), rt = (T % 104) >> 2;
    const int rg0 = rt * 10 < 246 ? rt * 10 : 246;
    const u32 lds0 = (u32)(size_t)smem;
#define DMA(ldsaddr, voff, rs, soff, mods) asm volatile("s_mov_b32 m0, %0\n\ts_nop 0\n\tbuffer_load_dwordx4 %1, %2, %3 offen" mods " lds" \
        :: "s"(ldsaddr), "v"(voff), "s"(rs), "s"(soff) : "memory")
#define DMA_SAME_M0(voff, rs, soff, mods) asm volatile("buffer_load_dwordx4 %0, %1, %2 offen" mods " lds" :: "v"(voff), "s"(rs), "s"(soff) : "memory")

    if (wid == 11) return;
    if (wid >= 9) {
        __builtin_amdgcn_s_setprio(WPRIO);
        const int h = wid - 9;
        const __amdgpu_buffer_rsrc_t rsB = mk_rsrc(P.w, (u32)KD * NL * 2u);
        u32 vB[16];
#pragma unroll
        for (int i = 0; i < 16; ++i) {
            const int row = i * 2 + (lane >> 5);
            vB[i] = (u32)((h * 32 + row) * (NL * 2) + ct * 512 + (((lane & 31) ^ ((row & 3) | (row & 8))) << 4));
        }
        const u32 lB = lds0 + 20480 + h * 16384;
#define LOAD_W(kt, so) do { _Pragma("unroll") for (int i = 0; i < 16; ++i) DMA(lB + (so) + i * 1024, vB[i], rsB, (kt) * 262144, ""); } while (0)
        LOAD_W(0, 0);
        LOAD_W(1, STAGE);
        WAIT_V(16);
        BAR();
        int so = 2 * STAGE;
        for (int t = 0; t < 62; ++t) {
            { TB(); LOAD_W(t + 2, so); TE(5); }
            so = so == 2 * STAGE ? 0 : so + STAGE;
            { TB(); WAIT_V(16); TE(6); }
            { TB(); BAR(); TE(7); }
        }
        WAIT_V(0);
        BAR();
        TFIX(5); TFIX(6); TFIX(7);
        STAMP_OUT();
        return;
    }

    if (wid == 8) {
        __builtin_amdgcn_s_setprio(XPRIO);
        const u32 call = rfl(__hip_atomic_load(seq, __ATOMIC_RELAXED, __HIP_MEMORY_SCOPE_AGENT)) + 1u;
        if (lane == 0) __hip_atomic_store(seq, call, __ATOMIC_RELAXED, __HIP_MEMORY_SCOPE_AGENT);
        const u32* cp = (const u32*)(ws + OFF_DONE + (rg0 + ((lane & 15) < 10 ? (lane & 15) : 0)) * 128);
        const u32 vA = (u32)lane * 16u;
        const u32 gA = OFF_XD + (u32)rg0 * 2048u;
        int ready = 0;
        auto look = [&]() {
            const int c = (ready >> 1) + (lane >> 4);
            const u32 f = __hip_atomic_load(cp + (c < 31 ? c : 31) * 8192, __ATOMIC_RELAXED, __HIP_MEMORY_SCOPE_AGENT);
            const u64 miss = __builtin_amdgcn_ballot_w64(f != call);
            ready += miss == 0ull ? 8 : 2 * (__builtin_ctzll(miss) >> 4);
        };
        auto need = [&](int kt) {
            if (kt < ready) return;
            const u64 t0 = __builtin_amdgcn_s_memrealtime();
            for (;;) {
                look();
                if (kt < ready) break;
                if (give_up(tmo, call, t0)) { ready = 64; break; }
                __builtin_amdgcn_s_sleep(POLL_SLEEP);
            }
        };
#define LOAD_X4(g, l) do { \
            DMA((l), vA, rsW, (g), " sc1"); DMA_SAME_M0(vA, rsW, (g), " offset:1024 sc1"); DMA_SAME_M0(vA, rsW, (g), " offset:2048 sc1"); DMA_SAME_M0(vA, rsW, (g), " offset:3072 sc1"); } while (0)
#define LOAD_X(kt, so) do { _Pragma("unroll") for (int i = 0; i < 5; ++i) LOAD_X4(gA + (kt) * 524288 + i * 4096, lds0 + (so) + i * 4096); } while (0)
        need(0);
        STAMP(2);
        LOAD_X(0, 0);
        LOAD_X(1, STAGE);
        WAIT_V(20);
        BAR();
        STAMP(3);
        int xi = 2, so = 2 * STAGE;
        for (int t = 0; t < 63; ++t) {
            if (xi == t + 1) {
                { TB(); need(xi); TE(10); }
                LOAD_X(xi, so);
                ++xi; so = so == 2 * STAGE ? 0 : so + STAGE;
            }
            if (xi < 64 && xi >= ready) { TB(); look(); TE(11); }
            if (xi < 64 && xi < ready) {
                LOAD_X(xi, so);
                ++xi; so = so == 2 * STAGE ? 0 : so + STAGE;
                TB(); WAIT_V(20); TE(12);
            } else {
                TB(); WAIT_V(0); TE(13);
            }
            { TB(); BAR(); TE(14); }
        }
        STAMP(4);
        TFIX(10); TFIX(11); TFIX(12); TFIX(13); TFIX(14);
        STAMP_OUT();
        return;
    }

    const int wm = wid >> 2, wn = wid & 3;
    u32 call = 0;
    if (wid == 0) {
        call = rfl(__hip_atomic_load(seq, __ATOMIC_RELAXED, __HIP_MEMORY_SCOPE_AGENT)) + 1u;
        if (lane == 0) __hip_atomic_store(seq, call, __ATOMIC_RELAXED, __HIP_MEMORY_SCOPE_AGENT);
    }

    u32 pa[2], pb[4];
#pragma unroll
    for (int h = 0; h < 2; ++h) pa[h] = lds0 + wm * 10240 + (lane & 15) * 128 + (((4 * h + (lane >> 4)) ^ ((lane >> 1) & 7)) << 4);
    {
        const int g = lane >> 4, q = (lane >> 2) & 3, p = lane & 3;
#pragma unroll
        for (int j = 0; j < 4; ++j) {
            const int jj = j ^ (p & 1);
            pb[j] = lds0 + 20480 + (8 * g + q) * 512 + (((wn * 8 + 2 * p + (jj >> 1)) ^ (q | ((g & 1) << 3))) << 4) + (jj & 1) * 8;
        }
    }
    f32x4 acc[5][4];
#pragma unroll
    for (int m = 0; m < 5; ++m)
#pragma unroll
        for (int j = 0; j < 4; ++j) acc[m][j] = (f32x4){0.f, 0.f, 0.f, 0.f};

    s16x4 f0b[8], f1b[8];
    bf16x8 f0a[5], f1a[5];
#define RD(i, fa, fb, h) do { \
        if ((i) < 8) fb[(i) & 7] = __builtin_amdgcn_ds_read_tr16_b64_v4i16((LAS s16x4*)(size_t)(pb[((i) & 7) >> 1] + (h) * 16384 + ((i) & 1) * 2048)); \
        else fa[(i) < 8 ? 0 : (i) - 8] = *(LAS bf16x8*)(size_t)(pa[h] + ((i) - 8) * 2048); } while (0)
#define MM(i, fa, fb) acc[(i) >> 2][(i) & 3] = __builtin_amdgcn_mfma_f32_16x16x32_bf16( \
        __builtin_bit_cast(bf16x8, __builtin_shufflevector(fb[((i) & 3) * 2], fb[((i) & 3) * 2 + 1], 0, 1, 2, 3, 4, 5, 6, 7)), fa[(i) >> 2], acc[(i) >> 2][(i) & 3], 0, 0, 0)
#define VADD(v, s) asm volatile("v_add_u32 %0, %1, %0" : "+v"(v) : "s"(s))
    BAR();
    STAMP(3);
    if (wm == 0) __builtin_amdgcn_s_setprio(1);
    int ph = 0;
    for (int t = 0; t < 64; ++t) {
        SCHED();
#pragma unroll
        for (int i = 0; i < 13; ++i) RD(i, f0a, f0b, 0);
#pragma unroll
        for (int i = 0; i < 13; ++i) RD(i, f1a, f1b, 1);
        SCHED();
        {
            const int dlt = ph == 2 ? -2 * STAGE : STAGE;
            ph = ph == 2 ? 0 : ph + 1;
            VADD(pa[0], dlt); VADD(pa[1], dlt);
#pragma unroll
            for (int j = 0; j < 4; ++j) VADD(pb[j], dlt);
        }
        const int who = t == 63 ? 2 : wm;
        asm volatile("s_cmp_lg_u32 %0, 0\n\ts_cbranch_scc1 1f\n\ts_waitcnt lgkmcnt(0)\n\ts_barrier\n1:" :: "s"(who) : "memory", "scc");
        SCHED();
#pragma unroll
        for (int i = 0; i < 20; ++i) MM(i, f0a, f0b);
#pragma unroll
        for (int i = 0; i < 20; ++i) MM(i, f1a, f1b);
        SCHED();
        asm volatile("s_cmp_lg_u32 %0, 1\n\ts_cbranch_scc1 1f\n\ts_barrier\n1:" :: "s"(who) : "memory", "scc");
    }
    __builtin_amdgcn_s_setprio(0);
    STAMP(4);
    TFIX(14);

    float mx = 0.f;
#pragma unroll
    for (int m = 0; m < 5; ++m)
#pragma unroll
        for (int j = 0; j < 4; ++j)
#pragma unroll
            for (int r = 0; r < 4; ++r) mx = fmaxf(mx, acc[m][j][r]);
    LAS u32* misc = (LAS u32*)(smem + LDS_MISC + 512);
    {
        const u32 wmx = wave_max_u32(__float_as_uint(mx));
        if (lane == 0) misc[wid] = wmx;
    }
    WAIT_L(0);
    BAR();
    if (wid == 0) {
        u32 m = misc[lane & 7];
        m = wave_max_u32(m);
        if (lane == 0) __hip_atomic_store((u64*)(ws + OFF_LMAX) + T, ((u64)call << 32) | m, __ATOMIC_RELAXED, __HIP_MEMORY_SCOPE_AGENT);
        const u64 t0 = __builtin_amdgcn_s_memrealtime();
        u32 g = 0;
        for (;;) {
            const u64 v = __hip_atomic_load((u64*)(ws + OFF_GMAX + (lane & 7) * 128), __ATOMIC_RELAXED, __HIP_MEMORY_SCOPE_SYSTEM);
            g = (u32)v;
            if (__builtin_amdgcn_ballot_w64((u32)(v >> 32) == call) == ~0ull) break;
            if (give_up(tmo, call, t0)) break;
            __builtin_amdgcn_s_sleep(1);
        }
        g = wave_max_u32(g);
        const u32 bad = rfl(__hip_atomic_load(tmo, __ATOMIC_RELAXED, __HIP_MEMORY_SCOPE_AGENT)) == call;
        if (lane == 0) { misc[8] = g; misc[9] = bad; }
    }
    WAIT_L(0);
    BAR();
    STAMP(5);
    const float amax = __uint_as_float(misc[8]);
    const bool bad = misc[9] != 0u;
    const float inv = 1.0f / (amax / 448.0f);
    {
        const int g = lane >> 4;
        char* op = (char*)P.out + (size_t)(rg0 * 16 + wm * 80 + (lane & 15)) * NL + ct * 256 + wn * 64 + g * 16;
#pragma unroll
        for (int m = 0; m < 5; ++m) {
            u32 q[4];
#pragma unroll
            for (int j = 0; j < 4; ++j) {
                int t = __builtin_amdgcn_cvt_pk_fp8_f32(fmaxf(acc[m][j][0], 0.f) * inv, fmaxf(acc[m][j][1], 0.f) * inv, 0, false);
                t = __builtin_amdgcn_cvt_pk_fp8_f32(fmaxf(acc[m][j][2], 0.f) * inv, fmaxf(acc[m][j][3], 0.f) * inv, t, true);
                q[j] = (u32)t;
            }
            const bool sw = g & 1;
            u32x4 o;
            o[0] = sw ? q[1] : q[0]; o[1] = sw ? q[0] : q[1]; o[2] = sw ? q[3] : q[2]; o[3] = sw ? q[2] : q[3];
            if (bad) o = (u32x4){0x7F7F7F7Fu, 0x7F7F7F7Fu, 0x7F7F7F7Fu, 0x7F7F7F7Fu};
            *(u32x4*)(op + (size_t)m * 16 * NL) = o;
        }
    }
    STAMP(6);
    STAMP_OUT();
}

}

extern "C" void kernel_launch(int rank, int nranks, ncclComm_t comm,
                              hipStream_t stream,
                              void* const* d_in, const int* in_sizes, int n_in,
                              void* d_out, int out_size,
                              const int* in_dtypes, int out_dtype,
                              void* const* d_out_all,
                              void* const* d_ws_all, size_t ws_bytes) {
    using namespace agk;
    if (nranks != NR || (unsigned)rank >= (unsigned)NR || ws_bytes < WS_USED || n_in < 2 ||
        in_sizes[0] != 512 * KD || in_sizes[1] != KD * NL || out_size != 4096 * NL) {
        fprintf(stderr, "unexpected problem shape\n");
        return;
    }
    static bool attr_done = false;
    if (!attr_done) {
        HIP_CHECK(hipFuncSetAttribute((const void*)ag_gemm, hipFuncAttributeMaxDynamicSharedMemorySize, LDS_BYTES));
        attr_done = true;
    }
    Params P;
    P.x = d_in[0]; P.w = d_in[1]; P.out = d_out;
    for (int p = 0; p < NR; ++p) P.ws[p] = (char*)d_ws_all[p];
    P.rank = rank; P.pad = 0;
    hipLaunchKernelGGL(ag_gemm, dim3(GRID), dim3(THREADS), LDS_BYTES, stream, P);
    HIP_CHECK(hipGetLastError());
}
```

```cpp
#include <hip/hip_runtime.h>
#include <rccl/rccl.h>
#include <cstdio>

#define HIP_CHECK(expr) do { hipError_t _e = (expr); if (_e != hipSuccess) { \
    fprintf(stderr, "HIP error %s at %s:%d\n", hipGetErrorString(_e), __FILE__, __LINE__); \
    return; } } while (0)

namespace agk {

typedef unsigned u32;
typedef unsigned long long u64;
typedef u32 u32x4 __attribute__((ext_vector_type(4)));
typedef float f32x4 __attribute__((ext_vector_type(4)));
typedef __bf16 bf16x8 __attribute__((ext_vector_type(8)));
typedef short s16x4 __attribute__((ext_vector_type(4)));
typedef unsigned short u16x2 __attribute__((ext_vector_type(2)));
#define LAS __attribute__((address_space(3)))
#define GAS __attribute__((address_space(1)))

constexpr int NR = 8;
constexpr int KD = 4096;
constexpr int NL = 2048;
constexpr int GRID = 256, THREADS = 768;
#define NPC 20
#define PWAVES 2
#define DWAVES 12
#define NRAWL 0
#define NRAWF 1
#define HDR_FIRST 0
#define STAGGER 20
constexpr int NCOMM = 48;
constexpr int NPUSH = NPC * PWAVES;
constexpr int NDEC = (NCOMM - NPC) * DWAVES;
constexpr int VRAW0 = 16 * NRAWF;
constexpr int VRAW = 1024 - 16 * NRAWL;
constexpr int STAGE = 52 * 1024;
constexpr int LDS_MISC = 3 * STAGE;
constexpr int LDS_BYTES = LDS_MISC + 1024;

constexpr u32 OFF_SEQ = 0;
constexpr u32 OFF_TMO = 16384;
constexpr u32 OFF_LMAX = 24576;
constexpr u32 OFF_GMAX = 28672;
constexpr u32 OFF_TMAX = 65536;
constexpr u32 OFF_TKT = 32768;
constexpr u32 OFF_DBG = 128u << 10;
constexpr u32 OFF_HDR = 512u << 10;
constexpr u32 OFF_DONE = 1u << 20;
constexpr u32 OFF_XD = 4u << 20;
constexpr u32 OFF_PK = 36u << 20;
constexpr u32 WS_USED = 60u << 20;

constexpr u64 DEADLINE_TICKS = 40000000ull;
constexpr int AUX_SYS = 17, AUX_AGT = 16;

struct Params {
    const void* x;
    const void* w;
    void* out;
    char* self;
    char* peer[NR];
    int rank;
    int pad;
};

#define WAIT_V(n) asm volatile("s_waitcnt vmcnt(" #n ")" ::: "memory")
#define WAIT_L(n) asm volatile("s_waitcnt lgkmcnt(" #n ")" ::: "memory")
#define BAR() do { asm volatile("" ::: "memory"); __builtin_amdgcn_s_barrier(); asm volatile("" ::: "memory"); } while (0)
#define SCHED() __builtin_amdgcn_sched_barrier(0)

__device__ __forceinline__ u32 rfl(u32 v) { return __builtin_amdgcn_readfirstlane(v); }
__device__ __forceinline__ __amdgpu_buffer_rsrc_t mk_rsrc(const void* p, u32 bytes) {
    return __builtin_amdgcn_make_buffer_rsrc((void*)p, 0, (int)bytes, 0x00020000);
}
#define HOLD(x) asm volatile("s_nop 1" :: "v"(x) : "memory")
#define HOLD4(x) asm volatile("s_nop 1" :: "v"(x[0]), "v"(x[1]), "v"(x[2]), "v"(x[3]) : "memory")
__device__ __forceinline__ u32 umax(u32 a, u32 b) { return a > b ? a : b; }
__device__ __forceinline__ u32 wave_max_u32(u32 v) {
    v = umax(v, (u32)__builtin_amdgcn_update_dpp(0, (int)v, 0x111, 0xf, 0xf, true));
    v = umax(v, (u32)__builtin_amdgcn_update_dpp(0, (int)v, 0x112, 0xf, 0xf, true));
    v = umax(v, (u32)__builtin_amdgcn_update_dpp(0, (int)v, 0x114, 0xf, 0xf, true));
    v = umax(v, (u32)__builtin_amdgcn_update_dpp(0, (int)v, 0x118, 0xf, 0xf, true));
    return umax(umax((u32)__builtin_amdgcn_readlane((int)v, 15), (u32)__builtin_amdgcn_readlane((int)v, 31)),
                umax((u32)__builtin_amdgcn_readlane((int)v, 47), (u32)__builtin_amdgcn_readlane((int)v, 63)));
}
__device__ __forceinline__ bool give_up(u32* tmo, u32 call, u64 t0) {
    const u64 dt = __builtin_amdgcn_s_memrealtime() - t0;
    if (dt < 20000ull) return false;
    const u32 t = rfl(__hip_atomic_load(tmo, __ATOMIC_RELAXED, __HIP_MEMORY_SCOPE_AGENT));
    if (t == call) return true;
    if (dt > DEADLINE_TICKS) {
        __hip_atomic_store(tmo, call, __ATOMIC_RELAXED, __HIP_MEMORY_SCOPE_AGENT);
        return true;
    }
    return false;
}

#define DEC_SLEEP 2
#define WPF_SLEEP 8
#define OUT_AUX 16
#define A2A 0
#define A2A_SLEEP 2
#define XPRIO 3
#define WPRIO 2
#define POLL_SLEEP 2
#define STAMP(i) do {} while (0)
#define TB() do {} while (0)
#define TE(i) do {} while (0)
#define TFIX(i) do {} while (0)
#define STAMP_OUT() do {} while (0)

#define XL0 ((u32)((lane >> 3) * (KD * 2) + (((lane & 7) ^ (lane >> 4)) << 4)))
#define XL1 ((u32)((8 + (lane >> 3)) * (KD * 2) + (((lane & 7) ^ (4 + (lane >> 4))) << 4)))
#define DIRECT(v) ((v) < VRAW0 || (v) >= VRAW)
#define U_SRC(v) ((((v) & 15) * 32 * KD + ((v) >> 4) * 64) * 2)
#define U_XD(src, v) (OFF_XD + (u32)((v) >> 4) * 524288u + (u32)((src) * 16 + ((v) & 15)) * 4096u)
#define U_DONE(src, v) (OFF_DONE + (u32)(((v) >> 4) * 128 + (src) * 16 + ((v) & 15)) * 128u)

#define MORD 7
__device__ constexpr int mord(int i) {
    return MORD == 1 || MORD == 5 || MORD == 6 ? (i >> 2) * 4 + (((i >> 2) & 1) ? 3 - (i & 3) : (i & 3))
         : MORD == 2 ? (i % 5) * 4 + i / 5
         : MORD == 3 || MORD == 7 ? (((i / 5) & 1) ? 4 - i % 5 : i % 5) * 4 + i / 5
         : i;
}

__global__ __launch_bounds__(THREADS, 3) void ag_gemm(Params P) {
    extern __shared__ __attribute__((aligned(16))) char smem_[];
    LAS char* smem = (LAS char*)smem_;
    const int lane = threadIdx.x & 63;
    const int wid = (int)rfl(threadIdx.x >> 6);
    const int b = blockIdx.x;
    const int rank = P.rank & 7;
    char* ws = P.self;
    u32* tmo = (u32*)(ws + OFF_TMO);
    STAMP(0);
    const __amdgpu_buffer_rsrc_t rsW = mk_rsrc(ws, WS_USED);
    const int xcd = b & 7, idx = b >> 3;
    u32* seq = (u32*)(ws + OFF_SEQ) + b * 16 + wid;

    auto peer_of = [&](int i) {
        u64 a = (u64)P.peer[7];
#pragma unroll
        for (int p = 0; p < 7; ++p) { u64 t = (u64)P.peer[p]; asm("" : "+s"(t)); a = i == p ? t : a; }
        return (GAS char*)a;
    };

    if (idx >= 26) {
        const int ci = (idx - 26) * 8 + xcd;
        if (ci < NPC) {
            if (wid >= PWAVES) return;
            const int pi = wid * NPC + ci;
            const __amdgpu_buffer_rsrc_t rsX = mk_rsrc(P.x, 512u * KD * 2u);
            const u32 xl0 = XL0, xl1 = XL1;
            u32 sq;
            asm volatile("global_load_dword %0, %1, %2 sc1" : "=&v"(sq) : "v"(0u), "s"(seq) : "memory");
            GAS char* hp = peer_of(lane >> 3) + (lane & 7) * 8;
            __amdgpu_buffer_rsrc_t rsP[7];
#pragma unroll
            for (int p = 0; p < 7; ++p) rsP[p] = mk_rsrc(P.peer[p], WS_USED);
            u32* tkt = (u32*)(ws + OFF_TKT);
            u32 call = 0, tbase = 0;
            int done = -1, cur = -1, nxt = pi;
            u32x4 pk[4] = {};
            u32 hw = 0, hw_done = 0;
            bool raw = false;
            for (;;) {
                asm volatile("" ::: "memory");
                u32 tk;
                asm volatile("s_cmp_lt_u32 %4, 1024\n\ts_cbranch_scc0 1f\n\ts_mov_b64 exec, 1\n\tglobal_atomic_add %0, %1, %2, %3 sc0\n\ts_mov_b64 exec, -1\n1:"
                             : "=&v"(tk) : "v"(0u), "v"(1u), "s"(tkt), "s"(cur) : "memory", "scc");
                u32x4 d[4];
                asm volatile("s_cmp_lt_u32 %8, 1024\n\ts_cbranch_scc0 1f\n\t"
                             "buffer_load_dwordx4 %0, %4, %6, %7 offen\n\tbuffer_load_dwordx4 %1, %5, %6, %7 offen\n\t"
                             "buffer_load_dwordx4 %2, %4, %6, %9 offen\n\tbuffer_load_dwordx4 %3, %5, %6, %9 offen\n1:"
                             : "=&v"(d[0]), "=&v"(d[1]), "=&v"(d[2]), "=&v"(d[3])
                             : "v"(xl0), "v"(xl1), "s"(rsX), "s"(U_SRC(nxt)), "s"(nxt), "s"(U_SRC(nxt) + 16 * KD * 2) : "memory", "scc");
                if (done >= 0 && lane < 56) {
                    const u32 o = !DIRECT(done) ? OFF_HDR + (u32)(rank * 1024 + done) * 64u : U_DONE(rank, done);
                    __hip_atomic_store((GAS u64*)(hp + o), ((u64)call << 32) | (!DIRECT(done) ? hw_done : call), __ATOMIC_RELAXED, __HIP_MEMORY_SCOPE_SYSTEM);
                }
                if (cur >= 1024) break;
                TB();
                if (cur >= 0) {
                    const u32 o01 = raw ? U_XD(rank, cur) : OFF_PK + (u32)(rank * 1024 + cur) * 3072u;
                    const u32 o23 = o01 + 2048u;
                    const u32 v2 = raw || lane < 52 ? lane * 16u : 0x80000000u;
#pragma unroll
                    for (int p = 0; p < 7; ++p) {
                        __builtin_amdgcn_raw_buffer_store_b128(pk[0], rsP[p], lane * 16u, o01, AUX_SYS);
                        __builtin_amdgcn_raw_buffer_store_b128(pk[1], rsP[p], lane * 16u + 1024u, o01, AUX_SYS);
                        __builtin_amdgcn_raw_buffer_store_b128(pk[2], rsP[p], v2, o23, AUX_SYS);
                    }
                    if (raw) {
#pragma unroll
                        for (int p = 0; p < 7; ++p) __builtin_amdgcn_raw_buffer_store_b128(pk[3], rsP[p], lane * 16u + 1024u, o23, AUX_SYS);
                    }
                }
                HOLD4(pk);
                TE(13);
                const int had = done;
                done = cur; hw_done = hw;
                asm volatile("s_waitcnt vmcnt(22)\n\ts_cmp_lt_i32 %5, 0\n\ts_cbranch_scc0 1f\n\ts_waitcnt vmcnt(21)\n\ts_cmp_lt_i32 %4, 0\n\ts_cbranch_scc0 1f\n\ts_waitcnt vmcnt(0)\n1:"
                             : "+v"(d[0]), "+v"(d[1]), "+v"(d[2]), "+v"(d[3]) : "s"(cur), "s"(HDR_FIRST ? -1 : had) : "memory", "scc");
                TE(14);
                raw = true;
                if (!DIRECT(nxt)) {
                    u16x2 mx = {0, 0};
#pragma unroll
                    for (int i = 0; i < 4; ++i)
#pragma unroll
                        for (int e = 0; e < 4; ++e) mx = __builtin_elementwise_max(mx, __builtin_bit_cast(u16x2, d[i][e] & 0x7FFF7FFFu));
                    const u32 el = (u32)(mx.x > mx.y ? mx.x : mx.y) >> 7;
                    u32 top = wave_max_u32(el);
                    while (top > 7u && __builtin_popcountll(__builtin_amdgcn_ballot_w64(el >= top)) < 16) --top;
                    const u32 low = top < 7u ? 0u : top - 7u;
                    const u16x2 bias = __builtin_bit_cast(u16x2, low * 0x00800080u);
                    u32 sg = 0, c2[2] = {0, 0}, em = 0;
                    u32x4 hi[2];
#pragma unroll
                    for (int k = 0; k < 8; ++k) {
                        const u32 d0 = d[k >> 1][(k & 1) * 2], d1 = d[k >> 1][(k & 1) * 2 + 1];
                        const u32 x0 = __builtin_bit_cast(u32, (u16x2)(__builtin_bit_cast(u16x2, d0) - bias));
                        const u32 x1 = __builtin_bit_cast(u32, (u16x2)(__builtin_bit_cast(u16x2, d1) - bias));
                        const u32 h = __builtin_amdgcn_perm(x1, x0, 0x07050301u);
                        pk[k >> 2][k & 3] = __builtin_amdgcn_perm(x1, x0, 0x06040200u);
                        hi[k >> 2][k & 3] = h;
                        sg |= (h >> k) & (0x80808080u >> k);
                        c2[k >> 2] |= (h & 0x03030303u) << (2 * (k & 3));
                        em |= (((h & 0x7C7C7C7Cu) + 0x7C7C7C7Cu) >> k) & (0x80808080u >> k);
                    }
                    LAS char* L = smem + wid * 4096;
                    *(LAS u32*)(L + lane * 4) = sg;
                    *(LAS u32*)(L + 256 + lane * 4) = c2[0];
                    *(LAS u32*)(L + 512 + lane * 4) = c2[1];
                    *(LAS u32x4*)(L + 1024 + lane * 32) = hi[0];
                    *(LAS u32x4*)(L + 1040 + lane * 32) = hi[1];
                    asm volatile("" ::: "memory");
                    u32 n = 0;
                    u64 m = __builtin_amdgcn_ballot_w64(em != 0u);
                    while (m != 0ull && n <= 46u) {
                        const u32 idx = n + __builtin_amdgcn_mbcnt_hi((u32)(m >> 32), __builtin_amdgcn_mbcnt_lo((u32)m, 0u));
                        const u32 bit = (u32)__builtin_ctz(em | 0x80000000u);
                        if (em != 0u) *(LAS unsigned short*)(L + 768 + idx * 2u) = (unsigned short)((u32)lane * 32u + (7u - (bit & 7u)) * 4u + (bit >> 3));
                        em &= em - 1u;
                        n += (u32)__builtin_popcountll(m);
                        m = __builtin_amdgcn_ballot_w64(em != 0u);
                    }
                    asm volatile("" ::: "memory");
                    {
                        const u32 pos = *(LAS unsigned short*)(L + 768 + lane * 2) & 2047u;
                        const u32 hb = *(LAS unsigned char*)(L + 1024 + pos);
                        *(LAS unsigned short*)(L + 768 + lane * 2) = (unsigned short)((u32)lane < n ? pos | (hb & 0x7Cu) << 9 : 0u);
                    }
                    asm volatile("" ::: "memory");
                    pk[2] = *(LAS u32x4*)(L + lane * 16);
                    hw = *(LAS u32*)(L + 828 + (lane & 7) * 4);
                    raw = n > 46u;
                    if ((lane & 7) == 0) hw = raw ? 256u : low;
                    asm volatile("" ::: "memory");
                }
                if (raw) {
#pragma unroll
                    for (int i = 0; i < 4; ++i) pk[i] = d[i];
                }
                asm volatile("s_waitcnt vmcnt(0)" : "+v"(tk), "+v"(sq) :: "memory");
                TE(15);
                const bool first = cur < 0;
                cur = nxt;
                if (first) {
                    call = rfl(sq) + 1u;
                    tbase = (call - 1u) * 1024u - 2u * NPUSH;
                    if (lane == 0) __hip_atomic_store(seq, call, __ATOMIC_RELAXED, __HIP_MEMORY_SCOPE_AGENT);
                    nxt = pi + NPUSH;
                    STAMP(2);
                    for (int i = NRAWF; i < wid; ++i) __builtin_amdgcn_s_sleep(STAGGER);
                } else {
                    nxt = (int)(rfl(tk) - tbase);
                }
            }
            STAMP(3);
            TFIX(13); TFIX(14); TFIX(15);
            STAMP_OUT();
            return;
        }
        if (wid >= DWAVES) return;
        const u32 call = rfl(__hip_atomic_load(seq, __ATOMIC_RELAXED, __HIP_MEMORY_SCOPE_AGENT)) + 1u;
        if (lane == 0) __hip_atomic_store(seq, call, __ATOMIC_RELAXED, __HIP_MEMORY_SCOPE_AGENT);
        {
            u32* pend = nullptr;
            LAS char* L = smem + wid * 2048;
            *(LAS u32x4*)(L + lane * 32) = (u32x4){0u, 0u, 0u, 0u};
            *(LAS u32x4*)(L + lane * 32 + 16) = (u32x4){0u, 0u, 0u, 0u};
            const int hg = lane < 8 ? lane : lane >= 32 && lane < 46 ? 1 + ((lane - 32) >> 1) : 0;
            for (int q = VRAW0 * 7 + wid * (NCOMM - NPC) + ci - NPC; q < VRAW * 7; q += NDEC) {
                const int v = q / 7, src = (rank + 1 + q - v * 7) & 7;
                const u32 xdb = U_XD(src, v);
                const u64* hp = (const u64*)(ws + OFF_HDR + src * 65536) + v * 8 + hg;
                u32 hw, h0;
                bool dead = false;
                {
                    const u64 t0 = __builtin_amdgcn_s_memrealtime();
                    for (;;) {
                        const u64 g = __hip_atomic_load(hp, __ATOMIC_RELAXED, __HIP_MEMORY_SCOPE_SYSTEM);
                        hw = (u32)g; h0 = rfl(hw);
                        u64 ok = __builtin_amdgcn_ballot_w64((u32)(g >> 32) == call);
                        asm volatile("s_waitcnt vmcnt(0)" : "+s"(ok), "+s"(h0), "+v"(hw) :: "memory");
                        if (pend) {
                            if (lane == 0) __hip_atomic_store(pend, call, __ATOMIC_RELAXED, __HIP_MEMORY_SCOPE_AGENT);
                            pend = nullptr;
                        }
                        if (ok == ~0ull) break;
                        if (give_up(tmo, call, t0)) { dead = true; break; }
                        __builtin_amdgcn_s_sleep(DEC_SLEEP);
                    }
                }
                if (dead) break;
                asm volatile("" ::: "memory");
                if (h0 < 256u) {
                    const u32 pk = OFF_PK + (u32)(src * 1024 + v) * 3072u;
                    const u32 e16 = __builtin_amdgcn_raw_buffer_load_b16(rsW, 2816u + (lane & 31) * 2u, pk, AUX_SYS);
                    const u32 sg = __builtin_amdgcn_raw_buffer_load_b32(rsW, 2048u + lane * 4u, pk, AUX_SYS);
                    u32 c2[2];
                    c2[0] = __builtin_amdgcn_raw_buffer_load_b32(rsW, 2304u + lane * 4u, pk, AUX_SYS);
                    c2[1] = __builtin_amdgcn_raw_buffer_load_b32(rsW, 2560u + lane * 4u, pk, AUX_SYS);
                    u32x4 lo[2];
                    lo[0] = __builtin_amdgcn_raw_buffer_load_b128(rsW, lane * 16u, pk, AUX_SYS);
                    lo[1] = __builtin_amdgcn_raw_buffer_load_b128(rsW, lane * 16u + 1024u, pk, AUX_SYS);
                    const u32 ent = lane < 32 ? e16 : lane < 46 ? (hw >> ((lane & 1) * 16)) & 0xFFFFu : 0u;
                    if (ent >= 2048u) *(LAS unsigned char*)(L + (ent & 2047u)) = (unsigned char)((ent >> 11) << 2);
                    asm volatile("" ::: "memory");
                    u32x4 ex[2];
                    ex[0] = *(LAS u32x4*)(L + lane * 32);
                    ex[1] = *(LAS u32x4*)(L + lane * 32 + 16);
                    asm volatile("" ::: "memory");
                    if (ent >= 2048u) *(LAS unsigned char*)(L + (ent & 2047u)) = 0;
                    const u16x2 bias = __builtin_bit_cast(u16x2, h0 * 0x00800080u);
#pragma unroll
                    for (int i = 0; i < 4; ++i) {
                        u32x4 o;
#pragma unroll
                        for (int j = 0; j < 2; ++j) {
                            const int k = 2 * i + j;
                            const u32 h = ((sg << k) & 0x80808080u) | ((c2[k >> 2] >> (2 * (k & 3))) & 0x03030303u) | ex[k >> 2][k & 3];
                            const u32 l = lo[k >> 2][k & 3];
                            o[2 * j] = __builtin_bit_cast(u32, (u16x2)(__builtin_bit_cast(u16x2, __builtin_amdgcn_perm(h, l, 0x05010400u)) + bias));
                            o[2 * j + 1] = __builtin_bit_cast(u32, (u16x2)(__builtin_bit_cast(u16x2, __builtin_amdgcn_perm(h, l, 0x07030602u)) + bias));
                        }
                        __builtin_amdgcn_raw_buffer_store_b128(o, rsW, lane * 16u + i * 1024u, xdb, AUX_AGT);
                        HOLD(o);
                    }
                }
                asm volatile("" ::: "memory");
                pend = (u32*)(ws + U_DONE(src, v));
            }
            WAIT_V(0);
            if (pend && lane == 0) __hip_atomic_store(pend, call, __ATOMIC_RELAXED, __HIP_MEMORY_SCOPE_AGENT);
            STAMP(3);
            TFIX(4); TFIX(5); TFIX(6);
        }
        if (!A2A && ci == 47 && wid == 0) {
            GAS u64* dst = (GAS u64*)(peer_of(lane & 7) + OFF_GMAX + rank * 128);
            const u64 t0 = __builtin_amdgcn_s_memrealtime();
            u32 m = 0;
            for (;;) {
                const u32x4 a = __builtin_amdgcn_raw_buffer_load_b128(rsW, OFF_LMAX + lane * 16, 0, AUX_AGT);
                const u32x4 c = __builtin_amdgcn_raw_buffer_load_b128(rsW, OFF_LMAX + 1024 + lane * 16, 0, AUX_AGT);
                const bool ok = a[1] == call && a[3] == call && (lane >= 40 || (c[1] == call && c[3] == call));
                m = a[0] > a[2] ? a[0] : a[2];
                if (lane < 40) { m = m > c[0] ? m : c[0]; m = m > c[2] ? m : c[2]; }
                if (__builtin_amdgcn_ballot_w64(ok) == ~0ull) break;
                if (give_up(tmo, call, t0)) break;
                __builtin_amdgcn_s_sleep(1);
            }
            STAMP(7);
            m = wave_max_u32(m);
            if (lane < NR) __hip_atomic_store(dst, ((u64)call << 32) | m, __ATOMIC_RELAXED, __HIP_MEMORY_SCOPE_SYSTEM);
            STAMP(8);
        }
        STAMP_OUT();
        return;
    }

    const int T = xcd * 26 + idx;
    const int ct = (T / 104) * 4 + (T % 104 & 3), rt = (T % 104) >> 2;
    const int rg0 = rt * 10;
    const int nblk = rt == 25 ? 3 : 5;
    const u32 lds0 = (u32)(size_t)smem;
#define DMA(ldsaddr, voff, rs, soff, mods) asm volatile("s_mov_b32 m0, %0\n\ts_nop 0\n\tbuffer_load_dwordx4 %1, %2, %3 offen" mods " lds" \
        :: "s"(ldsaddr), "v"(voff), "s"(rs), "s"(soff) : "memory")
#define DMA_SAME_M0(voff, rs, soff, mods) asm volatile("buffer_load_dwordx4 %0, %1, %2 offen" mods " lds" :: "v"(voff), "s"(rs), "s"(soff) : "memory")

    if (wid == 11) return;
    if (wid >= 9) {
        __builtin_amdgcn_s_setprio(WPRIO);
        const int h = wid - 9;
        const __amdgpu_buffer_rsrc_t rsB = mk_rsrc(P.w, (u32)KD * NL * 2u);
        u32 vB[16];
#pragma unroll
        for (int i = 0; i < 16; ++i) {
            const int row = i * 2 + (lane >> 5);
            vB[i] = (u32)((h * 32 + row) * (NL * 2) + ct * 512 + (((lane & 31) ^ ((row & 3) | (row & 8))) << 4));
        }
        const u32 lB = lds0 + 20480 + h * 16384;
#define LOAD_W(kt, so) do { _Pragma("unroll") for (int i = 0; i < 16; ++i) DMA(lB + (so) + i * 1024, vB[i], rsB, (kt) * 262144, ""); } while (0)
        LOAD_W(0, 0);
        LOAD_W(1, STAGE);
        WAIT_V(16);
        BAR();
        int so = 2 * STAGE;
        for (int t = 0; t < 62; ++t) {
            { TB(); LOAD_W(t + 2, so); TE(5); }
            so = so == 2 * STAGE ? 0 : so + STAGE;
            { TB(); WAIT_V(16); TE(6); }
            { TB(); BAR(); TE(7); }
        }
        WAIT_V(0);
        BAR();
        TFIX(5); TFIX(6); TFIX(7);
        STAMP_OUT();
        return;
    }

    if (wid == 8) {
        __builtin_amdgcn_s_setprio(XPRIO);
        const u32 call = rfl(__hip_atomic_load(seq, __ATOMIC_RELAXED, __HIP_MEMORY_SCOPE_AGENT)) + 1u;
        if (lane == 0) __hip_atomic_store(seq, call, __ATOMIC_RELAXED, __HIP_MEMORY_SCOPE_AGENT);
        const u32* cp = (const u32*)(ws + OFF_DONE + ((rg0 >> 1) + ((lane & 7) < nblk ? (lane & 7) : 0)) * 128);
        const u32 vA = (u32)lane * 16u;
        const u32 gA = OFF_XD + (u32)rg0 * 2048u;
        int ready = 0;
        auto look = [&]() {
            const int c = ready + (lane >> 3);
            const u32 f = __hip_atomic_load(cp + (c < 63 ? c : 63) * 4096, __ATOMIC_RELAXED, __HIP_MEMORY_SCOPE_SYSTEM);
            const u64 miss = __builtin_amdgcn_ballot_w64(f < call);
            ready += miss == 0ull ? 8 : __builtin_ctzll(miss) >> 3;
        };
        auto need = [&](int kt) {
            if (kt < ready) return;
            const u64 t0 = __builtin_amdgcn_s_memrealtime();
            for (;;) {
                look();
                if (kt < ready) break;
                if (give_up(tmo, call, t0)) { ready = 64; break; }
                __builtin_amdgcn_s_sleep(POLL_SLEEP);
            }
        };
#define LOAD_X4(g, l) do { \
            DMA((l), vA, rsW, (g), " sc0 sc1"); DMA_SAME_M0(vA, rsW, (g), " offset:1024 sc0 sc1"); DMA_SAME_M0(vA, rsW, (g), " offset:2048 sc0 sc1"); DMA_SAME_M0(vA, rsW, (g), " offset:3072 sc0 sc1"); } while (0)
#define XST(kt) do {} while (0)
#define LOAD_X(kt, so) do { XST(kt); _Pragma("unroll") for (int i = 0; i < 5; ++i) \
            LOAD_X4(gA + (kt) * 524288 + (i < nblk ? i : 0) * 4096, lds0 + (so) + (i < nblk ? i : 0) * 4096); } while (0)
        int xi = 0, so = 0;
        for (int t = -1; t < 63; ++t) {
            if (xi == t + 1) {
                { TB(); need(xi); TE(10); }
                LOAD_X(xi, so);
                ++xi; so = so == 2 * STAGE ? 0 : so + STAGE;
            }
            if (xi < 64 && xi >= ready) { TB(); look(); TE(11); }
            if (xi < 64 && xi < ready) {
                LOAD_X(xi, so);
                ++xi; so = so == 2 * STAGE ? 0 : so + STAGE;
                TB(); WAIT_V(20); TE(12);
            } else {
                TB(); WAIT_V(0); TE(13);
            }
            { TB(); BAR(); TE(14); }
        }
        STAMP(4);
        TFIX(10); TFIX(11); TFIX(12); TFIX(13); TFIX(14);
        STAMP_OUT();
        return;
    }

    const int wm = wid >> 2, wn = wid & 3;
    const u32 call = rfl(__hip_atomic_load(seq, __ATOMIC_RELAXED, __HIP_MEMORY_SCOPE_AGENT)) + 1u;
    if (lane == 0) __hip_atomic_store(seq, call, __ATOMIC_RELAXED, __HIP_MEMORY_SCOPE_AGENT);
    if (wid * 208 + T < 1024) {
        const int v = wid * 208 + T;
        const __amdgpu_buffer_rsrc_t rsX = mk_rsrc(P.x, 512u * KD * 2u);
        u32x4 d[4];
#pragma unroll
        for (int i = 0; i < 4; ++i) d[i] = __builtin_amdgcn_raw_buffer_load_b128(rsX, (i & 1) ? XL1 : XL0, U_SRC(v) + (i >> 1) * 16 * KD * 2, 0);
#pragma unroll
        for (int i = 0; i < 4; ++i) __builtin_amdgcn_raw_buffer_store_b128(d[i], rsW, lane * 16u + i * 1024u, U_XD(rank, v), AUX_AGT);
        HOLD4(d);
        WAIT_V(0);
        if (lane == 0) __hip_atomic_store((u32*)(ws + U_DONE(rank, v)), call, __ATOMIC_RELAXED, __HIP_MEMORY_SCOPE_AGENT);
    }

    {
        if (WPF_SLEEP) __builtin_amdgcn_s_sleep(WPF_SLEEP);
        const __amdgpu_buffer_rsrc_t rsB = mk_rsrc(P.w, (u32)KD * NL * 2u);
        u32 t[3];
#pragma unroll
        for (int i = 0; i < 3; ++i) {
            const u32 r = (u32)(i * 1664 + T * 8 + wid);
            t[i] = __builtin_amdgcn_raw_buffer_load_b32(rsB, lane * 64u, (r < 4096u ? r : 0u) * 4096u, 0);
        }
        asm volatile("" :: "v"(t[0]), "v"(t[1]), "v"(t[2]));
    }
    if (nblk < 5) {
#pragma unroll
        for (int i = 0; i < 3; ++i) *(LAS u32x4*)(smem + i * STAGE + 12288 + wid * 1024 + lane * 16) = (u32x4){0u, 0u, 0u, 0u};
        WAIT_L(0);
    }
    u32 pa[2], pb[4];
#pragma unroll
    for (int h = 0; h < 2; ++h) pa[h] = lds0 + wm * 10240 + (lane & 15) * 128 + (((4 * h + (lane >> 4)) ^ ((lane >> 1) & 7)) << 4);
    {
        const int g = lane >> 4, q = (lane >> 2) & 3, p = lane & 3;
#pragma unroll
        for (int j = 0; j < 4; ++j) {
            const int jj = j ^ (p & 1);
            pb[j] = lds0 + 20480 + (8 * g + q) * 512 + (((wn * 8 + 2 * p + (jj >> 1)) ^ (q | ((g & 1) << 3))) << 4) + (jj & 1) * 8;
        }
    }
    f32x4 acc[5][4];
#pragma unroll
    for (int m = 0; m < 5; ++m)
#pragma unroll
        for (int j = 0; j < 4; ++j) acc[m][j] = (f32x4){0.f, 0.f, 0.f, 0.f};

    s16x4 f0b[8], f1b[8];
    bf16x8 f0a[5], f1a[5];
#define RD(i, fa, fb, h) do { \
        if ((i) < 8) fb[(i) & 7] = __builtin_amdgcn_ds_read_tr16_b64_v4i16((LAS s16x4*)(size_t)(pb[((i) & 7) >> 1] + (h) * 16384 + ((i) & 1) * 2048)); \
        else fa[(i) < 8 ? 0 : (i) - 8] = *(LAS bf16x8*)(size_t)(pa[h] + ((i) - 8) * 2048); } while (0)
#define MM(i, fa, fb) acc[(i) >> 2][(i) & 3] = __builtin_amdgcn_mfma_f32_16x16x32_bf16( \
        __builtin_bit_cast(bf16x8, __builtin_shufflevector(fb[((i) & 3) * 2], fb[((i) & 3) * 2 + 1], 0, 1, 2, 3, 4, 5, 6, 7)), fa[(i) >> 2], acc[(i) >> 2][(i) & 3], 0, 0, 0)
#define VADD(v, s) asm volatile("v_add_u32 %0, %1, %0" : "+v"(v) : "s"(s))
    BAR();
    STAMP(3);
    if (wm == 0) __builtin_amdgcn_s_setprio(1);
    int ph = 0;
    for (int t = 0; t < 64; ++t) {
        SCHED();
#pragma unroll
        for (int i = 0; i < 13; ++i) RD(i, f0a, f0b, 0);
#pragma unroll
        for (int i = 0; i < 13; ++i) RD(i, f1a, f1b, 1);
        SCHED();
        {
            const int dlt = ph == 2 ? -2 * STAGE : STAGE;
            ph = ph == 2 ? 0 : ph + 1;
            VADD(pa[0], dlt); VADD(pa[1], dlt);
#pragma unroll
            for (int j = 0; j < 4; ++j) VADD(pb[j], dlt);
        }
        const int who = t == 63 ? 2 : wm;
        asm volatile("s_cmp_lg_u32 %0, 0\n\ts_cbranch_scc1 1f\n\ts_waitcnt lgkmcnt(0)\n\ts_barrier\n1:" :: "s"(who) : "memory", "scc");
        SCHED();
#pragma unroll
        for (int i = 0; i < 20; ++i) {
            if (MORD >= 6 && (i & 1)) { MM(mord(i), f1a, f1b); SCHED(); MM(mord(i), f0a, f0b); SCHED(); }
            else { MM(mord(i), f0a, f0b); SCHED(); MM(mord(i), f1a, f1b); SCHED(); }
        }
        SCHED();
        asm volatile("s_cmp_lg_u32 %0, 1\n\ts_cbranch_scc1 1f\n\ts_barrier\n1:" :: "s"(who) : "memory", "scc");
    }
    __builtin_amdgcn_s_setprio(0);
    STAMP(4);
    TFIX(14);

    GAS u64* tdst = (GAS u64*)(peer_of(lane & 7) + OFF_TMAX) + rank * 256 + T;
    float mx = 0.f;
#pragma unroll
    for (int m = 0; m < 5; ++m)
#pragma unroll
        for (int j = 0; j < 4; ++j) {
            mx = __builtin_fmaxf(__builtin_fmaxf(mx, acc[m][j][0]), acc[m][j][1]);
            mx = __builtin_fmaxf(__builtin_fmaxf(mx, acc[m][j][2]), acc[m][j][3]);
        }
    LAS u32* misc = (LAS u32*)(smem + LDS_MISC + 512);
    {
        const u32 wmx = wave_max_u32(__float_as_uint(mx));
        if (lane == 0) misc[wid] = wmx;
    }
    WAIT_L(0);
    BAR();
    if (wid == 0) {
        const u32 m = wave_max_u32(misc[lane & 7]);
        if (!A2A) { if (lane == 0) __hip_atomic_store((u64*)(ws + OFF_LMAX) + T, ((u64)call << 32) | m, __ATOMIC_RELAXED, __HIP_MEMORY_SCOPE_AGENT); }
        else if (lane < NR) __hip_atomic_store(tdst, ((u64)call << 32) | m, __ATOMIC_RELAXED, __HIP_MEMORY_SCOPE_SYSTEM);
    }
    {
        const bool sw = (lane >> 4) & 1;
#pragma unroll
        for (int m = 0; m < 5; ++m)
#pragma unroll
            for (int j = 0; j < 4; j += 2)
#pragma unroll
                for (int r = 0; r < 4; ++r) {
                    const float u = __builtin_fmaxf(acc[m][j][r], 0.f), v = __builtin_fmaxf(acc[m][j + 1][r], 0.f);
                    acc[m][j][r] = sw ? v : u; acc[m][j + 1][r] = sw ? u : v;
                }
#pragma unroll
        for (int m = 0; m < 5; ++m) asm volatile("" : "+v"(acc[m][0]), "+v"(acc[m][1]), "+v"(acc[m][2]), "+v"(acc[m][3]));
    }
    const __amdgpu_buffer_rsrc_t rsO = mk_rsrc(P.out, 4096u * NL);
    u32 oo = (u32)((rg0 * 16 + wm * 80 + (lane & 15)) * NL + ct * 256 + wn * 64 + (lane >> 4) * 16);
    asm volatile("" : "+v"(oo));
    if (A2A) {
        const u64 t0 = __builtin_amdgcn_s_memrealtime();
        u32 m;
        bool bad = false;
        for (;;) {
            const u32x4 a = __builtin_amdgcn_raw_buffer_load_b128(rsW, lane * 16u, OFF_TMAX + wid * 2048, AUX_SYS);
            const u32x4 c = __builtin_amdgcn_raw_buffer_load_b128(rsW, lane < 40 ? OFF_TMAX + wid * 2048 + 1024 + lane * 16u : OFF_TMO, 0, AUX_SYS);
            const bool ok = a[1] == call && a[3] == call && (lane >= 40 || (c[1] == call && c[3] == call));
            m = umax(a[0], a[2]);
            if (lane < 40) m = umax(m, umax(c[0], c[2]));
            bad = (u32)__builtin_amdgcn_readlane((int)c[0], 63) == call;
            if (__builtin_amdgcn_ballot_w64(ok) == ~0ull) break;
            if (give_up(tmo, call, t0)) { bad = true; break; }
            __builtin_amdgcn_s_sleep(A2A_SLEEP);
        }
        m = bad ? 0x7FC00000u : wave_max_u32(m);
        if (lane == 0) misc[16 + wid] = m;
        WAIT_L(0);
    } else
    if (wid == 0) {
        const u64* gp = lane == 8 ? (const u64*)tmo : (const u64*)(ws + OFF_GMAX + (lane & 7) * 128);
        const u64 t0 = __builtin_amdgcn_s_memrealtime();
        u64 v;
        bool bad = false;
        for (;;) {
            v = __hip_atomic_load(gp, __ATOMIC_RELAXED, __HIP_MEMORY_SCOPE_SYSTEM);
            if (((u32)__builtin_amdgcn_ballot_w64((u32)(v >> 32) == call) & 0xFFu) == 0xFFu) break;
            if (give_up(tmo, call, t0)) { bad = true; break; }
            __builtin_amdgcn_s_sleep(1);
        }
        bad = bad || (u32)__builtin_amdgcn_readlane((int)(u32)v, 8) == call;
        const float amax = __uint_as_float(wave_max_u32(lane == 8 ? 0u : (u32)v));
        const float inv = bad ? __builtin_nanf("") : 1.0f / (amax / 448.0f);
        if (lane == 0) misc[8] = __float_as_uint(inv);
        WAIT_L(0);
    }
    BAR();
    STAMP(5);
    {
        float inv;
        if (A2A) {
            const u32x4 a = *(LAS u32x4*)(misc + 16), c = *(LAS u32x4*)(misc + 20);
            const u32 m = umax(umax(umax(a[0], a[1]), umax(a[2], a[3])), umax(umax(c[0], c[1]), umax(c[2], c[3])));
            inv = 1.0f / (__uint_as_float(m) / 448.0f);
        } else inv = __uint_as_float(misc[8]);
        const f32x4 inv4 = {inv, inv, inv, inv};
#pragma unroll
        for (int m = 0; m < 5; ++m) {
            u32x4 o;
#pragma unroll
            for (int j = 0; j < 4; ++j) {
                const f32x4 y = acc[m][j] * inv4;
                int t = __builtin_amdgcn_cvt_pk_fp8_f32(y[0], y[1], 0, false);
                t = __builtin_amdgcn_cvt_pk_fp8_f32(y[2], y[3], t, true);
                o[j] = (u32)t;
            }
            __builtin_amdgcn_raw_buffer_store_b128(o, rsO, oo + m * 16 * NL, 0, OUT_AUX);
            HOLD(o);
        }
    }
    STAMP(6);
    STAMP_OUT();
}

}

extern "C" void kernel_launch(int rank, int nranks, ncclComm_t comm,
                              hipStream_t stream,
                              void* const* d_in, const int* in_sizes, int n_in,
                              void* d_out, int out_size,
                              const int* in_dtypes, int out_dtype,
                              void* const* d_out_all,
                              void* const* d_ws_all, size_t ws_bytes) {
    using namespace agk;
    if (nranks != NR || (unsigned)rank >= (unsigned)NR || ws_bytes < WS_USED || n_in < 2 ||
        in_sizes[0] != 512 * KD || in_sizes[1] != KD * NL || out_size != 4096 * NL) {
        fprintf(stderr, "unexpected problem shape\n");
        return;
    }
    static bool attr_done = false;
    if (!attr_done) {
        HIP_CHECK(hipFuncSetAttribute((const void*)ag_gemm, hipFuncAttributeMaxDynamicSharedMemorySize, LDS_BYTES));
        attr_done = true;
    }
    Params P;
    P.x = d_in[0]; P.w = d_in[1]; P.out = d_out;
    P.self = (char*)d_ws_all[rank];
    for (int p = 0; p < NR; ++p) P.peer[p] = (char*)d_ws_all[(rank + 1 + p) % NR];
    P.rank = rank; P.pad = 0;
    hipLaunchKernelGGL(ag_gemm, dim3(GRID), dim3(THREADS), LDS_BYTES, stream, P);
    HIP_CHECK(hipGetLastError());
}
```

```cpp
#include <hip/hip_runtime.h>
#include <rccl/rccl.h>
#include <cstdio>

#define HIP_CHECK(expr) do { hipError_t _e = (expr); if (_e != hipSuccess) { \
    fprintf(stderr, "HIP error %s at %s:%d\n", hipGetErrorString(_e), __FILE__, __LINE__); \
    return; } } while (0)

namespace agk {

typedef unsigned u32;
typedef unsigned long long u64;
typedef u32 u32x4 __attribute__((ext_vector_type(4)));
typedef float f32x4 __attribute__((ext_vector_type(4)));
typedef __bf16 bf16x8 __attribute__((ext_vector_type(8)));
typedef short s16x4 __attribute__((ext_vector_type(4)));
typedef unsigned short u16x2 __attribute__((ext_vector_type(2)));
#define LAS __attribute__((address_space(3)))
#define GAS __attribute__((address_space(1)))

constexpr int NR = 8;
constexpr int KD = 4096;
constexpr int NL = 2048;
constexpr int GRID = 256, THREADS = 768;
#define NPC 22
#define PWAVES 2
#define DWAVES 12
#define NRAWL 2
#define NRAWF 1
#define HDR_FIRST 0
#define STAGGER 20
constexpr int NCOMM = 48;
constexpr int NPUSH = NPC * PWAVES;
#define SHARE 0
#define NDC (SHARE ? NPC : NCOMM - NPC)
constexpr int DC0 = SHARE ? 0 : NPC, DW0 = SHARE ? PWAVES : 0;
constexpr int NDEC = NDC * (DWAVES - DW0);
constexpr int VRAW0 = 16 * NRAWF;
constexpr int VRAW = 1024 - 16 * NRAWL;
constexpr int STAGE = 52 * 1024;
constexpr int LDS_MISC = 3 * STAGE;
constexpr int LDS_BYTES = LDS_MISC + 1024;

constexpr u32 OFF_SEQ = 0;
constexpr u32 OFF_TMO = 16384;
constexpr u32 OFF_TMAX = 65536;
constexpr u32 OFF_TKT = 32768;
constexpr u32 OFF_DBG = 128u << 10;
constexpr u32 OFF_HDR = 512u << 10;
constexpr u32 OFF_DONE = 1u << 20;
constexpr u32 OFF_XD = 4u << 20;
constexpr u32 OFF_PK = 36u << 20;
constexpr u32 WS_USED = 60u << 20;

constexpr u64 DEADLINE_TICKS = 40000000ull;
constexpr int AUX_SYS = 17, AUX_AGT = 16;

struct Params {
    const void* x;
    const void* w;
    void* out;
    char* self;
    char* peer[NR];
    int rank;
    int pad;
};

#define WAIT_V(n) asm volatile("s_waitcnt vmcnt(" #n ")" ::: "memory")
#define WAIT_L(n) asm volatile("s_waitcnt lgkmcnt(" #n ")" ::: "memory")
#define BAR() do { asm volatile("" ::: "memory"); __builtin_amdgcn_s_barrier(); asm volatile("" ::: "memory"); } while (0)
#define SCHED() __builtin_amdgcn_sched_barrier(0)

__device__ __forceinline__ u32 rfl(u32 v) { return __builtin_amdgcn_readfirstlane(v); }
__device__ __forceinline__ __amdgpu_buffer_rsrc_t mk_rsrc(const void* p, u32 bytes) {
    return __builtin_amdgcn_make_buffer_rsrc((void*)p, 0, (int)bytes, 0x00020000);
}
#define HOLD(x) asm volatile("s_nop 1" :: "v"(x) : "memory")
#define HOLD4(x) asm volatile("s_nop 1" :: "v"(x[0]), "v"(x[1]), "v"(x[2]), "v"(x[3]) : "memory")
__device__ __forceinline__ u32 umax(u32 a, u32 b) { return a > b ? a : b; }
__device__ __forceinline__ u32 wave_max_u32(u32 v) {
    v = umax(v, (u32)__builtin_amdgcn_update_dpp(0, (int)v, 0x111, 0xf, 0xf, true));
    v = umax(v, (u32)__builtin_amdgcn_update_dpp(0, (int)v, 0x112, 0xf, 0xf, true));
    v = umax(v, (u32)__builtin_amdgcn_update_dpp(0, (int)v, 0x114, 0xf, 0xf, true));
    v = umax(v, (u32)__builtin_amdgcn_update_dpp(0, (int)v, 0x118, 0xf, 0xf, true));
    return umax(umax((u32)__builtin_amdgcn_readlane((int)v, 15), (u32)__builtin_amdgcn_readlane((int)v, 31)),
                umax((u32)__builtin_amdgcn_readlane((int)v, 47), (u32)__builtin_amdgcn_readlane((int)v, 63)));
}
__device__ __forceinline__ bool give_up(u32* tmo, u32 call, u64 t0) {
    const u64 dt = __builtin_amdgcn_s_memrealtime() - t0;
    if (dt < 20000ull) return false;
    const u32 t = rfl(__hip_atomic_load(tmo, __ATOMIC_RELAXED, __HIP_MEMORY_SCOPE_AGENT));
    if (t == call) return true;
    if (dt > DEADLINE_TICKS) {
        __hip_atomic_store(tmo, call, __ATOMIC_RELAXED, __HIP_MEMORY_SCOPE_AGENT);
        return true;
    }
    return false;
}

#define DEC_SLEEP 1
#define WPF_SLEEP 8
#define OUT_AUX 16
#define TOUCH 0
#define EPI_SCHED 0
#define A2A_SLEEP 1
#define PACE 0
#define SKEW 0
#define PUBW 0
#define LASTSW 0
#define XPRIO 3
#define WPRIO 2
#define POLL_SLEEP 1
#define STAMP(i) do {} while (0)
#define TB() do {} while (0)
#define TE(i) do {} while (0)
#define TFIX(i) do {} while (0)
#define STAMP_OUT() do {} while (0)

#define XL0 ((u32)((lane >> 3) * (KD * 2) + (((lane & 7) ^ (lane >> 4)) << 4)))
#define XL1 ((u32)((8 + (lane >> 3)) * (KD * 2) + (((lane & 7) ^ (4 + (lane >> 4))) << 4)))
#define DIRECT(v) ((v) < VRAW0 || (v) >= VRAW)
#define U_SRC(v) ((((v) & 15) * 32 * KD + ((v) >> 4) * 64) * 2)
#define U_XD(src, v) (OFF_XD + (u32)((v) >> 4) * 524288u + (u32)((src) * 16 + ((v) & 15)) * 4096u)
#define U_DONE(src, v) (OFF_DONE + (u32)(((v) >> 4) * 128 + (src) * 16 + ((v) & 15)) * 128u)

#define MORD 7
__device__ constexpr int mord(int i) {
    return MORD == 1 || MORD == 5 || MORD == 6 ? (i >> 2) * 4 + (((i >> 2) & 1) ? 3 - (i & 3) : (i & 3))
         : MORD == 2 ? (i % 5) * 4 + i / 5
         : MORD == 3 || MORD == 7 ? (((i / 5) & 1) ? 4 - i % 5 : i % 5) * 4 + i / 5
         : i;
}

__global__ __launch_bounds__(THREADS, 3) void ag_gemm(Params P) {
    extern __shared__ __attribute__((aligned(16))) char smem_[];
    LAS char* smem = (LAS char*)smem_;
    const int lane = threadIdx.x & 63;
    const int wid = (int)rfl(threadIdx.x >> 6);
    const int b = blockIdx.x;
    const int rank = P.rank & 7;
    char* ws = P.self;
    u32* tmo = (u32*)(ws + OFF_TMO);
    STAMP(0);
    const __amdgpu_buffer_rsrc_t rsW = mk_rsrc(ws, WS_USED);
    const int xcd = b & 7, idx = b >> 3;
    u32* seq = (u32*)(ws + OFF_SEQ) + b * 16 + wid;

    auto peer_of = [&](int i) {
        u64 a = (u64)P.peer[7];
#pragma unroll
        for (int p = 0; p < 7; ++p) { u64 t = (u64)P.peer[p]; asm("" : "+s"(t)); a = i == p ? t : a; }
        return (GAS char*)a;
    };

    const int ng = (xcd & 1) ? 26 - SKEW : 26 + SKEW;
    if (idx >= ng) {
        const int ci = idx >= 26 + SKEW ? (31 - idx) * 8 + xcd : (6 - SKEW) * 8 + (idx - ng) * 4 + (xcd >> 1);
        if (ci < NPC && wid < PWAVES) {
            const int pi = wid * NPC + ci;
            const __amdgpu_buffer_rsrc_t rsX = mk_rsrc(P.x, 512u * KD * 2u);
            const u32 xl0 = XL0, xl1 = XL1;
            u32 sq;
            asm volatile("global_load_dword %0, %1, %2 sc1" : "=&v"(sq) : "v"(0u), "s"(seq) : "memory");
            GAS char* hp = peer_of(lane >> 3) + (lane & 7) * 8;
            __amdgpu_buffer_rsrc_t rsP[7];
#pragma unroll
            for (int p = 0; p < 7; ++p) rsP[p] = mk_rsrc(P.peer[p], WS_USED);
            u32* tkt = (u32*)(ws + OFF_TKT);
            u32 call = 0, tbase = 0;
            int done = -1, cur = -1, nxt = pi;
            u32x4 pk[4] = {};
            u64 hw = 0, hw_done = 0;
            bool more = false;
            bool raw = false;
            for (;;) {
                asm volatile("" ::: "memory");
                u32 tk;
                asm volatile("s_cmp_lt_u32 %4, 1024\n\ts_cbranch_scc0 1f\n\ts_mov_b64 exec, 1\n\tglobal_atomic_add %0, %1, %2, %3 sc0\n\ts_mov_b64 exec, -1\n1:"
                             : "=&v"(tk) : "v"(0u), "v"(1u), "s"(tkt), "s"(cur) : "memory", "scc");
                u32x4 d[4];
                asm volatile("s_cmp_lt_u32 %8, 1024\n\ts_cbranch_scc0 1f\n\t"
                             "buffer_load_dwordx4 %0, %4, %6, %7 offen\n\tbuffer_load_dwordx4 %1, %5, %6, %7 offen\n\t"
                             "buffer_load_dwordx4 %2, %4, %6, %9 offen\n\tbuffer_load_dwordx4 %3, %5, %6, %9 offen\n1:"
                             : "=&v"(d[0]), "=&v"(d[1]), "=&v"(d[2]), "=&v"(d[3])
                             : "v"(xl0), "v"(xl1), "s"(rsX), "s"(U_SRC(nxt)), "s"(nxt), "s"(U_SRC(nxt) + 16 * KD * 2) : "memory", "scc");
                if (done >= 0 && lane < 56) {
                    const u32 o = !DIRECT(done) ? OFF_HDR + (u32)(rank * 1024 + done) * 64u : U_DONE(rank, done);
                    __hip_atomic_store((GAS u64*)(hp + o), DIRECT(done) ? ((u64)call << 32) | call : hw_done | (u64)call << ((lane & 7) ? 55 : 32), __ATOMIC_RELAXED, __HIP_MEMORY_SCOPE_SYSTEM);
                }
                if (cur >= 1024) break;
                TB();
                if (cur >= 0) {
                    const u32 o01 = raw ? U_XD(rank, cur) : OFF_PK + (u32)(rank * 1024 + cur) * 3072u;
                    const u32 o23 = o01 + 2048u;
                    const u32 v2 = raw || lane < (more ? 52 : 48) ? lane * 16u : 0x80000000u;
#pragma unroll
                    for (int p = 0; p < 7; ++p) {
                        __builtin_amdgcn_raw_buffer_store_b128(pk[0], rsP[p], lane * 16u, o01, AUX_SYS);
                        __builtin_amdgcn_raw_buffer_store_b128(pk[1], rsP[p], lane * 16u + 1024u, o01, AUX_SYS);
                        __builtin_amdgcn_raw_buffer_store_b128(pk[2], rsP[p], v2, o23, AUX_SYS);
                    }
                    if (raw) {
#pragma unroll
                        for (int p = 0; p < 7; ++p) __builtin_amdgcn_raw_buffer_store_b128(pk[3], rsP[p], lane * 16u + 1024u, o23, AUX_SYS);
                    }
                }
                HOLD4(pk);
                TE(13);
                const int had = done;
                done = cur; hw_done = hw;
                asm volatile("s_waitcnt vmcnt(22)\n\ts_cmp_lt_i32 %5, 0\n\ts_cbranch_scc0 1f\n\ts_waitcnt vmcnt(21)\n\ts_cmp_lt_i32 %4, 0\n\ts_cbranch_scc0 1f\n\ts_waitcnt vmcnt(0)\n1:"
                             : "+v"(d[0]), "+v"(d[1]), "+v"(d[2]), "+v"(d[3]) : "s"(cur), "s"(HDR_FIRST ? -1 : had) : "memory", "scc");
                TE(14);
                raw = true;
                if (!DIRECT(nxt)) {
                    u16x2 mx = {0, 0};
#pragma unroll
                    for (int i = 0; i < 4; ++i)
#pragma unroll
                        for (int e = 0; e < 4; ++e) mx = __builtin_elementwise_max(mx, __builtin_bit_cast(u16x2, d[i][e] & 0x7FFF7FFFu));
                    const u32 el = (u32)(mx.x > mx.y ? mx.x : mx.y) >> 7;
                    u32 top = wave_max_u32(el);
                    while (top > 7u && __builtin_popcountll(__builtin_amdgcn_ballot_w64(el >= top)) < 16) --top;
                    const u32 low = top < 7u ? 0u : top - 7u;
                    const u16x2 bias = __builtin_bit_cast(u16x2, low * 0x00800080u);
                    u32 sg = 0, c2[2] = {0, 0}, em = 0;
                    u32x4 hi[2];
#pragma unroll
                    for (int k = 0; k < 8; ++k) {
                        const u32 d0 = d[k >> 1][(k & 1) * 2], d1 = d[k >> 1][(k & 1) * 2 + 1];
                        const u32 x0 = __builtin_bit_cast(u32, (u16x2)(__builtin_bit_cast(u16x2, d0) - bias));
                        const u32 x1 = __builtin_bit_cast(u32, (u16x2)(__builtin_bit_cast(u16x2, d1) - bias));
                        const u32 h = __builtin_amdgcn_perm(x1, x0, 0x07050301u);
                        pk[k >> 2][k & 3] = __builtin_amdgcn_perm(x1, x0, 0x06040200u);
                        hi[k >> 2][k & 3] = h;
                        sg |= (h >> k) & (0x80808080u >> k);
                        c2[k >> 2] |= (h & 0x03030303u) << (2 * (k & 3));
                        em |= (((h & 0x7C7C7C7Cu) + 0x7C7C7C7Cu) >> k) & (0x80808080u >> k);
                    }
                    LAS char* L = smem + wid * 4096;
                    *(LAS u32*)(L + lane * 4) = sg;
                    *(LAS u32*)(L + 256 + lane * 4) = c2[0];
                    *(LAS u32*)(L + 512 + lane * 4) = c2[1];
                    *(LAS u32x4*)(L + 1024 + lane * 32) = hi[0];
                    *(LAS u32x4*)(L + 1040 + lane * 32) = hi[1];
                    asm volatile("" ::: "memory");
                    u32 n = 0;
                    u64 m = __builtin_amdgcn_ballot_w64(em != 0u);
                    while (m != 0ull && n <= 64u) {
                        const u32 idx = n + __builtin_amdgcn_mbcnt_hi((u32)(m >> 32), __builtin_amdgcn_mbcnt_lo((u32)m, 0u));
                        const u32 bit = (u32)__builtin_ctz(em | 0x80000000u);
                        if (em != 0u) *(LAS unsigned short*)(L + 3072 + idx * 2u) = (unsigned short)((u32)lane * 32u + (7u - (bit & 7u)) * 4u + (bit >> 3));
                        em &= em - 1u;
                        n += (u32)__builtin_popcountll(m);
                        m = __builtin_amdgcn_ballot_w64(em != 0u);
                    }
                    if (lane < 16) *(LAS u32*)(L + 768 + lane * 4) = 0u;
                    asm volatile("" ::: "memory");
                    const u32 pos = *(LAS unsigned short*)(L + 3072 + lane * 2) & 2047u;
                    const u32 hb = *(LAS unsigned char*)(L + 1024 + pos) & 0x7Cu;
                    const u64 ms = __builtin_amdgcn_ballot_w64((u32)lane < n && hb == 0x7Cu);
                    const u32 is = __builtin_amdgcn_mbcnt_hi((u32)(ms >> 32), __builtin_amdgcn_mbcnt_lo((u32)ms, 0u));
                    const bool sh = hb == 0x7Cu && is < 36u;
                    const u32 il = (u32)lane - (is < 36u ? is : 36u);
                    const u32 nsa = (u32)__builtin_popcountll(ms), ns = nsa < 36u ? nsa : 36u;
                    if ((u32)lane < n && (sh || il < 32u)) *(LAS unsigned short*)(L + (sh ? 832u + is * 2u : 768u + il * 2u)) = (unsigned short)(sh ? pos : pos | hb << 9);
                    asm volatile("" ::: "memory");
                    pk[2] = *(LAS u32x4*)(L + lane * 16);
                    raw = n > 64u || n - ns > 32u;
                    more = n != ns;
                    u32 e[5];
#pragma unroll
                    for (int i = 0; i < 5; ++i) e[i] = *(LAS unsigned short*)(L + 824 + (lane & 7) * 10 + i * 2) & 2047u;
                    hw = (lane & 7) ? (u64)(e[0] | e[1] << 11 | e[2] << 22) | (u64)(e[2] >> 10 | e[3] << 1 | e[4] << 12) << 32
                                    : (u64)(low | (raw ? 256u : 0u) | (more ? 512u : 0u) | ns << 10 | e[4] << 16);
                    asm volatile("" ::: "memory");
                }
                if (raw) {
#pragma unroll
                    for (int i = 0; i < 4; ++i) pk[i] = d[i];
                }
                asm volatile("s_waitcnt vmcnt(0)" : "+v"(tk), "+v"(sq) :: "memory");
                TE(15);
                const bool first = cur < 0;
                cur = nxt;
                if (first) {
                    call = rfl(sq) + 1u;
                    tbase = (call - 1u) * 1024u - 2u * NPUSH;
                    if (lane == 0) __hip_atomic_store(seq, call, __ATOMIC_RELAXED, __HIP_MEMORY_SCOPE_AGENT);
                    nxt = pi + NPUSH;
                    STAMP(2);
                    for (int i = NRAWF; i < wid; ++i) __builtin_amdgcn_s_sleep(STAGGER);
                } else {
                    nxt = (int)(rfl(tk) - tbase);
                }
            }
            STAMP(3);
            TFIX(13); TFIX(14); TFIX(15);
            STAMP_OUT();
            return;
        }
        if (wid >= DWAVES || ci < DC0 || ci >= DC0 + NDC) return;
        const u32 call = rfl(__hip_atomic_load(seq, __ATOMIC_RELAXED, __HIP_MEMORY_SCOPE_AGENT)) + 1u;
        if (lane == 0) __hip_atomic_store(seq, call, __ATOMIC_RELAXED, __HIP_MEMORY_SCOPE_AGENT);
        {
            u32* pend = nullptr;
            LAS char* L = smem + 65536 + wid * 2048;
            *(LAS u32x4*)(L + lane * 32) = (u32x4){0u, 0u, 0u, 0u};
            *(LAS u32x4*)(L + lane * 32 + 16) = (u32x4){0u, 0u, 0u, 0u};
            const int hg = lane < 36 ? (lane + 4) / 5 : 0;
            const int hs = hg ? 11 * (lane + 4 - 5 * hg) : 16;
            const u32 tsh = hg ? 23u : 0u, tag = hg ? call & 511u : call;
            for (int q = VRAW0 * 7 + (wid - DW0) * NDC + ci - DC0; q < VRAW * 7; q += NDEC) {
                const int v = q / 7, src = (rank + 1 + q - v * 7) & 7;
                const u32 xdb = U_XD(src, v);
                const u64* hp = (const u64*)(ws + OFF_HDR + src * 65536) + v * 8 + hg;
                u32 hw, h0;
                bool dead = false;
                {
                    const u64 t0 = __builtin_amdgcn_s_memrealtime();
                    for (;;) {
                        const u64 g = __hip_atomic_load(hp, __ATOMIC_RELAXED, __HIP_MEMORY_SCOPE_SYSTEM);
                        if (TOUCH) { const u32 tw = __builtin_amdgcn_raw_buffer_load_b32(rsW, (lane < 45 ? lane : 0) * 64u, OFF_PK + (u32)(src * 1024 + v) * 3072u, AUX_SYS); asm volatile("" :: "v"(tw)); }
                        hw = (u32)(g >> hs) & 2047u; h0 = rfl((u32)g);
                        u64 ok = __builtin_amdgcn_ballot_w64((u32)(g >> 32) >> tsh == tag);
                        asm volatile("s_waitcnt vmcnt(0)" : "+s"(ok), "+s"(h0), "+v"(hw) :: "memory");
                        if (pend) {
                            if (lane == 0) __hip_atomic_store(pend, call, __ATOMIC_RELAXED, __HIP_MEMORY_SCOPE_AGENT);
                            pend = nullptr;
                        }
                        if (ok == ~0ull) break;
                        if (give_up(tmo, call, t0)) { dead = true; break; }
                        __builtin_amdgcn_s_sleep(DEC_SLEEP);
                    }
                }
                if (dead) break;
                asm volatile("" ::: "memory");
                if (!(h0 & 256u)) {
                    const u32 pk = OFF_PK + (u32)(src * 1024 + v) * 3072u;
                    const u32 ent = __builtin_amdgcn_raw_buffer_load_b16(rsW, (h0 & 512u) && lane < 32 ? 2816u + lane * 2u : 0x80000000u, pk, AUX_SYS);
                    const u32 sg = __builtin_amdgcn_raw_buffer_load_b32(rsW, 2048u + lane * 4u, pk, AUX_SYS);
                    u32 c2[2];
                    c2[0] = __builtin_amdgcn_raw_buffer_load_b32(rsW, 2304u + lane * 4u, pk, AUX_SYS);
                    c2[1] = __builtin_amdgcn_raw_buffer_load_b32(rsW, 2560u + lane * 4u, pk, AUX_SYS);
                    u32x4 lo[2];
                    lo[0] = __builtin_amdgcn_raw_buffer_load_b128(rsW, lane * 16u, pk, AUX_SYS);
                    lo[1] = __builtin_amdgcn_raw_buffer_load_b128(rsW, lane * 16u + 1024u, pk, AUX_SYS);
                    const bool sh = (u32)lane < ((h0 >> 10) & 63u);
                    if (sh) *(LAS unsigned char*)(L + hw) = 0x7C;
                    if (ent >= 2048u) *(LAS unsigned char*)(L + (ent & 2047u)) = (unsigned char)((ent >> 11) << 2);
                    asm volatile("" ::: "memory");
                    u32x4 ex[2];
                    ex[0] = *(LAS u32x4*)(L + lane * 32);
                    ex[1] = *(LAS u32x4*)(L + lane * 32 + 16);
                    asm volatile("" ::: "memory");
                    if (sh) *(LAS unsigned char*)(L + hw) = 0;
                    if (ent >= 2048u) *(LAS unsigned char*)(L + (ent & 2047u)) = 0;
                    const u16x2 bias = __builtin_bit_cast(u16x2, (h0 & 255u) * 0x00800080u);
#pragma unroll
                    for (int i = 0; i < 4; ++i) {
                        u32x4 o;
#pragma unroll
                        for (int j = 0; j < 2; ++j) {
                            const int k = 2 * i + j;
                            const u32 h = ((sg << k) & 0x80808080u) | ((c2[k >> 2] >> (2 * (k & 3))) & 0x03030303u) | ex[k >> 2][k & 3];
                            const u32 l = lo[k >> 2][k & 3];
                            o[2 * j] = __builtin_bit_cast(u32, (u16x2)(__builtin_bit_cast(u16x2, __builtin_amdgcn_perm(h, l, 0x05010400u)) + bias));
                            o[2 * j + 1] = __builtin_bit_cast(u32, (u16x2)(__builtin_bit_cast(u16x2, __builtin_amdgcn_perm(h, l, 0x07030602u)) + bias));
                        }
                        __builtin_amdgcn_raw_buffer_store_b128(o, rsW, lane * 16u + i * 1024u, xdb, AUX_AGT);
                        HOLD(o);
                    }
                }
                asm volatile("" ::: "memory");
                pend = (u32*)(ws + U_DONE(src, v));
            }
            WAIT_V(0);
            if (pend && lane == 0) __hip_atomic_store(pend, call, __ATOMIC_RELAXED, __HIP_MEMORY_SCOPE_AGENT);
            STAMP(3);
            TFIX(4); TFIX(5); TFIX(6);
        }
        STAMP_OUT();
        return;
    }

    const int T = xcd * 26 + (xcd & 1) * SKEW + idx;
    const int ct = (T / 104) * 4 + (T % 104 & 3), rt = (T % 104) >> 2;
    const int rg0 = rt * 10;
    const int nblk = rt == 25 ? 3 : 5;
    const u32 lds0 = (u32)(size_t)smem;
#define DMA(ldsaddr, voff, rs, soff, mods) asm volatile("s_mov_b32 m0, %0\n\ts_nop 0\n\tbuffer_load_dwordx4 %1, %2, %3 offen" mods " lds" \
        :: "s"(ldsaddr), "v"(voff), "s"(rs), "s"(soff) : "memory")
#define DMA_SAME_M0(voff, rs, soff, mods) asm volatile("buffer_load_dwordx4 %0, %1, %2 offen" mods " lds" :: "v"(voff), "s"(rs), "s"(soff) : "memory")

    if (wid == 11) return;
    if (wid >= 9) {
        __builtin_amdgcn_s_setprio(WPRIO);
        const int h = wid - 9;
        const __amdgpu_buffer_rsrc_t rsB = mk_rsrc(P.w, (u32)KD * NL * 2u);
        u32 vB[16];
#pragma unroll
        for (int i = 0; i < 16; ++i) {
            const int row = i * 2 + (lane >> 5);
            vB[i] = (u32)((h * 32 + row) * (NL * 2) + ct * 512 + (((lane & 31) ^ ((row & 3) | (row & 8))) << 4));
        }
        const u32 lB = lds0 + 20480 + h * 16384;
#define LOAD_W(kt, so) do { _Pragma("unroll") for (int i = 0; i < 16; ++i) DMA(lB + (so) + i * 1024, vB[i], rsB, (kt) * 262144, ""); } while (0)
        LOAD_W(0, 0);
        LOAD_W(1, STAGE);
        WAIT_V(16);
        BAR();
        int so = 2 * STAGE;
        for (int t = 0; t < 62; ++t) {
            { TB(); LOAD_W(t + 2, so); TE(5); }
            so = so == 2 * STAGE ? 0 : so + STAGE;
            { TB(); WAIT_V(16); TE(6); }
            { TB(); BAR(); TE(7); }
        }
        WAIT_V(0);
        BAR();
        TFIX(5); TFIX(6); TFIX(7);
        STAMP_OUT();
        return;
    }

    if (wid == 8) {
        __builtin_amdgcn_s_setprio(XPRIO);
        const u32 call = rfl(__hip_atomic_load(seq, __ATOMIC_RELAXED, __HIP_MEMORY_SCOPE_AGENT)) + 1u;
        if (lane == 0) __hip_atomic_store(seq, call, __ATOMIC_RELAXED, __HIP_MEMORY_SCOPE_AGENT);
        const u32* cp = (const u32*)(ws + OFF_DONE + (PACE && (lane & 7) == 7 ? (((rg0 >> 1) + 16) & 127) : (rg0 >> 1) + ((lane & 7) < nblk ? (lane & 7) : 0)) * 128);
        const u32 vA = (u32)lane * 16u;
        const u32 gA = OFF_XD + (u32)rg0 * 2048u;
        int ready = 0;
        auto look = [&]() {
            const int c = ready + (lane >> 3);
            const u32 f = __hip_atomic_load(cp + (c < 63 ? c : 63) * 4096, __ATOMIC_RELAXED, __HIP_MEMORY_SCOPE_SYSTEM);
            const u64 miss = __builtin_amdgcn_ballot_w64(f < call);
            ready += miss == 0ull ? 8 : __builtin_ctzll(miss) >> 3;
        };
        auto need = [&](int kt) {
            if (kt < ready) return;
            const u64 t0 = __builtin_amdgcn_s_memrealtime();
            for (;;) {
                look();
                if (kt < ready) break;
                if (give_up(tmo, call, t0)) { ready = 64; break; }
                __builtin_amdgcn_s_sleep(POLL_SLEEP);
            }
        };
#define LOAD_X4(g, l) do { \
            DMA((l), vA, rsW, (g), " sc0 sc1"); DMA_SAME_M0(vA, rsW, (g), " offset:1024 sc0 sc1"); DMA_SAME_M0(vA, rsW, (g), " offset:2048 sc0 sc1"); DMA_SAME_M0(vA, rsW, (g), " offset:3072 sc0 sc1"); } while (0)
#define XST(kt) do {} while (0)
#define LOAD_X(kt, so) do { XST(kt); _Pragma("unroll") for (int i = 0; i < 5; ++i) \
            LOAD_X4(gA + (kt) * 524288 + (i < nblk ? i : 0) * 4096, lds0 + (so) + (i < nblk ? i : 0) * 4096); } while (0)
        int xi = 0, so = 0;
        for (int t = -1; t < 63; ++t) {
            if (xi == t + 1) {
                { TB(); need(xi); TE(10); }
                LOAD_X(xi, so);
                ++xi; so = so == 2 * STAGE ? 0 : so + STAGE;
            }
            if (xi < 64 && xi >= ready) { TB(); look(); TE(11); }
            if (xi < 64 && xi < ready) {
                LOAD_X(xi, so);
                ++xi; so = so == 2 * STAGE ? 0 : so + STAGE;
                TB(); WAIT_V(20); TE(12);
            } else {
                TB(); WAIT_V(0); TE(13);
            }
            { TB(); BAR(); TE(14); }
        }
        STAMP(4);
        TFIX(10); TFIX(11); TFIX(12); TFIX(13); TFIX(14);
        STAMP_OUT();
        return;
    }

    const int wm = wid >> 2, wn = wid & 3;
    const u32 call = rfl(__hip_atomic_load(seq, __ATOMIC_RELAXED, __HIP_MEMORY_SCOPE_AGENT)) + 1u;
    if (lane == 0) __hip_atomic_store(seq, call, __ATOMIC_RELAXED, __HIP_MEMORY_SCOPE_AGENT);
    if (wid * 208 + T < 1024) {
        const int v = wid * 208 + T;
        const __amdgpu_buffer_rsrc_t rsX = mk_rsrc(P.x, 512u * KD * 2u);
        u32x4 d[4];
#pragma unroll
        for (int i = 0; i < 4; ++i) d[i] = __builtin_amdgcn_raw_buffer_load_b128(rsX, (i & 1) ? XL1 : XL0, U_SRC(v) + (i >> 1) * 16 * KD * 2, 0);
#pragma unroll
        for (int i = 0; i < 4; ++i) __builtin_amdgcn_raw_buffer_store_b128(d[i], rsW, lane * 16u + i * 1024u, U_XD(rank, v), AUX_AGT);
        HOLD4(d);
        WAIT_V(0);
        if (lane == 0) __hip_atomic_store((u32*)(ws + U_DONE(rank, v)), call, __ATOMIC_RELAXED, __HIP_MEMORY_SCOPE_AGENT);
    }

    {
        if (WPF_SLEEP) __builtin_amdgcn_s_sleep(WPF_SLEEP);
        const __amdgpu_buffer_rsrc_t rsB = mk_rsrc(P.w, (u32)KD * NL * 2u);
        u32 t[3];
#pragma unroll
        for (int i = 0; i < 3; ++i) {
            const u32 r = (u32)(i * 1664 + T * 8 + wid);
            t[i] = __builtin_amdgcn_raw_buffer_load_b32(rsB, lane * 64u, (r < 4096u ? r : 0u) * 4096u, 0);
        }
        asm volatile("" :: "v"(t[0]), "v"(t[1]), "v"(t[2]));
    }
    if (nblk < 5) {
#pragma unroll
        for (int i = 0; i < 3; ++i) *(LAS u32x4*)(smem + i * STAGE + 12288 + wid * 1024 + lane * 16) = (u32x4){0u, 0u, 0u, 0u};
        WAIT_L(0);
    }
    u32 pa[2], pb[4];
#pragma unroll
    for (int h = 0; h < 2; ++h) pa[h] = lds0 + wm * 10240 + (lane & 15) * 128 + (((4 * h + (lane >> 4)) ^ ((lane >> 1) & 7)) << 4);
    {
        const int g = lane >> 4, q = (lane >> 2) & 3, p = lane & 3;
#pragma unroll
        for (int j = 0; j < 4; ++j) {
            const int jj = j ^ (p & 1);
            pb[j] = lds0 + 20480 + (8 * g + q) * 512 + (((wn * 8 + 2 * p + (jj >> 1)) ^ (q | ((g & 1) << 3))) << 4) + (jj & 1) * 8;
        }
    }
    f32x4 acc[5][4];
#pragma unroll
    for (int m = 0; m < 5; ++m)
#pragma unroll
        for (int j = 0; j < 4; ++j) acc[m][j] = (f32x4){0.f, 0.f, 0.f, 0.f};

    s16x4 f0b[8], f1b[8];
    bf16x8 f0a[5], f1a[5];
#define RD(i, fa, fb, h) do { \
        if ((i) < 8) fb[(i) & 7] = __builtin_amdgcn_ds_read_tr16_b64_v4i16((LAS s16x4*)(size_t)(pb[((i) & 7) >> 1] + (h) * 16384 + ((i) & 1) * 2048)); \
        else fa[(i) < 8 ? 0 : (i) - 8] = *(LAS bf16x8*)(size_t)(pa[h] + ((i) - 8) * 2048); } while (0)
#define MM(i, fa, fb) acc[(i) >> 2][(i) & 3] = __builtin_amdgcn_mfma_f32_16x16x32_bf16( \
        __builtin_bit_cast(bf16x8, __builtin_shufflevector(fb[((i) & 3) * 2], fb[((i) & 3) * 2 + 1], 0, 1, 2, 3, 4, 5, 6, 7)), fa[(i) >> 2], acc[(i) >> 2][(i) & 3], 0, 0, 0)
#define VADD(v, s) asm volatile("v_add_u32 %0, %1, %0" : "+v"(v) : "s"(s))
    BAR();
    STAMP(3);
    if (wm == 0) __builtin_amdgcn_s_setprio(1);
    int ph = 0;
    for (int t = 0; t < 64; ++t) {
        SCHED();
#pragma unroll
        for (int i = 0; i < 13; ++i) RD(i, f0a, f0b, 0);
#pragma unroll
        for (int i = 0; i < 13; ++i) RD(i, f1a, f1b, 1);
        SCHED();
        {
            const int dlt = ph == 2 ? -2 * STAGE : STAGE;
            ph = ph == 2 ? 0 : ph + 1;
            VADD(pa[0], dlt); VADD(pa[1], dlt);
#pragma unroll
            for (int j = 0; j < 4; ++j) VADD(pb[j], dlt);
        }
        const int who = t == 63 ? 2 : (LASTSW && t >= 63 - LASTSW) ? 1 : wm;
        asm volatile("s_cmp_lg_u32 %0, 0\n\ts_cbranch_scc1 1f\n\ts_waitcnt lgkmcnt(0)\n\ts_barrier\n1:" :: "s"(who) : "memory", "scc");
        SCHED();
#pragma unroll
        for (int i = 0; i < 20; ++i) {
            if (MORD >= 6 && (i & 1)) { MM(mord(i), f1a, f1b); SCHED(); MM(mord(i), f0a, f0b); SCHED(); }
            else { MM(mord(i), f0a, f0b); SCHED(); MM(mord(i), f1a, f1b); SCHED(); }
        }
        SCHED();
        asm volatile("s_cmp_lg_u32 %0, 1\n\ts_cbranch_scc1 1f\n\ts_barrier\n1:" :: "s"(who) : "memory", "scc");
    }
    __builtin_amdgcn_s_setprio(0);
    STAMP(4);
    TFIX(14);

    const u32 tmx = OFF_TMAX + (call & 1u) * 16384u;
    float mx[4] = {0.f, 0.f, 0.f, 0.f};
#pragma unroll
    for (int m = 0; m < 5; ++m)
#pragma unroll
        for (int j = 0; j < 4; ++j) {
            mx[j] = __builtin_fmaxf(__builtin_fmaxf(mx[j], acc[m][j][0]), acc[m][j][1]);
            mx[j] = __builtin_fmaxf(__builtin_fmaxf(mx[j], acc[m][j][2]), acc[m][j][3]);
        }
    LAS u32* misc = (LAS u32*)(smem + LDS_MISC + 512);
    {
        const u32 wmx = wave_max_u32(umax(umax(__float_as_uint(mx[0]), __float_as_uint(mx[1])), umax(__float_as_uint(mx[2]), __float_as_uint(mx[3]))));
        if (lane == 0) misc[wid] = wmx;
    }
    if (wid == PUBW) {
        GAS u64* tdst = (GAS u64*)(peer_of(lane & 7) + tmx) + rank * 256 + T;
        asm volatile("s_waitcnt lgkmcnt(0)" : "+v"(tdst) :: "memory");
        BAR();
        const u32x4 a = *(LAS u32x4*)misc, c = *(LAS u32x4*)(misc + 4);
        const u32 m = umax(umax(umax(a[0], a[1]), umax(a[2], a[3])), umax(umax(c[0], c[1]), umax(c[2], c[3])));
        if (lane < NR) __hip_atomic_store(tdst, ((u64)call << 32) | m, __ATOMIC_RELAXED, __HIP_MEMORY_SCOPE_SYSTEM);
    } else {
        WAIT_L(0);
        BAR();
    }
    {
        const bool sw = (lane >> 4) & 1;
#pragma unroll
        for (int m = 0; m < 5; ++m)
#pragma unroll
            for (int j = 0; j < 4; j += 2)
#pragma unroll
                for (int r = 0; r < 4; ++r) {
                    const float u = __builtin_fmaxf(acc[m][j][r], 0.f), v = __builtin_fmaxf(acc[m][j + 1][r], 0.f);
                    acc[m][j][r] = sw ? v : u; acc[m][j + 1][r] = sw ? u : v;
                }
#pragma unroll
        for (int m = 0; m < 5; ++m) asm volatile("" : "+v"(acc[m][0]), "+v"(acc[m][1]), "+v"(acc[m][2]), "+v"(acc[m][3]));
    }
    const __amdgpu_buffer_rsrc_t rsO = mk_rsrc(P.out, 4096u * NL);
    u32 oo = (u32)((rg0 * 16 + wm * 80 + (lane & 15)) * NL + ct * 256 + wn * 64 + (lane >> 4) * 16);
    asm volatile("" : "+v"(oo));
    {
        const u64 t0 = __builtin_amdgcn_s_memrealtime();
        u32 m;
        bool bad;
        for (;;) {
            const u32x4 a = __builtin_amdgcn_raw_buffer_load_b128(rsW, lane * 16u, tmx + wid * 2048, AUX_SYS);
            const u32x4 c = __builtin_amdgcn_raw_buffer_load_b128(rsW, lane < 40 ? tmx + wid * 2048 + 1024 + lane * 16u : OFF_TMO, 0, AUX_SYS);
            const bool ok = a[1] == call && a[3] == call && (lane >= 40 || (c[1] == call && c[3] == call));
            m = umax(a[0], a[2]);
            if (lane < 40) m = umax(m, umax(c[0], c[2]));
            bad = (u32)__builtin_amdgcn_readlane((int)c[0], 63) == call;
            if (__builtin_amdgcn_ballot_w64(ok) == ~0ull) break;
            if (give_up(tmo, call, t0)) { bad = true; break; }
            if (A2A_SLEEP) __builtin_amdgcn_s_sleep(A2A_SLEEP);
        }
        m = bad ? 0x7FC00000u : wave_max_u32(m);
        if (lane == 0) misc[16 + wid] = m;
        WAIT_L(0);
    }
    BAR();
    STAMP(5);
    {
        const u32x4 a = *(LAS u32x4*)(misc + 16), c = *(LAS u32x4*)(misc + 20);
        const u32 m = umax(umax(umax(a[0], a[1]), umax(a[2], a[3])), umax(umax(c[0], c[1]), umax(c[2], c[3])));
        const float inv = 448.0f / __uint_as_float(m);
        const f32x4 inv4 = {inv, inv, inv, inv};
#pragma unroll
        for (int mt = 0; mt < 5; ++mt) {
            u32x4 o;
#pragma unroll
            for (int j = 0; j < 4; ++j) {
                const f32x4 y = acc[mt][j] * inv4;
                int t = __builtin_amdgcn_cvt_pk_fp8_f32(y[0], y[1], __float_as_int(y[0]), false);
                t = __builtin_amdgcn_cvt_pk_fp8_f32(y[2], y[3], t, true);
                o[j] = (u32)t;
            }
            __builtin_amdgcn_raw_buffer_store_b128(o, rsO, oo + mt * 16 * NL, 0, OUT_AUX);
            HOLD(o);
        }
    }
    STAMP(6);
    STAMP_OUT();
}

}

extern "C" void kernel_launch(int rank, int nranks, ncclComm_t comm,
                              hipStream_t stream,
                              void* const* d_in, const int* in_sizes, int n_in,
                              void* d_out, int out_size,
                              const int* in_dtypes, int out_dtype,
                              void* const* d_out_all,
                              void* const* d_ws_all, size_t ws_bytes) {
    using namespace agk;
    if (nranks != NR || (unsigned)rank >= (unsigned)NR || ws_bytes < WS_USED || n_in < 2 ||
        in_sizes[0] != 512 * KD || in_sizes[1] != KD * NL || out_size != 4096 * NL) {
        fprintf(stderr, "unexpected problem shape\n");
        return;
    }
    static bool attr_done = false;
    if (!attr_done) {
        HIP_CHECK(hipFuncSetAttribute((const void*)ag_gemm, hipFuncAttributeMaxDynamicSharedMemorySize, LDS_BYTES));
        attr_done = true;
    }
    Params P;
    P.x = d_in[0]; P.w = d_in[1]; P.out = d_out;
    P.self = (char*)d_ws_all[rank];
    for (int p = 0; p < NR; ++p) P.peer[p] = (char*)d_ws_all[(rank + 1 + p) % NR];
    P.rank = rank; P.pad = 0;
    hipLaunchKernelGGL(ag_gemm, dim3(GRID), dim3(THREADS), LDS_BYTES, stream, P);
    HIP_CHECK(hipGetLastError());
}
```

```cpp
#include <hip/hip_runtime.h>
#include <rccl/rccl.h>
#include <cstdio>

#define HIP_CHECK(expr) do { hipError_t _e = (expr); if (_e != hipSuccess) { \
    fprintf(stderr, "HIP error %s at %s:%d\n", hipGetErrorString(_e), __FILE__, __LINE__); \
    return; } } while (0)

namespace agk {

typedef unsigned u32;
typedef unsigned long long u64;
typedef u32 u32x4 __attribute__((ext_vector_type(4)));
typedef float f32x4 __attribute__((ext_vector_type(4)));
typedef __bf16 bf16x8 __attribute__((ext_vector_type(8)));
typedef short s16x4 __attribute__((ext_vector_type(4)));
typedef unsigned short u16x2 __attribute__((ext_vector_type(2)));
#define LAS __attribute__((address_space(3)))
#define GAS __attribute__((address_space(1)))

constexpr int NR = 8;
constexpr int KD = 4096;
constexpr int NL = 2048;
constexpr int GRID = 256, THREADS = 768;
#define NPC 22
#define BK_N 48
#define BK_S 16
#define HS 64
#define TS 32
#define XS 1
#define DSL 1
#define DS 64
#define XLOOK 1
constexpr int PWAVES = 2, NPUSH = NPC * PWAVES;
constexpr int NDC = 48 - NPC, NDEC = NDC * 12;
constexpr int VRAW0 = 16, VRAW = 1024 - 32;
constexpr int STAGE = 52 * 1024;
constexpr int LDS_MISC = 3 * STAGE;
constexpr int LDS_BYTES = LDS_MISC + 1024;

constexpr u32 OFF_SEQ = 0;
constexpr u32 OFF_TMO = 16384;
constexpr u32 OFF_TMAX = 512u << 10;
constexpr u32 OFF_TKT = 32768;
constexpr u32 OFF_DBG = 128u << 10;
constexpr u32 OFF_HDR = 2u << 20;
constexpr u32 OFF_DONE = 1u << 20;
constexpr u32 OFF_XD = 4u << 20;
constexpr u32 OFF_PK = 36u << 20;
constexpr u32 WS_USED = 60u << 20;

constexpr u64 DEADLINE_TICKS = 40000000ull;
constexpr int AUX_SYS = 17, AUX_AGT = 16;

struct Params {
    const void* x;
    const void* w;
    void* out;
    char* self;
    char* peer[NR];
    int rank;
    int pad;
};

#define WAIT_V(n) asm volatile("s_waitcnt vmcnt(" #n ")" ::: "memory")
#define WAIT_L(n) asm volatile("s_waitcnt lgkmcnt(" #n ")" ::: "memory")
#define BAR() do { asm volatile("" ::: "memory"); __builtin_amdgcn_s_barrier(); asm volatile("" ::: "memory"); } while (0)
#define SCHED() __builtin_amdgcn_sched_barrier(0)

__device__ __forceinline__ u32 rfl(u32 v) { return __builtin_amdgcn_readfirstlane(v); }
__device__ __forceinline__ __amdgpu_buffer_rsrc_t mk_rsrc(const void* p, u32 bytes) {
    return __builtin_amdgcn_make_buffer_rsrc((void*)p, 0, (int)bytes, 0x00020000);
}
#define HOLD(x) asm volatile("s_nop 1" :: "v"(x) : "memory")
#define HOLD4(x) asm volatile("s_nop 1" :: "v"(x[0]), "v"(x[1]), "v"(x[2]), "v"(x[3]) : "memory")
__device__ __forceinline__ u32 umax(u32 a, u32 b) { return a > b ? a : b; }
__device__ __forceinline__ u32 wave_max_u32(u32 v) {
    v = umax(v, (u32)__builtin_amdgcn_update_dpp(0, (int)v, 0x111, 0xf, 0xf, true));
    v = umax(v, (u32)__builtin_amdgcn_update_dpp(0, (int)v, 0x112, 0xf, 0xf, true));
    v = umax(v, (u32)__builtin_amdgcn_update_dpp(0, (int)v, 0x114, 0xf, 0xf, true));
    v = umax(v, (u32)__builtin_amdgcn_update_dpp(0, (int)v, 0x118, 0xf, 0xf, true));
    return umax(umax((u32)__builtin_amdgcn_readlane((int)v, 15), (u32)__builtin_amdgcn_readlane((int)v, 31)),
                umax((u32)__builtin_amdgcn_readlane((int)v, 47), (u32)__builtin_amdgcn_readlane((int)v, 63)));
}
__device__ __forceinline__ bool give_up(u32* tmo, u32 call, u64 t0) {
    const u64 dt = __builtin_amdgcn_s_memrealtime() - t0;
    if (dt < 20000ull) return false;
    const u32 t = rfl(__hip_atomic_load(tmo, __ATOMIC_RELAXED, __HIP_MEMORY_SCOPE_AGENT));
    if (t == call) return true;
    if (dt > DEADLINE_TICKS) {
        __hip_atomic_store(tmo, call, __ATOMIC_RELAXED, __HIP_MEMORY_SCOPE_AGENT);
        return true;
    }
    return false;
}

#define STAMP(i) do {} while (0)
#define STAMP_OUT() do {} while (0)

#define XL0 ((u32)((lane >> 3) * (KD * 2) + (((lane & 7) ^ (lane >> 4)) << 4)))
#define XL1 ((u32)((8 + (lane >> 3)) * (KD * 2) + (((lane & 7) ^ (4 + (lane >> 4))) << 4)))
#define DIRECT(v) ((v) < VRAW0 || (v) >= VRAW)
#define U_SRC(v) ((((v) & 15) * 32 * KD + ((v) >> 4) * 64) * 2)
#define U_XD(src, v) (OFF_XD + (u32)((v) >> 4) * 524288u + (u32)((src) * 16 + ((v) & 15)) * 4096u)
#define U_DONE(src, v) (OFF_DONE + (u32)(((v) >> 4) * 128 + (src) * 16 + ((v) & 15)) * DS)

__device__ constexpr int mord(int i) { return (((i / 5) & 1) ? 4 - i % 5 : i % 5) * 4 + i / 5; }

__global__ __launch_bounds__(THREADS, 3) void ag_gemm(Params P) {
    extern __shared__ __attribute__((aligned(16))) char smem_[];
    LAS char* smem = (LAS char*)smem_;
    const int lane = threadIdx.x & 63;
    const int wid = (int)rfl(threadIdx.x >> 6);
    const int b = blockIdx.x;
    const int rank = P.rank & 7;
    char* ws = P.self;
    u32* tmo = (u32*)(ws + OFF_TMO);
    STAMP(0);
    const __amdgpu_buffer_rsrc_t rsW = mk_rsrc(ws, WS_USED);
    u32* seq = (u32*)(ws + OFF_SEQ) + b * 16 + wid;

    auto peer_of = [&](int i) {
        u64 a = (u64)P.peer[7];
#pragma unroll
        for (int p = 0; p < 7; ++p) { u64 t = (u64)P.peer[p]; asm("" : "+s"(t)); a = i == p ? t : a; }
        return (GAS char*)a;
    };

    if (b < 48) {
        if (b < NPC) {
            if (wid >= PWAVES) return;
            const int pi = wid * NPC + b;
            const __amdgpu_buffer_rsrc_t rsX = mk_rsrc(P.x, 512u * KD * 2u);
            const u32 xl0 = XL0, xl1 = XL1;
            u32x4 d[4];
            u32 sq;
            asm volatile("buffer_load_dwordx4 %0, %5, %7, %8 offen\n\tbuffer_load_dwordx4 %1, %6, %7, %8 offen\n\t"
                         "buffer_load_dwordx4 %2, %5, %7, %9 offen\n\tbuffer_load_dwordx4 %3, %6, %7, %9 offen\n\tglobal_load_dword %4, %10, %11 sc1"
                         : "=&v"(d[0]), "=&v"(d[1]), "=&v"(d[2]), "=&v"(d[3]), "=&v"(sq)
                         : "v"(xl0), "v"(xl1), "s"(rsX), "s"(U_SRC(pi)), "s"(U_SRC(pi) + 16 * KD * 2), "v"(0u), "s"(seq) : "memory");
            SCHED();
            GAS char* hp = peer_of(lane >> 3) + (lane & 7) * 8;
            __amdgpu_buffer_rsrc_t rsP[7];
#pragma unroll
            for (int p = 0; p < 7; ++p) rsP[p] = mk_rsrc(P.peer[p], WS_USED);
            u32* tkt = (u32*)(ws + OFF_TKT);
            u32 call = 0, tbase = 0;
            int done = -1, cur = -1, nxt = pi;
            u32x4 pk[4] = {};
            u64 hw = 0, hw_done = 0;
            bool more = false;
            u32 low = 256u;
            bool raw = false;
            for (;;) {
                asm volatile("" ::: "memory");
                u32 tk;
                asm volatile("s_cmp_lt_u32 %4, 1024\n\ts_cbranch_scc0 1f\n\ts_mov_b64 exec, 1\n\tglobal_atomic_add %0, %1, %2, %3 sc0\n\ts_mov_b64 exec, -1\n1:"
                             : "=&v"(tk) : "v"(0u), "v"(1u), "s"(tkt), "s"(cur) : "memory", "scc");
                asm volatile("s_cmp_lt_u32 %8, 1024\n\ts_cbranch_scc0 1f\n\t"
                             "buffer_load_dwordx4 %0, %4, %6, %7 offen\n\tbuffer_load_dwordx4 %1, %5, %6, %7 offen\n\t"
                             "buffer_load_dwordx4 %2, %4, %6, %9 offen\n\tbuffer_load_dwordx4 %3, %5, %6, %9 offen\n1:"
                             : "+v"(d[0]), "+v"(d[1]), "+v"(d[2]), "+v"(d[3])
                             : "v"(xl0), "v"(xl1), "s"(rsX), "s"(U_SRC(nxt)), "s"(cur < 0 ? 1024 : nxt), "s"(U_SRC(nxt) + 16 * KD * 2) : "memory", "scc");
                if (done >= 0 && lane < 56 && (!DIRECT(done) || (lane & 7) == 0)) {
                    const u32 o = !DIRECT(done) ? OFF_HDR + (u32)(rank * 1024 + done) * HS : U_DONE(rank, done);
                    __hip_atomic_store((GAS u64*)(hp + o), DIRECT(done) ? ((u64)call << 32) | call : hw_done | (u64)call << ((lane & 7) ? 55 : 32), __ATOMIC_RELAXED, __HIP_MEMORY_SCOPE_SYSTEM);
                }
                if (cur >= 1024) break;
                if (cur >= 0) {
                    const u32 o01 = raw ? U_XD(rank, cur) : OFF_PK + (u32)(rank * 1024 + cur) * 3072u;
                    const u32 o23 = o01 + 2048u;
                    const u32 v2 = raw || lane < (more ? 52 : 48) ? lane * 16u : 0x80000000u;
#pragma unroll
                    for (int p = 0; p < 7; ++p) {
                        __builtin_amdgcn_raw_buffer_store_b128(pk[0], rsP[p], lane * 16u, o01, AUX_SYS);
                        __builtin_amdgcn_raw_buffer_store_b128(pk[1], rsP[p], lane * 16u + 1024u, o01, AUX_SYS);
                        __builtin_amdgcn_raw_buffer_store_b128(pk[2], rsP[p], v2, o23, AUX_SYS);
                    }
                    if (raw) {
#pragma unroll
                        for (int p = 0; p < 7; ++p) __builtin_amdgcn_raw_buffer_store_b128(pk[3], rsP[p], lane * 16u + 1024u, o23, AUX_SYS);
                    }
                }
                HOLD4(pk);
                const int had = done;
                done = cur; hw_done = hw;
                asm volatile("s_waitcnt vmcnt(22)\n\ts_cmp_lt_i32 %5, 0\n\ts_cbranch_scc0 1f\n\ts_waitcnt vmcnt(21)\n\ts_cmp_lt_i32 %4, 0\n\ts_cbranch_scc0 1f\n\ts_waitcnt vmcnt(0)\n1:"
                             : "+v"(d[0]), "+v"(d[1]), "+v"(d[2]), "+v"(d[3]) : "s"(cur), "s"(had) : "memory", "scc");
                raw = true;
                if (!DIRECT(nxt)) {
                    LAS char* L = smem + wid * 4096;
                    u32 ns;
                    for (bool fresh = low > 255u;; fresh = true) {
                        if (fresh) {
                            u16x2 mx = {0, 0};
#pragma unroll
                            for (int i = 0; i < 4; ++i)
#pragma unroll
                                for (int e = 0; e < 4; ++e) mx = __builtin_elementwise_max(mx, __builtin_bit_cast(u16x2, d[i][e] & 0x7FFF7FFFu));
                            const u32 el = (u32)(mx.x > mx.y ? mx.x : mx.y) >> 7;
                            u32 top = wave_max_u32(el);
                            while (top > 7u && __builtin_popcountll(__builtin_amdgcn_ballot_w64(el >= top)) < 16) --top;
                            low = rfl(top < 7u ? 0u : top - 7u);
                        }
                        const u16x2 bias = __builtin_bit_cast(u16x2, low * 0x00800080u);
                        u32 sg = 0, c2[2] = {0, 0}, em = 0, mix = 0;
                        u32x4 hi[2];
#pragma unroll
                        for (int k = 0; k < 8; ++k) {
                            const u32 d0 = d[k >> 1][(k & 1) * 2], d1 = d[k >> 1][(k & 1) * 2 + 1];
                            const u32 x0 = __builtin_bit_cast(u32, (u16x2)(__builtin_bit_cast(u16x2, d0) - bias));
                            const u32 x1 = __builtin_bit_cast(u32, (u16x2)(__builtin_bit_cast(u16x2, d1) - bias));
                            const u32 h = __builtin_amdgcn_perm(x1, x0, 0x07050301u);
                            pk[k >> 2][k & 3] = __builtin_amdgcn_perm(x1, x0, 0x06040200u);
                            hi[k >> 2][k & 3] = h;
                            sg |= (h >> k) & (0x80808080u >> k);
                            c2[k >> 2] |= (h & 0x03030303u) << (2 * (k & 3));
                            em |= (k ? h >> (k - 1) : h << 1) & (0x80808080u >> k);
                            mix |= h ^ (h >> 1);
                        }
                        *(LAS u32*)(L + lane * 4) = sg;
                        *(LAS u32*)(L + 256 + lane * 4) = c2[0];
                        *(LAS u32*)(L + 512 + lane * 4) = c2[1];
                        more = __builtin_amdgcn_ballot_w64((mix & 0x3C3C3C3Cu) != 0u) != 0ull;
                        if (more) {
                            em = 0;
#pragma unroll
                            for (int k = 0; k < 8; ++k) em |= (((hi[k >> 2][k & 3] & 0x7C7C7C7Cu) + 0x7C7C7C7Cu) >> k) & (0x80808080u >> k);
                        }
                        u32 n = 0;
                        u64 m = __builtin_amdgcn_ballot_w64(em != 0u);
                        while (m != 0ull && n <= 64u) {
                            const u32 idx = n + __builtin_amdgcn_mbcnt_hi((u32)(m >> 32), __builtin_amdgcn_mbcnt_lo((u32)m, 0u));
                            const u32 bit = (u32)__builtin_ctz(em | 0x80000000u);
                            if (em != 0u) *(LAS unsigned short*)(L + 832 + idx * 2u) = (unsigned short)((u32)lane * 32u + (7u - (bit & 7u)) * 4u + (bit >> 3));
                            em &= em - 1u;
                            n += (u32)__builtin_popcountll(m);
                            m = __builtin_amdgcn_ballot_w64(em != 0u);
                        }
                        asm volatile("" ::: "memory");
                        ns = n;
                        raw = false;
                        more = more || n > 36u;
                        if (!more) break;
                        const u32 pos = *(LAS unsigned short*)(L + 832 + lane * 2) & 2047u;
                        *(LAS u32x4*)(L + 2048 + lane * 32) = hi[0];
                        *(LAS u32x4*)(L + 2064 + lane * 32) = hi[1];
                        if (lane < 16) *(LAS u32*)(L + 768 + lane * 4) = 0u;
                        asm volatile("" ::: "memory");
                        const u32 hb = *(LAS unsigned char*)(L + 2048 + pos) & 0x7Cu;
                        const u64 ms = __builtin_amdgcn_ballot_w64((u32)lane < n && hb == 0x7Cu);
                        const u32 is = __builtin_amdgcn_mbcnt_hi((u32)(ms >> 32), __builtin_amdgcn_mbcnt_lo((u32)ms, 0u));
                        const bool sh = hb == 0x7Cu && is < 36u;
                        const u32 il = (u32)lane - (is < 36u ? is : 36u);
                        const u32 nsa = (u32)__builtin_popcountll(ms);
                        ns = nsa < 36u ? nsa : 36u;
                        if ((u32)lane < n && (sh || il < 32u)) *(LAS unsigned short*)(L + (sh ? 832u + is * 2u : 768u + il * 2u)) = (unsigned short)(sh ? pos : pos | hb << 9);
                        asm volatile("" ::: "memory");
                        raw = n > 64u || n - ns > 32u;
                        if (!raw || fresh) break;
                    }
                    pk[2] = *(LAS u32x4*)(L + lane * 16);
                    u32 e[5];
#pragma unroll
                    for (int i = 0; i < 5; ++i) e[i] = *(LAS unsigned short*)(L + 824 + (lane & 7) * 10 + i * 2) & 2047u;
                    hw = (lane & 7) ? (u64)(e[0] | e[1] << 11 | e[2] << 22) | (u64)(e[2] >> 10 | e[3] << 1 | e[4] << 12) << 32
                                    : (u64)(low | (raw ? 256u : 0u) | (more ? 512u : 0u) | ns << 10 | e[4] << 16);
                    asm volatile("" ::: "memory");
                }
                if (raw) {
#pragma unroll
                    for (int i = 0; i < 4; ++i) pk[i] = d[i];
                }
                asm volatile("s_waitcnt vmcnt(0)" : "+v"(tk), "+v"(sq) :: "memory");
                const bool first = cur < 0;
                cur = nxt;
                if (first) {
                    call = rfl(sq) + 1u;
                    tbase = (call - 1u) * 1024u - 2u * NPUSH;
                    if (lane == 0) __hip_atomic_store(seq, call, __ATOMIC_RELAXED, __HIP_MEMORY_SCOPE_AGENT);
                    nxt = pi + NPUSH;
                    STAMP(2);
                } else {
                    const u32 u = rfl(tk) - tbase;
                    nxt = (int)(u < 1024u ? u : 1024u);
                }
            }
            STAMP(3);
            STAMP_OUT();
            return;
        }
        const u32 call = rfl(__hip_atomic_load(seq, __ATOMIC_RELAXED, __HIP_MEMORY_SCOPE_AGENT)) + 1u;
        if (lane == 0) __hip_atomic_store(seq, call, __ATOMIC_RELAXED, __HIP_MEMORY_SCOPE_AGENT);
        {
            u32* pend = nullptr;
            LAS char* L = smem + wid * 2048;
            *(LAS u32x4*)(L + lane * 32) = (u32x4){0u, 0u, 0u, 0u};
            *(LAS u32x4*)(L + lane * 32 + 16) = (u32x4){0u, 0u, 0u, 0u};
            const int hg = lane < 36 ? (lane + 4) / 5 : 0;
            const int hs = hg ? 11 * (lane + 4 - 5 * hg) : 16;
            const u32 tsh = hg ? 23u : 0u, tag = hg ? call & 511u : call;
            for (int q = VRAW0 * 7 + wid * NDC + b - NPC; q < VRAW * 7; q += NDEC) {
                const int v = q / 7, src = (rank + 1 + q - v * 7) & 7;
                const u32 xdb = U_XD(src, v);
                const u64* hp = (const u64*)(ws + OFF_HDR + (src * 1024 + v) * HS) + hg;
                u32 hw, h0;
                bool dead = false;
                {
                    const u64 t0 = __builtin_amdgcn_s_memrealtime();
                    for (;;) {
                        const u64 g = __hip_atomic_load(hp, __ATOMIC_RELAXED, __HIP_MEMORY_SCOPE_SYSTEM);
                        hw = (u32)(g >> hs) & 2047u; h0 = rfl((u32)g);
                        u64 ok = __builtin_amdgcn_ballot_w64((u32)(g >> 32) >> tsh == tag);
                        asm volatile("s_waitcnt vmcnt(0)" : "+s"(ok), "+s"(h0), "+v"(hw) :: "memory");
                        if (pend) {
                            if (lane == 0) __hip_atomic_store(pend, call, __ATOMIC_RELAXED, __HIP_MEMORY_SCOPE_AGENT);
                            pend = nullptr;
                        }
                        if (ok == ~0ull) break;
                        if (give_up(tmo, call, t0)) { dead = true; break; }
                        __builtin_amdgcn_s_sleep(DSL);
                    }
                }
                if (dead) break;
                asm volatile("" ::: "memory");
                if (!(h0 & 256u)) {
                    const u32 pk = OFF_PK + (u32)(src * 1024 + v) * 3072u;
                    const u32 sg = __builtin_amdgcn_raw_buffer_load_b32(rsW, 2048u + lane * 4u, pk, AUX_SYS);
                    u32 c2[2];
                    c2[0] = __builtin_amdgcn_raw_buffer_load_b32(rsW, 2304u + lane * 4u, pk, AUX_SYS);
                    c2[1] = __builtin_amdgcn_raw_buffer_load_b32(rsW, 2560u + lane * 4u, pk, AUX_SYS);
                    u32x4 lo[2];
                    lo[0] = __builtin_amdgcn_raw_buffer_load_b128(rsW, lane * 16u, pk, AUX_SYS);
                    lo[1] = __builtin_amdgcn_raw_buffer_load_b128(rsW, lane * 16u + 1024u, pk, AUX_SYS);
                    const bool sh = (u32)lane < ((h0 >> 10) & 63u);
                    if (sh) *(LAS unsigned char*)(L + hw) = 0x7C;
                    u32 ent = 0;
                    if (h0 & 512u) {
                        ent = __builtin_amdgcn_raw_buffer_load_b16(rsW, 2816u + (lane & 31) * 2u, pk, AUX_SYS);
                        if (ent >= 2048u) *(LAS unsigned char*)(L + (ent & 2047u)) = (unsigned char)((ent >> 11) << 2);
                    }
                    asm volatile("" ::: "memory");
                    u32x4 ex[2];
                    ex[0] = *(LAS u32x4*)(L + lane * 32);
                    ex[1] = *(LAS u32x4*)(L + lane * 32 + 16);
                    asm volatile("" ::: "memory");
                    if (sh) *(LAS unsigned char*)(L + hw) = 0;
                    if (ent >= 2048u) *(LAS unsigned char*)(L + (ent & 2047u)) = 0;
                    const u16x2 bias = __builtin_bit_cast(u16x2, (h0 & 255u) * 0x00800080u);
#pragma unroll
                    for (int i = 0; i < 4; ++i) {
                        u32x4 o;
#pragma unroll
                        for (int j = 0; j < 2; ++j) {
                            const int k = 2 * i + j;
                            const u32 h = ((sg << k) & 0x80808080u) | ((c2[k >> 2] >> (2 * (k & 3))) & 0x03030303u) | ex[k >> 2][k & 3];
                            const u32 l = lo[k >> 2][k & 3];
                            o[2 * j] = __builtin_bit_cast(u32, (u16x2)(__builtin_bit_cast(u16x2, __builtin_amdgcn_perm(h, l, 0x05010400u)) + bias));
                            o[2 * j + 1] = __builtin_bit_cast(u32, (u16x2)(__builtin_bit_cast(u16x2, __builtin_amdgcn_perm(h, l, 0x07030602u)) + bias));
                        }
                        __builtin_amdgcn_raw_buffer_store_b128(o, rsW, lane * 16u + i * 1024u, xdb, AUX_AGT);
                        HOLD(o);
                    }
                }
                asm volatile("" ::: "memory");
                pend = (u32*)(ws + U_DONE(src, v));
            }
            WAIT_V(0);
            if (pend && lane == 0) __hip_atomic_store(pend, call, __ATOMIC_RELAXED, __HIP_MEMORY_SCOPE_AGENT);
            STAMP(3);
        }
        STAMP_OUT();
        return;
    }

    const int T = (b & 7) * 26 + (b >> 3) - 6;
    const int ct = (T / 104) * 4 + (T % 104 & 3), rt = (T % 104) >> 2;
    const int rg0 = rt * 10;
    const int nblk = rt == 25 ? 3 : 5;
    const u32 lds0 = (u32)(size_t)smem;
#define DMA(ldsaddr, voff, rs, soff, mods) asm volatile("s_mov_b32 m0, %0\n\ts_nop 0\n\tbuffer_load_dwordx4 %1, %2, %3 offen" mods " lds" \
        :: "s"(ldsaddr), "v"(voff), "s"(rs), "s"(soff) : "memory")
#define DMA_SAME_M0(voff, rs, soff, mods) asm volatile("buffer_load_dwordx4 %0, %1, %2 offen" mods " lds" :: "v"(voff), "s"(rs), "s"(soff) : "memory")

    if (wid == 11) return;
    if (wid >= 9) {
        __builtin_amdgcn_s_setprio(2);
        const int h = wid - 9;
        const __amdgpu_buffer_rsrc_t rsB = mk_rsrc(P.w, (u32)KD * NL * 2u);
        u32 vB[16];
#pragma unroll
        for (int i = 0; i < 16; ++i) {
            const int row = i * 2 + (lane >> 5);
            vB[i] = (u32)((h * 32 + row) * (NL * 2) + ct * 512 + (((lane & 31) ^ ((row & 3) | (row & 8))) << 4));
        }
        const u32 lB = lds0 + 20480 + h * 16384;
#define LOAD_W(kt, so) do { _Pragma("unroll") for (int i = 0; i < 16; ++i) DMA(lB + (so) + i * 1024, vB[i], rsB, (kt) * 262144, ""); } while (0)
        LOAD_W(0, 0);
        LOAD_W(1, STAGE);
        WAIT_V(16);
        BAR();
        int so = 2 * STAGE;
        for (int t = 0; t < 62; ++t) {
            LOAD_W(t + 2, so);
            so = so == 2 * STAGE ? 0 : so + STAGE;
            WAIT_V(16);
            BAR();
        }
        WAIT_V(0);
        BAR();
        STAMP_OUT();
        return;
    }

    if (wid == 8) {
        __builtin_amdgcn_s_setprio(3);
        const u32 call = rfl(__hip_atomic_load(seq, __ATOMIC_RELAXED, __HIP_MEMORY_SCOPE_AGENT)) + 1u;
        if (lane == 0) __hip_atomic_store(seq, call, __ATOMIC_RELAXED, __HIP_MEMORY_SCOPE_AGENT);
        const u32* cp = (const u32*)(ws + OFF_DONE + ((rg0 >> 1) + ((lane & 7) < nblk ? (lane & 7) : 0)) * DS);
        const u32 vA = (u32)lane * 16u;
        const u32 gA = OFF_XD + (u32)rg0 * 2048u;
        int ready = 0;
        auto look = [&]() {
            const int c = ready + ((lane >> 3) & (XLOOK - 1));
            const u32 f = __hip_atomic_load(cp + (c < 63 ? c : 63) * (32 * DS), __ATOMIC_RELAXED, __HIP_MEMORY_SCOPE_SYSTEM);
            const u64 miss = __builtin_amdgcn_ballot_w64(f < call);
            ready += miss == 0ull ? XLOOK : __builtin_ctzll(miss) >> 3;
        };
        auto need = [&](int kt) {
            if (kt < ready) return;
            const u64 t0 = __builtin_amdgcn_s_memrealtime();
            for (;;) {
                look();
                if (kt < ready) break;
                if (give_up(tmo, call, t0)) { ready = 64; break; }
                __builtin_amdgcn_s_sleep(XS);
            }
        };
#define LOAD_X4(g, l) do { \
            DMA((l), vA, rsW, (g), " sc0 sc1"); DMA_SAME_M0(vA, rsW, (g), " offset:1024 sc0 sc1"); DMA_SAME_M0(vA, rsW, (g), " offset:2048 sc0 sc1"); DMA_SAME_M0(vA, rsW, (g), " offset:3072 sc0 sc1"); } while (0)
#define LOAD_X(kt, so) do { _Pragma("unroll") for (int i = 0; i < 5; ++i) \
            LOAD_X4(gA + (kt) * 524288 + (i < nblk ? i : 0) * 4096, lds0 + (so) + (i < nblk ? i : 0) * 4096); } while (0)
        int xi = 0, so = 0;
        for (int t = -1; t < 63; ++t) {
            if (xi == t + 1) {
                need(xi);
                LOAD_X(xi, so);
                ++xi; so = so == 2 * STAGE ? 0 : so + STAGE;
            }
            if (xi < 64 && xi >= ready) look();
            if (xi < 64 && xi < ready) {
                LOAD_X(xi, so);
                ++xi; so = so == 2 * STAGE ? 0 : so + STAGE;
                WAIT_V(20);
            } else {
                WAIT_V(0);
            }
            BAR();
        }
        STAMP(4);
        STAMP_OUT();
        return;
    }

    const int wm = wid >> 2, wn = wid & 3;
    const u32 call = rfl(__hip_atomic_load(seq, __ATOMIC_RELAXED, __HIP_MEMORY_SCOPE_AGENT)) + 1u;
    if (lane == 0) __hip_atomic_store(seq, call, __ATOMIC_RELAXED, __HIP_MEMORY_SCOPE_AGENT);
    if (wid * 208 + T < 1024) {
        const int v = wid * 208 + T;
        const __amdgpu_buffer_rsrc_t rsX = mk_rsrc(P.x, 512u * KD * 2u);
        u32x4 d[4];
#pragma unroll
        for (int i = 0; i < 4; ++i) d[i] = __builtin_amdgcn_raw_buffer_load_b128(rsX, (i & 1) ? XL1 : XL0, U_SRC(v) + (i >> 1) * 16 * KD * 2, 0);
#pragma unroll
        for (int i = 0; i < 4; ++i) __builtin_amdgcn_raw_buffer_store_b128(d[i], rsW, lane * 16u + i * 1024u, U_XD(rank, v), AUX_AGT);
        HOLD4(d);
        WAIT_V(0);
        if (lane == 0) __hip_atomic_store((u32*)(ws + U_DONE(rank, v)), call, __ATOMIC_RELAXED, __HIP_MEMORY_SCOPE_AGENT);
    }

    {
        __builtin_amdgcn_s_sleep(8);
        const __amdgpu_buffer_rsrc_t rsB = mk_rsrc(P.w, (u32)KD * NL * 2u);
        u32 t[3];
#pragma unroll
        for (int i = 0; i < 3; ++i) {
            const u32 r = (u32)(i * 1664 + T * 8 + wid);
            t[i] = __builtin_amdgcn_raw_buffer_load_b32(rsB, lane * 64u, (r < 4096u ? r : 0u) * 4096u, 0);
        }
        asm volatile("" :: "v"(t[0]), "v"(t[1]), "v"(t[2]));
    }
    if (nblk < 5) {
#pragma unroll
        for (int i = 0; i < 3; ++i) *(LAS u32x4*)(smem + i * STAGE + 12288 + wid * 1024 + lane * 16) = (u32x4){0u, 0u, 0u, 0u};
        WAIT_L(0);
    }
    u32 pa[2], pb[4];
#pragma unroll
    for (int h = 0; h < 2; ++h) pa[h] = lds0 + wm * 10240 + (lane & 15) * 128 + (((4 * h + (lane >> 4)) ^ ((lane >> 1) & 7)) << 4);
    {
        const int g = lane >> 4, q = (lane >> 2) & 3, p = lane & 3;
#pragma unroll
        for (int j = 0; j < 4; ++j) {
            const int jj = j ^ (p & 1);
            pb[j] = lds0 + 20480 + (8 * g + q) * 512 + (((wn * 8 + 2 * p + (jj >> 1)) ^ (q | ((g & 1) << 3))) << 4) + (jj & 1) * 8;
        }
    }
    f32x4 acc[5][4];
#pragma unroll
    for (int m = 0; m < 5; ++m)
#pragma unroll
        for (int j = 0; j < 4; ++j) acc[m][j] = (f32x4){0.f, 0.f, 0.f, 0.f};

    s16x4 f0b[8], f1b[8];
    bf16x8 f0a[5], f1a[5];
#define RD(i, fa, fb, h) do { \
        if ((i) < 8) fb[(i) & 7] = __builtin_amdgcn_ds_read_tr16_b64_v4i16((LAS s16x4*)(size_t)(pb[((i) & 7) >> 1] + (h) * 16384 + ((i) & 1) * 2048)); \
        else fa[(i) < 8 ? 0 : (i) - 8] = *(LAS bf16x8*)(size_t)(pa[h] + ((i) - 8) * 2048); } while (0)
#define MM(i, fa, fb) acc[(i) >> 2][(i) & 3] = __builtin_amdgcn_mfma_f32_16x16x32_bf16( \
        __builtin_bit_cast(bf16x8, __builtin_shufflevector(fb[((i) & 3) * 2], fb[((i) & 3) * 2 + 1], 0, 1, 2, 3, 4, 5, 6, 7)), fa[(i) >> 2], acc[(i) >> 2][(i) & 3], 0, 0, 0)
#define VADD(v, s) asm volatile("v_add_u32 %0, %1, %0" : "+v"(v) : "s"(s))
    BAR();
    STAMP(3);
    if (wm == 0) __builtin_amdgcn_s_setprio(1);
    int ph = 0;
    for (int t = 0; t < 64; ++t) {
        SCHED();
#pragma unroll
        for (int i = 0; i < 13; ++i) RD(i, f0a, f0b, 0);
#pragma unroll
        for (int i = 0; i < 13; ++i) RD(i, f1a, f1b, 1);
        SCHED();
        {
            const int dlt = ph == 2 ? -2 * STAGE : STAGE;
            ph = ph == 2 ? 0 : ph + 1;
            VADD(pa[0], dlt); VADD(pa[1], dlt);
#pragma unroll
            for (int j = 0; j < 4; ++j) VADD(pb[j], dlt);
        }
        const int who = t == 63 ? 2 : wm;
        asm volatile("s_cmp_lg_u32 %0, 0\n\ts_cbranch_scc1 1f\n\ts_waitcnt lgkmcnt(0)\n\ts_barrier\n1:" :: "s"(who) : "memory", "scc");
        SCHED();
#pragma unroll
        for (int i = 0; i < 20; ++i) {
            if (i & 1) { MM(mord(i), f1a, f1b); SCHED(); MM(mord(i), f0a, f0b); SCHED(); }
            else { MM(mord(i), f0a, f0b); SCHED(); MM(mord(i), f1a, f1b); SCHED(); }
        }
        SCHED();
        asm volatile("s_cmp_lg_u32 %0, 1\n\ts_cbranch_scc1 1f\n\ts_barrier\n1:" :: "s"(who) : "memory", "scc");
    }
    __builtin_amdgcn_s_setprio(0);
    STAMP(4);

    const u32 tmx = OFF_TMAX + (call & 1u) * (2048u * TS);
    float mx[4] = {0.f, 0.f, 0.f, 0.f};
#pragma unroll
    for (int m = 0; m < 5; ++m)
#pragma unroll
        for (int j = 0; j < 4; ++j) {
            mx[j] = __builtin_fmaxf(__builtin_fmaxf(mx[j], acc[m][j][0]), acc[m][j][1]);
            mx[j] = __builtin_fmaxf(__builtin_fmaxf(mx[j], acc[m][j][2]), acc[m][j][3]);
        }
    LAS u32* misc = (LAS u32*)(smem + LDS_MISC + 512);
    {
        const u32 wmx = wave_max_u32(umax(umax(__float_as_uint(mx[0]), __float_as_uint(mx[1])), umax(__float_as_uint(mx[2]), __float_as_uint(mx[3]))));
        if (lane == 0) misc[wid] = wmx;
    }
    if (wid == 0) {
        GAS u64* tdst = (GAS u64*)(peer_of(lane & 7) + tmx + (rank * 256 + T) * TS);
        asm volatile("s_waitcnt lgkmcnt(0)" : "+v"(tdst) :: "memory");
        BAR();
        const u32x4 a = *(LAS u32x4*)misc, c = *(LAS u32x4*)(misc + 4);
        const u32 m = umax(umax(umax(a[0], a[1]), umax(a[2], a[3])), umax(umax(c[0], c[1]), umax(c[2], c[3])));
        if (lane < NR) __hip_atomic_store(tdst, ((u64)call << 32) | m, __ATOMIC_RELAXED, __HIP_MEMORY_SCOPE_SYSTEM);
    } else {
        WAIT_L(0);
        BAR();
    }
    {
        const bool sw = (lane >> 4) & 1;
#pragma unroll
        for (int m = 0; m < 5; ++m)
#pragma unroll
            for (int j = 0; j < 4; j += 2)
#pragma unroll
                for (int r = 0; r < 4; ++r) {
                    const float u = __builtin_fmaxf(acc[m][j][r], 0.f), v = __builtin_fmaxf(acc[m][j + 1][r], 0.f);
                    acc[m][j][r] = sw ? v : u; acc[m][j + 1][r] = sw ? u : v;
                }
#pragma unroll
        for (int m = 0; m < 5; ++m) asm volatile("" : "+v"(acc[m][0]), "+v"(acc[m][1]), "+v"(acc[m][2]), "+v"(acc[m][3]));
    }
    const __amdgpu_buffer_rsrc_t rsO = mk_rsrc(P.out, 4096u * NL);
    u32 oo = (u32)((rg0 * 16 + wm * 80 + (lane & 15)) * NL + ct * 256 + wn * 64 + (lane >> 4) * 16);
    asm volatile("" : "+v"(oo));
    {
        const u64 t0 = __builtin_amdgcn_s_memrealtime();
        u32 m = 0;
        bool bad;
        u32 o[4];
#pragma unroll
        for (int i = 0; i < 4; ++i) o[i] = i * 64 + lane < 208 ? tmx + (u32)(wid * 256 + i * 64 + lane) * TS : lane == 63 ? OFF_TMO : 0x80000000u;
        for (;;) {
            typedef u32 u32x2 __attribute__((ext_vector_type(2)));
            u32x2 a[4];
#pragma unroll
            for (int i = 0; i < 4; ++i) a[i] = __builtin_amdgcn_raw_buffer_load_b64(rsW, o[i], 0, AUX_SYS);
            bad = (u32)__builtin_amdgcn_readlane((int)a[3][0], 63) == call;
#pragma unroll
            for (int i = 0; i < 4; ++i)
                if (a[i][1] == call) { m = umax(m, a[i][0]); o[i] = 0x80000000u; }
            const u64 miss = __builtin_amdgcn_ballot_w64((o[0] & o[1] & o[2] & (lane == 63 ? ~0u : o[3])) >> 31 == 0u);
            if (miss == 0ull) break;
            if (give_up(tmo, call, t0)) { bad = true; break; }
            if (__builtin_popcountll(miss) > BK_N) __builtin_amdgcn_s_sleep(BK_S);
            __builtin_amdgcn_s_sleep(1);
        }
        m = bad ? 0x7FC00000u : wave_max_u32(m);
        if (lane == 0) misc[16 + wid] = m;
        WAIT_L(0);
    }
    BAR();
    STAMP(5);
    {
        const u32x4 a = *(LAS u32x4*)(misc + 16), c = *(LAS u32x4*)(misc + 20);
        const u32 m = umax(umax(umax(a[0], a[1]), umax(a[2], a[3])), umax(umax(c[0], c[1]), umax(c[2], c[3])));
        float amax = __uint_as_float(m);
        if (rfl(m) < 0x0D800000u) {
            amax *= 0x1p64f;
#pragma unroll
            for (int mt = 0; mt < 5; ++mt)
#pragma unroll
                for (int j = 0; j < 4; ++j) acc[mt][j] *= 0x1p64f;
        }
        const float inv = 448.0f / amax;
        const f32x4 inv4 = {inv, inv, inv, inv};
#pragma unroll
        for (int mt = 0; mt < 5; ++mt) {
            u32x4 o;
#pragma unroll
            for (int j = 0; j < 4; ++j) {
                const f32x4 y = acc[mt][j] * inv4;
                int t = __builtin_amdgcn_cvt_pk_fp8_f32(y[0], y[1], __float_as_int(y[0]), false);
                t = __builtin_amdgcn_cvt_pk_fp8_f32(y[2], y[3], t, true);
                o[j] = (u32)t;
            }
            __builtin_amdgcn_raw_buffer_store_b128(o, rsO, oo + mt * 16 * NL, 0, AUX_AGT);
            HOLD(o);
        }
    }
    STAMP(6);
    STAMP_OUT();
}

}

extern "C" void kernel_launch(int rank, int nranks, ncclComm_t comm,
                              hipStream_t stream,
                              void* const* d_in, const int* in_sizes, int n_in,
                              void* d_out, int out_size,
                              const int* in_dtypes, int out_dtype,
                              void* const* d_out_all,
                              void* const* d_ws_all, size_t ws_bytes) {
    using namespace agk;
    if (nranks != NR || (unsigned)rank >= (unsigned)NR || ws_bytes < WS_USED || n_in < 2 ||
        in_sizes[0] != 512 * KD || in_sizes[1] != KD * NL || out_size != 4096 * NL) {
        fprintf(stderr, "unexpected problem shape\n");
        return;
    }
    static bool attr_done = false;
    if (!attr_done) {
        HIP_CHECK(hipFuncSetAttribute((const void*)ag_gemm, hipFuncAttributeMaxDynamicSharedMemorySize, LDS_BYTES));
        attr_done = true;
    }
    Params P;
    P.x = d_in[0]; P.w = d_in[1]; P.out = d_out;
    P.self = (char*)d_ws_all[rank];
    for (int p = 0; p < NR; ++p) P.peer[p] = (char*)d_ws_all[(rank + 1 + p) % NR];
    P.rank = rank; P.pad = 0;
    hipLaunchKernelGGL(ag_gemm, dim3(GRID), dim3(THREADS), LDS_BYTES, stream, P);
    HIP_CHECK(hipGetLastError());
}
```

```cpp
#include <hip/hip_runtime.h>
#include <rccl/rccl.h>
#include <cstdio>

#define HIP_CHECK(expr) do { hipError_t _e = (expr); if (_e != hipSuccess) { \
    fprintf(stderr, "HIP error %s at %s:%d\n", hipGetErrorString(_e), __FILE__, __LINE__); \
    return; } } while (0)

namespace agk {

typedef unsigned u32;
typedef unsigned long long u64;
typedef u32 u32x4 __attribute__((ext_vector_type(4)));
typedef u32 u32x2 __attribute__((ext_vector_type(2)));
typedef float f32x4 __attribute__((ext_vector_type(4)));
typedef __bf16 bf16x8 __attribute__((ext_vector_type(8)));
typedef short s16x4 __attribute__((ext_vector_type(4)));
typedef unsigned short u16x2 __attribute__((ext_vector_type(2)));
#define LAS __attribute__((address_space(3)))
#define GAS __attribute__((address_space(1)))

constexpr int NR = 8;
constexpr int KD = 4096;
constexpr int NL = 2048;
constexpr int GRID = 256, THREADS = 768;
constexpr int NPC = 22, PWAVES = 2, NPUSH = NPC * PWAVES;
constexpr int NDC = 48 - NPC, NDEC = NDC * 12;
constexpr int VRAW0 = 16, VRAW = 1024 - 32;
constexpr int STAGE = 52 * 1024;
constexpr int LDS_MISC = 3 * STAGE;
constexpr int LDS_BYTES = LDS_MISC + 1024;

constexpr u32 OFF_SEQ = 0;
constexpr u32 OFF_TMO = 16384;
constexpr u32 OFF_TKT = 32768;
constexpr u32 OFF_TMAX = 512u << 10;
constexpr u32 OFF_DONE = 1u << 20;
constexpr u32 OFF_HDR = 2u << 20;
constexpr u32 OFF_XD = 4u << 20;
constexpr u32 OFF_PK = 36u << 20;
constexpr u32 WS_USED = 60u << 20;

constexpr u64 DEADLINE_TICKS = 40000000ull;
constexpr int AUX_SYS = 17, AUX_AGT = 16;

struct Params {
    const void* x;
    const void* w;
    void* out;
    char* self;
    char* peer[NR];
    int rank;
    int pad;
};

#define WAIT_V(n) asm volatile("s_waitcnt vmcnt(" #n ")" ::: "memory")
#define WAIT_L(n) asm volatile("s_waitcnt lgkmcnt(" #n ")" ::: "memory")
#define BAR() do { asm volatile("" ::: "memory"); __builtin_amdgcn_s_barrier(); asm volatile("" ::: "memory"); } while (0)
#define SCHED() __builtin_amdgcn_sched_barrier(0)

__device__ __forceinline__ u32 rfl(u32 v) { return __builtin_amdgcn_readfirstlane(v); }
__device__ __forceinline__ __amdgpu_buffer_rsrc_t mk_rsrc(const void* p, u32 bytes) {
    return __builtin_amdgcn_make_buffer_rsrc((void*)p, 0, (int)bytes, 0x00020000);
}
#define HOLD(x) asm volatile("s_nop 1" :: "v"(x) : "memory")
#define HOLD4(x) asm volatile("s_nop 1" :: "v"(x[0]), "v"(x[1]), "v"(x[2]), "v"(x[3]) : "memory")
__device__ __forceinline__ u32 umax(u32 a, u32 b) { return a > b ? a : b; }
__device__ __forceinline__ u32 wave_max_u32(u32 v) {
    v = umax(v, (u32)__builtin_amdgcn_update_dpp(0, (int)v, 0x111, 0xf, 0xf, true));
    v = umax(v, (u32)__builtin_amdgcn_update_dpp(0, (int)v, 0x112, 0xf, 0xf, true));
    v = umax(v, (u32)__builtin_amdgcn_update_dpp(0, (int)v, 0x114, 0xf, 0xf, true));
    v = umax(v, (u32)__builtin_amdgcn_update_dpp(0, (int)v, 0x118, 0xf, 0xf, true));
    return umax(umax((u32)__builtin_amdgcn_readlane((int)v, 15), (u32)__builtin_amdgcn_readlane((int)v, 31)),
                umax((u32)__builtin_amdgcn_readlane((int)v, 47), (u32)__builtin_amdgcn_readlane((int)v, 63)));
}
__device__ __forceinline__ bool give_up(u32* tmo, u32 call, u64 t0) {
    const u64 dt = __builtin_amdgcn_s_memrealtime() - t0;
    if (dt < 20000ull) return false;
    const u32 t = rfl(__hip_atomic_load(tmo, __ATOMIC_RELAXED, __HIP_MEMORY_SCOPE_AGENT));
    if (t == call) return true;
    if (dt > DEADLINE_TICKS) {
        __hip_atomic_store(tmo, call, __ATOMIC_RELAXED, __HIP_MEMORY_SCOPE_AGENT);
        return true;
    }
    return false;
}

#define XL0 ((u32)((lane >> 3) * (KD * 2) + (((lane & 7) ^ (lane >> 4)) << 4)))
#define XL1 ((u32)((8 + (lane >> 3)) * (KD * 2) + (((lane & 7) ^ (4 + (lane >> 4))) << 4)))
#define DIRECT(v) ((v) < VRAW0 || (v) >= VRAW)
#define U_SRC(v) ((((v) & 15) * 32 * KD + ((v) >> 4) * 64) * 2)
#define U_XD(src, v) (OFF_XD + (u32)((v) >> 4) * 524288u + (u32)((src) * 16 + ((v) & 15)) * 4096u)
#define U_DONE(src, v) (OFF_DONE + (u32)(((v) >> 4) * 128 + (src) * 16 + ((v) & 15)) * 64u)

__device__ constexpr int mord(int i) { return (((i / 5) & 1) ? 4 - i % 5 : i % 5) * 4 + i / 5; }

__global__ __launch_bounds__(THREADS, 3) void ag_gemm(Params P) {
    extern __shared__ __attribute__((aligned(16))) char smem_[];
    LAS char* smem = (LAS char*)smem_;
    const int lane = threadIdx.x & 63;
    const int wid = (int)rfl(threadIdx.x >> 6);
    const int b = blockIdx.x;
    const int rank = P.rank & 7;
    char* ws = P.self;
    const void* px = P.x;
    asm volatile("" : "+s"(px));
    u32* tmo = (u32*)(ws + OFF_TMO);
    const __amdgpu_buffer_rsrc_t rsW = mk_rsrc(ws, WS_USED);
    u32* seq = (u32*)(ws + OFF_SEQ) + b * 16 + wid;

    auto peer_of = [&](int i) {
        u64 a = (u64)P.peer[7];
#pragma unroll
        for (int p = 0; p < 7; ++p) { u64 t = (u64)P.peer[p]; asm("" : "+s"(t)); a = i == p ? t : a; }
        return (GAS char*)a;
    };

    if (__builtin_expect(b < 48, 1)) {
        if (__builtin_expect(b < NPC, 1)) {
            if (__builtin_expect(wid >= PWAVES, 0)) return;
            const int pi = wid * NPC + b;
            const __amdgpu_buffer_rsrc_t rsX = mk_rsrc(px, 512u * KD * 2u);
            const u32 xl0 = XL0, xl1 = XL1;
            u32x4 d[4];
            u32 sq;
            asm volatile("buffer_load_dwordx4 %0, %5, %7, %8 offen\n\tbuffer_load_dwordx4 %1, %6, %7, %8 offen\n\t"
                         "buffer_load_dwordx4 %2, %5, %7, %9 offen\n\tbuffer_load_dwordx4 %3, %6, %7, %9 offen\n\tglobal_load_dword %4, %10, %11 sc1"
                         : "=&v"(d[0]), "=&v"(d[1]), "=&v"(d[2]), "=&v"(d[3]), "=&v"(sq)
                         : "v"(xl0), "v"(xl1), "s"(rsX), "s"(U_SRC(pi)), "s"(U_SRC(pi) + 16 * KD * 2), "v"(0u), "s"(seq) : "memory");
            SCHED();
            GAS char* hp = peer_of(lane >> 3) + (lane & 7) * 8;
            __amdgpu_buffer_rsrc_t rsP[7];
#pragma unroll
            for (int p = 0; p < 7; ++p) rsP[p] = mk_rsrc(P.peer[p], WS_USED);
            u32* tkt = (u32*)(ws + OFF_TKT);
            u32 call = 0, tbase = 0;
            int done = -1, cur = -1, nxt = pi;
            u32x4 pk[4] = {};
            u64 hw = 0, hw_done = 0;
            bool more = false;
            u32 low = 256u;
            bool raw = false;
            for (;;) {
                asm volatile("" ::: "memory");
                u32 tk;
                asm volatile("s_cmp_lt_u32 %4, 1024\n\ts_cbranch_scc0 1f\n\ts_mov_b64 exec, 1\n\tglobal_atomic_add %0, %1, %2, %3 sc0\n\ts_mov_b64 exec, -1\n1:"
                             : "=&v"(tk) : "v"(0u), "v"(1u), "s"(tkt), "s"(cur) : "memory", "scc");
                asm volatile("s_cmp_lt_u32 %8, 1024\n\ts_cbranch_scc0 1f\n\t"
                             "buffer_load_dwordx4 %0, %4, %6, %7 offen\n\tbuffer_load_dwordx4 %1, %5, %6, %7 offen\n\t"
                             "buffer_load_dwordx4 %2, %4, %6, %9 offen\n\tbuffer_load_dwordx4 %3, %5, %6, %9 offen\n1:"
                             : "+v"(d[0]), "+v"(d[1]), "+v"(d[2]), "+v"(d[3])
                             : "v"(xl0), "v"(xl1), "s"(rsX), "s"(U_SRC(nxt)), "s"(cur < 0 ? 1024 : nxt), "s"(U_SRC(nxt) + 16 * KD * 2) : "memory", "scc");
                if (done >= 0 && lane < 56 && (!DIRECT(done) || (lane & 7) == 0)) {
                    const u32 o = !DIRECT(done) ? OFF_HDR + (u32)(rank * 1024 + done) * 64u : U_DONE(rank, done);
                    __hip_atomic_store((GAS u64*)(hp + o), DIRECT(done) ? ((u64)call << 32) | call : hw_done | (u64)call << ((lane & 7) ? 55 : 32), __ATOMIC_RELAXED, __HIP_MEMORY_SCOPE_SYSTEM);
                }
                if (cur >= 1024) break;
                if (cur >= 0) {
                    const u32 o01 = raw ? U_XD(rank, cur) : OFF_PK + (u32)(rank * 1024 + cur) * 3072u;
                    const u32 o23 = o01 + 2048u;
                    const u32 v2 = raw || lane < (more ? 52 : 48) ? lane * 16u : 0x80000000u;
#pragma unroll
                    for (int p = 0; p < 7; ++p) {
                        __builtin_amdgcn_raw_buffer_store_b128(pk[0], rsP[p], lane * 16u, o01, AUX_SYS);
                        __builtin_amdgcn_raw_buffer_store_b128(pk[1], rsP[p], lane * 16u + 1024u, o01, AUX_SYS);
                        __builtin_amdgcn_raw_buffer_store_b128(pk[2], rsP[p], v2, o23, AUX_SYS);
                    }
                    if (raw) {
#pragma unroll
                        for (int p = 0; p < 7; ++p) __builtin_amdgcn_raw_buffer_store_b128(pk[3], rsP[p], lane * 16u + 1024u, o23, AUX_SYS);
                    }
                }
                HOLD4(pk);
                const int had = done;
                done = cur; hw_done = hw;
                asm volatile("s_waitcnt vmcnt(22)\n\ts_cmp_lt_i32 %5, 0\n\ts_cbranch_scc0 1f\n\ts_waitcnt vmcnt(21)\n\ts_cmp_lt_i32 %4, 0\n\ts_cbranch_scc0 1f\n\ts_waitcnt vmcnt(0)\n1:"
                             : "+v"(d[0]), "+v"(d[1]), "+v"(d[2]), "+v"(d[3]) : "s"(cur), "s"(had) : "memory", "scc");
                raw = true;
                if (!DIRECT(nxt)) {
                    LAS char* L = smem + wid * 4096;
                    u32 ns;
                    for (bool fresh = low > 255u;; fresh = true) {
                        if (fresh) {
                            u16x2 mx = {0, 0};
#pragma unroll
                            for (int i = 0; i < 4; ++i)
#pragma unroll
                                for (int e = 0; e < 4; ++e) mx = __builtin_elementwise_max(mx, __builtin_bit_cast(u16x2, d[i][e] & 0x7FFF7FFFu));
                            const u32 el = (u32)(mx.x > mx.y ? mx.x : mx.y) >> 7;
                            u32 top = wave_max_u32(el);
                            while (top > 7u && __builtin_popcountll(__builtin_amdgcn_ballot_w64(el >= top)) < 16) --top;
                            low = rfl(top < 7u ? 0u : top - 7u);
                        }
                        const u16x2 bias = __builtin_bit_cast(u16x2, low * 0x00800080u);
                        u32 sg = 0, c2[2] = {0, 0}, em = 0, mix = 0;
                        u32x4 hi[2];
#pragma unroll
                        for (int k = 0; k < 8; ++k) {
                            const u32 d0 = d[k >> 1][(k & 1) * 2], d1 = d[k >> 1][(k & 1) * 2 + 1];
                            const u32 x0 = __builtin_bit_cast(u32, (u16x2)(__builtin_bit_cast(u16x2, d0) - bias));
                            const u32 x1 = __builtin_bit_cast(u32, (u16x2)(__builtin_bit_cast(u16x2, d1) - bias));
                            const u32 h = __builtin_amdgcn_perm(x1, x0, 0x07050301u);
                            pk[k >> 2][k & 3] = __builtin_amdgcn_perm(x1, x0, 0x06040200u);
                            hi[k >> 2][k & 3] = h;
                            sg |= (h >> k) & (0x80808080u >> k);
                            c2[k >> 2] |= (h & 0x03030303u) << (2 * (k & 3));
                            em |= (k ? h >> (k - 1) : h << 1) & (0x80808080u >> k);
                            mix |= h ^ (h >> 1);
                        }
                        *(LAS u32*)(L + lane * 4) = sg;
                        *(LAS u32*)(L + 256 + lane * 4) = c2[0];
                        *(LAS u32*)(L + 512 + lane * 4) = c2[1];
                        more = __builtin_amdgcn_ballot_w64((mix & 0x3C3C3C3Cu) != 0u) != 0ull;
                        if (more) {
                            em = 0;
#pragma unroll
                            for (int k = 0; k < 8; ++k) em |= (((hi[k >> 2][k & 3] & 0x7C7C7C7Cu) + 0x7C7C7C7Cu) >> k) & (0x80808080u >> k);
                        }
                        u32 n = 0;
                        u64 m = __builtin_amdgcn_ballot_w64(em != 0u);
                        while (m != 0ull && n <= 64u) {
                            const u32 idx = n + __builtin_amdgcn_mbcnt_hi((u32)(m >> 32), __builtin_amdgcn_mbcnt_lo((u32)m, 0u));
                            const u32 bit = (u32)__builtin_ctz(em | 0x80000000u);
                            if (em != 0u) *(LAS unsigned short*)(L + 832 + idx * 2u) = (unsigned short)((u32)lane * 32u + (7u - (bit & 7u)) * 4u + (bit >> 3));
                            em &= em - 1u;
                            n += (u32)__builtin_popcountll(m);
                            m = __builtin_amdgcn_ballot_w64(em != 0u);
                        }
                        asm volatile("" ::: "memory");
                        ns = n;
                        raw = false;
                        more = more || n > 36u;
                        if (!more) break;
                        const u32 pos = *(LAS unsigned short*)(L + 832 + lane * 2) & 2047u;
                        *(LAS u32x4*)(L + 2048 + lane * 32) = hi[0];
                        *(LAS u32x4*)(L + 2064 + lane * 32) = hi[1];
                        if (lane < 16) *(LAS u32*)(L + 768 + lane * 4) = 0u;
                        asm volatile("" ::: "memory");
                        const u32 hb = *(LAS unsigned char*)(L + 2048 + pos) & 0x7Cu;
                        const u64 ms = __builtin_amdgcn_ballot_w64((u32)lane < n && hb == 0x7Cu);
                        const u32 is = __builtin_amdgcn_mbcnt_hi((u32)(ms >> 32), __builtin_amdgcn_mbcnt_lo((u32)ms, 0u));
                        const bool sh = hb == 0x7Cu && is < 36u;
                        const u32 il = (u32)lane - (is < 36u ? is : 36u);
                        const u32 nsa = (u32)__builtin_popcountll(ms);
                        ns = nsa < 36u ? nsa : 36u;
                        if ((u32)lane < n && (sh || il < 32u)) *(LAS unsigned short*)(L + (sh ? 832u + is * 2u : 768u + il * 2u)) = (unsigned short)(sh ? pos : pos | hb << 9);
                        asm volatile("" ::: "memory");
                        raw = n > 64u || n - ns > 32u;
                        if (!raw || fresh) break;
                    }
                    pk[2] = *(LAS u32x4*)(L + lane * 16);
                    u32 e[5];
#pragma unroll
                    for (int i = 0; i < 5; ++i) e[i] = *(LAS unsigned short*)(L + 824 + (lane & 7) * 10 + i * 2) & 2047u;
                    hw = (lane & 7) ? (u64)(e[0] | e[1] << 11 | e[2] << 22) | (u64)(e[2] >> 10 | e[3] << 1 | e[4] << 12) << 32
                                    : (u64)(low | (raw ? 256u : 0u) | (more ? 512u : 0u) | ns << 10 | e[4] << 16);
                    asm volatile("" ::: "memory");
                }
                if (raw) {
#pragma unroll
                    for (int i = 0; i < 4; ++i) pk[i] = d[i];
                }
                asm volatile("s_waitcnt vmcnt(0)" : "+v"(tk), "+v"(sq) :: "memory");
                const bool first = cur < 0;
                cur = nxt;
                if (first) {
                    call = rfl(sq) + 1u;
                    tbase = (call - 1u) * 1024u - 2u * NPUSH;
                    if (lane == 0) __hip_atomic_store(seq, call, __ATOMIC_RELAXED, __HIP_MEMORY_SCOPE_AGENT);
                    nxt = pi + NPUSH;
                } else {
                    const u32 u = rfl(tk) - tbase;
                    nxt = (int)(u < 1024u ? u : 1024u);
                }
            }
            return;
        }
        const u32 call = rfl(__hip_atomic_load(seq, __ATOMIC_RELAXED, __HIP_MEMORY_SCOPE_AGENT)) + 1u;
        if (lane == 0) __hip_atomic_store(seq, call, __ATOMIC_RELAXED, __HIP_MEMORY_SCOPE_AGENT);
        {
            u32* pend = nullptr;
            LAS char* L = smem + wid * 2048;
            *(LAS u32x4*)(L + lane * 32) = (u32x4){0u, 0u, 0u, 0u};
            *(LAS u32x4*)(L + lane * 32 + 16) = (u32x4){0u, 0u, 0u, 0u};
            const int hg = lane < 36 ? (lane + 4) / 5 : 0;
            const int hs = hg ? 11 * (lane + 4 - 5 * hg) : 16;
            const u32 tsh = hg ? 23u : 0u, tag = hg ? call & 511u : call;
            for (int q = VRAW0 * 7 + wid * NDC + b - NPC; q < VRAW * 7; q += NDEC) {
                const int v = q / 7, src = (rank + 1 + q - v * 7) & 7;
                const u32 xdb = U_XD(src, v);
                const u64* hp = (const u64*)(ws + OFF_HDR + (src * 1024 + v) * 64u) + hg;
                u32 hw, h0;
                bool dead = false;
                {
                    const u64 t0 = __builtin_amdgcn_s_memrealtime();
                    for (;;) {
                        const u64 g = __hip_atomic_load(hp, __ATOMIC_RELAXED, __HIP_MEMORY_SCOPE_SYSTEM);
                        hw = (u32)(g >> hs) & 2047u; h0 = rfl((u32)g);
                        u64 ok = __builtin_amdgcn_ballot_w64((u32)(g >> 32) >> tsh == tag);
                        asm volatile("s_waitcnt vmcnt(0)" : "+s"(ok), "+s"(h0), "+v"(hw) :: "memory");
                        if (pend) {
                            if (lane == 0) __hip_atomic_store(pend, call, __ATOMIC_RELAXED, __HIP_MEMORY_SCOPE_AGENT);
                            pend = nullptr;
                        }
                        if (ok == ~0ull) break;
                        if (give_up(tmo, call, t0)) { dead = true; break; }
                        __builtin_amdgcn_s_sleep(1);
                    }
                }
                if (dead) break;
                asm volatile("" ::: "memory");
                if (!(h0 & 256u)) {
                    const u32 pk = OFF_PK + (u32)(src * 1024 + v) * 3072u;
                    const u32 sg = __builtin_amdgcn_raw_buffer_load_b32(rsW, 2048u + lane * 4u, pk, AUX_SYS);
                    u32 c2[2];
                    c2[0] = __builtin_amdgcn_raw_buffer_load_b32(rsW, 2304u + lane * 4u, pk, AUX_SYS);
                    c2[1] = __builtin_amdgcn_raw_buffer_load_b32(rsW, 2560u + lane * 4u, pk, AUX_SYS);
                    u32x4 lo[2];
                    lo[0] = __builtin_amdgcn_raw_buffer_load_b128(rsW, lane * 16u, pk, AUX_SYS);
                    lo[1] = __builtin_amdgcn_raw_buffer_load_b128(rsW, lane * 16u + 1024u, pk, AUX_SYS);
                    const bool sh = (u32)lane < ((h0 >> 10) & 63u);
                    if (sh) *(LAS unsigned char*)(L + hw) = 0x7C;
                    u32 ent = 0;
                    if (h0 & 512u) {
                        ent = __builtin_amdgcn_raw_buffer_load_b16(rsW, 2816u + (lane & 31) * 2u, pk, AUX_SYS);
                        if (ent >= 2048u) *(LAS unsigned char*)(L + (ent & 2047u)) = (unsigned char)((ent >> 11) << 2);
                    }
                    asm volatile("" ::: "memory");
                    u32x4 ex[2];
                    ex[0] = *(LAS u32x4*)(L + lane * 32);
                    ex[1] = *(LAS u32x4*)(L + lane * 32 + 16);
                    asm volatile("" ::: "memory");
                    if (sh) *(LAS unsigned char*)(L + hw) = 0;
                    if (ent >= 2048u) *(LAS unsigned char*)(L + (ent & 2047u)) = 0;
                    const u16x2 bias = __builtin_bit_cast(u16x2, (h0 & 255u) * 0x00800080u);
#pragma unroll
                    for (int i = 0; i < 4; ++i) {
                        u32x4 o;
#pragma unroll
                        for (int j = 0; j < 2; ++j) {
                            const int k = 2 * i + j;
                            const u32 h = ((sg << k) & 0x80808080u) | ((c2[k >> 2] >> (2 * (k & 3))) & 0x03030303u) | ex[k >> 2][k & 3];
                            const u32 l = lo[k >> 2][k & 3];
                            o[2 * j] = __builtin_bit_cast(u32, (u16x2)(__builtin_bit_cast(u16x2, __builtin_amdgcn_perm(h, l, 0x05010400u)) + bias));
                            o[2 * j + 1] = __builtin_bit_cast(u32, (u16x2)(__builtin_bit_cast(u16x2, __builtin_amdgcn_perm(h, l, 0x07030602u)) + bias));
                        }
                        __builtin_amdgcn_raw_buffer_store_b128(o, rsW, lane * 16u + i * 1024u, xdb, AUX_AGT);
                        HOLD(o);
                    }
                }
                asm volatile("" ::: "memory");
                pend = (u32*)(ws + U_DONE(src, v));
            }
            WAIT_V(0);
            if (pend && lane == 0) __hip_atomic_store(pend, call, __ATOMIC_RELAXED, __HIP_MEMORY_SCOPE_AGENT);
        }
        return;
    }

    const int T = (b & 7) * 26 + (b >> 3) - 6;
    const int ct = (T / 104) * 4 + (T % 104 & 3), rt = (T % 104) >> 2;
    const int rg0 = rt * 10;
    const int nblk = rt == 25 ? 3 : 5;
    const u32 lds0 = (u32)(size_t)smem;
#define DMA(ldsaddr, voff, rs, soff, mods) asm volatile("s_mov_b32 m0, %0\n\ts_nop 0\n\tbuffer_load_dwordx4 %1, %2, %3 offen" mods " lds" \
        :: "s"(ldsaddr), "v"(voff), "s"(rs), "s"(soff) : "memory")
#define DMA_SAME_M0(voff, rs, soff, mods) asm volatile("buffer_load_dwordx4 %0, %1, %2 offen" mods " lds" :: "v"(voff), "s"(rs), "s"(soff) : "memory")

    if (wid == 11) return;
    if (wid >= 9) {
        __builtin_amdgcn_s_setprio(2);
        const int h = wid - 9;
        const __amdgpu_buffer_rsrc_t rsB = mk_rsrc(P.w, (u32)KD * NL * 2u);
        u32 vB[16];
#pragma unroll
        for (int i = 0; i < 16; ++i) {
            const int row = i * 2 + (lane >> 5);
            vB[i] = (u32)((h * 32 + row) * (NL * 2) + ct * 512 + (((lane & 31) ^ ((row & 3) | (row & 8))) << 4));
        }
        const u32 lB = lds0 + 20480 + h * 16384;
#define LOAD_W(kt, so) do { _Pragma("unroll") for (int i = 0; i < 16; ++i) DMA(lB + (so) + i * 1024, vB[i], rsB, (kt) * 262144, ""); } while (0)
        LOAD_W(0, 0);
        LOAD_W(1, STAGE);
        WAIT_V(16);
        BAR();
        int so = 2 * STAGE;
        for (int t = 0; t < 62; ++t) {
            LOAD_W(t + 2, so);
            so = so == 2 * STAGE ? 0 : so + STAGE;
            WAIT_V(16);
            BAR();
        }
        WAIT_V(0);
        BAR();
        return;
    }

    if (wid == 8) {
        __builtin_amdgcn_s_setprio(3);
        const u32 call = rfl(__hip_atomic_load(seq, __ATOMIC_RELAXED, __HIP_MEMORY_SCOPE_AGENT)) + 1u;
        if (lane == 0) __hip_atomic_store(seq, call, __ATOMIC_RELAXED, __HIP_MEMORY_SCOPE_AGENT);
        const u32* cp = (const u32*)(ws + OFF_DONE + ((rg0 >> 1) + ((lane & 7) < nblk ? (lane & 7) : 0)) * 64u);
        const u32 vA = (u32)lane * 16u;
        const u32 gA = OFF_XD + (u32)rg0 * 2048u;
        int ready = 0;
        auto look = [&]() {
            const u32 f = __hip_atomic_load(cp + ready * 2048, __ATOMIC_RELAXED, __HIP_MEMORY_SCOPE_SYSTEM);
            if (__builtin_amdgcn_ballot_w64(f < call) == 0ull) ++ready;
        };
        auto need = [&](int kt) {
            if (kt < ready) return;
            const u64 t0 = __builtin_amdgcn_s_memrealtime();
            for (;;) {
                look();
                if (kt < ready) break;
                if (give_up(tmo, call, t0)) { ready = 64; break; }
                __builtin_amdgcn_s_sleep(1);
            }
        };
#define LOAD_X4(g, l) do { \
            DMA((l), vA, rsW, (g), " sc0 sc1"); DMA_SAME_M0(vA, rsW, (g), " offset:1024 sc0 sc1"); DMA_SAME_M0(vA, rsW, (g), " offset:2048 sc0 sc1"); DMA_SAME_M0(vA, rsW, (g), " offset:3072 sc0 sc1"); } while (0)
#define LOAD_X(kt, so) do { _Pragma("unroll") for (int i = 0; i < 5; ++i) \
            LOAD_X4(gA + (kt) * 524288 + (i < nblk ? i : 0) * 4096, lds0 + (so) + (i < nblk ? i : 0) * 4096); } while (0)
        int xi = 0, so = 0;
        for (int t = -1; t < 63; ++t) {
            if (xi == t + 1) {
                need(xi);
                LOAD_X(xi, so);
                ++xi; so = so == 2 * STAGE ? 0 : so + STAGE;
            }
            if (xi < 64 && xi >= ready) look();
            if (xi < 64 && xi < ready) {
                LOAD_X(xi, so);
                ++xi; so = so == 2 * STAGE ? 0 : so + STAGE;
                WAIT_V(20);
            } else {
                WAIT_V(0);
            }
            BAR();
        }
        return;
    }

    const int wm = wid >> 2, wn = wid & 3;
    const u32 call = rfl(__hip_atomic_load(seq, __ATOMIC_RELAXED, __HIP_MEMORY_SCOPE_AGENT)) + 1u;
    if (lane == 0) __hip_atomic_store(seq, call, __ATOMIC_RELAXED, __HIP_MEMORY_SCOPE_AGENT);
    if (wid * 208 + T < 1024) {
        const int v = wid * 208 + T;
        const __amdgpu_buffer_rsrc_t rsX = mk_rsrc(px, 512u * KD * 2u);
        u32x4 d[4];
#pragma unroll
        for (int i = 0; i < 4; ++i) d[i] = __builtin_amdgcn_raw_buffer_load_b128(rsX, (i & 1) ? XL1 : XL0, U_SRC(v) + (i >> 1) * 16 * KD * 2, 0);
#pragma unroll
        for (int i = 0; i < 4; ++i) __builtin_amdgcn_raw_buffer_store_b128(d[i], rsW, lane * 16u + i * 1024u, U_XD(rank, v), AUX_AGT);
        HOLD4(d);
        WAIT_V(0);
        if (lane == 0) __hip_atomic_store((u32*)(ws + U_DONE(rank, v)), call, __ATOMIC_RELAXED, __HIP_MEMORY_SCOPE_AGENT);
    }

    {
        __builtin_amdgcn_s_sleep(8);
        const __amdgpu_buffer_rsrc_t rsB = mk_rsrc(P.w, (u32)KD * NL * 2u);
        u32 t[3];
#pragma unroll
        for (int i = 0; i < 3; ++i) {
            const u32 r = (u32)(i * 1664 + T * 8 + wid);
            t[i] = __builtin_amdgcn_raw_buffer_load_b32(rsB, lane * 64u, (r < 4096u ? r : 0u) * 4096u, 0);
        }
        asm volatile("" :: "v"(t[0]), "v"(t[1]), "v"(t[2]));
    }
    if (nblk < 5) {
#pragma unroll
        for (int i = 0; i < 3; ++i) *(LAS u32x4*)(smem + i * STAGE + 12288 + wid * 1024 + lane * 16) = (u32x4){0u, 0u, 0u, 0u};
        WAIT_L(0);
    }
    u32 pa[2], pb[4];
#pragma unroll
    for (int h = 0; h < 2; ++h) pa[h] = lds0 + wm * 10240 + (lane & 15) * 128 + (((4 * h + (lane >> 4)) ^ ((lane >> 1) & 7)) << 4);
    {
        const int g = lane >> 4, q = (lane >> 2) & 3, p = lane & 3;
#pragma unroll
        for (int j = 0; j < 4; ++j) {
            const int jj = j ^ (p & 1);
            pb[j] = lds0 + 20480 + (8 * g + q) * 512 + (((wn * 8 + 2 * p + (jj >> 1)) ^ (q | ((g & 1) << 3))) << 4) + (jj & 1) * 8;
        }
    }
    f32x4 acc[5][4];
#pragma unroll
    for (int m = 0; m < 5; ++m)
#pragma unroll
        for (int j = 0; j < 4; ++j) acc[m][j] = (f32x4){0.f, 0.f, 0.f, 0.f};

    s16x4 f0b[8], f1b[8];
    bf16x8 f0a[5], f1a[5];
#define RD(i, fa, fb, h) do { \
        if ((i) < 8) fb[(i) & 7] = __builtin_amdgcn_ds_read_tr16_b64_v4i16((LAS s16x4*)(size_t)(pb[((i) & 7) >> 1] + (h) * 16384 + ((i) & 1) * 2048)); \
        else fa[(i) < 8 ? 0 : (i) - 8] = *(LAS bf16x8*)(size_t)(pa[h] + ((i) - 8) * 2048); } while (0)
#define MM(i, fa, fb) acc[(i) >> 2][(i) & 3] = __builtin_amdgcn_mfma_f32_16x16x32_bf16( \
        __builtin_bit_cast(bf16x8, __builtin_shufflevector(fb[((i) & 3) * 2], fb[((i) & 3) * 2 + 1], 0, 1, 2, 3, 4, 5, 6, 7)), fa[(i) >> 2], acc[(i) >> 2][(i) & 3], 0, 0, 0)
#define VADD(v, s) asm volatile("v_add_u32 %0, %1, %0" : "+v"(v) : "s"(s))
    BAR();
    if (wm == 0) __builtin_amdgcn_s_setprio(1);
    int ph = 0;
    for (int t = 0; t < 64; ++t) {
        SCHED();
#pragma unroll
        for (int i = 0; i < 13; ++i) RD(i, f0a, f0b, 0);
#pragma unroll
        for (int i = 0; i < 13; ++i) RD(i, f1a, f1b, 1);
        SCHED();
        {
            const int dlt = ph == 2 ? -2 * STAGE : STAGE;
            ph = ph == 2 ? 0 : ph + 1;
            VADD(pa[0], dlt); VADD(pa[1], dlt);
#pragma unroll
            for (int j = 0; j < 4; ++j) VADD(pb[j], dlt);
        }
        const int who = t == 63 ? 2 : wm;
        asm volatile("s_cmp_lg_u32 %0, 0\n\ts_cbranch_scc1 1f\n\ts_waitcnt lgkmcnt(0)\n\ts_barrier\n1:" :: "s"(who) : "memory", "scc");
        SCHED();
#pragma unroll
        for (int i = 0; i < 20; ++i) {
            if (i & 1) { MM(mord(i), f1a, f1b); SCHED(); MM(mord(i), f0a, f0b); SCHED(); }
            else { MM(mord(i), f0a, f0b); SCHED(); MM(mord(i), f1a, f1b); SCHED(); }
        }
        SCHED();
        asm volatile("s_cmp_lg_u32 %0, 1\n\ts_cbranch_scc1 1f\n\ts_barrier\n1:" :: "s"(who) : "memory", "scc");
    }
    __builtin_amdgcn_s_setprio(0);

    const u32 tmx = OFF_TMAX + (call & 1u) * 65536u;
    float mx[4] = {0.f, 0.f, 0.f, 0.f};
#pragma unroll
    for (int m = 0; m < 5; ++m)
#pragma unroll
        for (int j = 0; j < 4; ++j) {
            mx[j] = __builtin_fmaxf(__builtin_fmaxf(mx[j], acc[m][j][0]), acc[m][j][1]);
            mx[j] = __builtin_fmaxf(__builtin_fmaxf(mx[j], acc[m][j][2]), acc[m][j][3]);
        }
    LAS u32* misc = (LAS u32*)(smem + LDS_MISC + 512);
    {
        const u32 wmx = wave_max_u32(umax(umax(__float_as_uint(mx[0]), __float_as_uint(mx[1])), umax(__float_as_uint(mx[2]), __float_as_uint(mx[3]))));
        if (lane == 0) misc[wid] = wmx;
    }
    if (wid == 0) {
        GAS u64* tdst = (GAS u64*)(peer_of(lane & 7) + tmx + (rank * 256 + T) * 32u);
        asm volatile("s_waitcnt lgkmcnt(0)" : "+v"(tdst) :: "memory");
        BAR();
        const u32x4 a = *(LAS u32x4*)misc, c = *(LAS u32x4*)(misc + 4);
        const u32 m = umax(umax(umax(a[0], a[1]), umax(a[2], a[3])), umax(umax(c[0], c[1]), umax(c[2], c[3])));
        if (lane < NR) __hip_atomic_store(tdst, ((u64)call << 32) | m, __ATOMIC_RELAXED, __HIP_MEMORY_SCOPE_SYSTEM);
    } else {
        WAIT_L(0);
        BAR();
    }
    {
        const bool sw = (lane >> 4) & 1;
#pragma unroll
        for (int m = 0; m < 5; ++m)
#pragma unroll
            for (int j = 0; j < 4; j += 2)
#pragma unroll
                for (int r = 0; r < 4; ++r) {
                    const float u = __builtin_fmaxf(acc[m][j][r], 0.f), v = __builtin_fmaxf(acc[m][j + 1][r], 0.f);
                    acc[m][j][r] = sw ? v : u; acc[m][j + 1][r] = sw ? u : v;
                }
#pragma unroll
        for (int m = 0; m < 5; ++m) asm volatile("" : "+v"(acc[m][0]), "+v"(acc[m][1]), "+v"(acc[m][2]), "+v"(acc[m][3]));
    }
    const __amdgpu_buffer_rsrc_t rsO = mk_rsrc(P.out, 4096u * NL);
    u32 oo = (u32)((rg0 * 16 + wm * 80 + (lane & 15)) * NL + ct * 256 + wn * 64 + (lane >> 4) * 16);
    asm volatile("" : "+v"(oo));
    {
        const u64 t0 = __builtin_amdgcn_s_memrealtime();
        u32 m = 0;
        bool bad;
        u32 o[4];
#pragma unroll
        for (int i = 0; i < 4; ++i) o[i] = i * 64 + lane < 208 ? tmx + (u32)(wid * 256 + i * 64 + lane) * 32u : lane == 63 ? OFF_TMO : 0x80000000u;
        for (;;) {
            asm volatile("" ::: "memory");
            u32x2 a[4];
#pragma unroll
            for (int i = 0; i < 4; ++i) a[i] = __builtin_amdgcn_raw_buffer_load_b64(rsW, o[i], 0, AUX_SYS);
            bad = (u32)__builtin_amdgcn_readlane((int)a[3][0], 63) == call;
#pragma unroll
            for (int i = 0; i < 4; ++i)
                if (a[i][1] == call) { m = umax(m, a[i][0]); o[i] = 0x80000000u; }
            const u64 miss = __builtin_amdgcn_ballot_w64((o[0] & o[1] & o[2] & (lane == 63 ? ~0u : o[3])) >> 31 == 0u);
            if (miss == 0ull) break;
            if (give_up(tmo, call, t0)) { bad = true; break; }
            if (__builtin_popcountll(miss) > 48) __builtin_amdgcn_s_sleep(16);
            __builtin_amdgcn_s_sleep(1);
        }
        m = bad ? 0x7FC00000u : wave_max_u32(m);
        if (lane == 0) misc[16 + wid] = m;
        WAIT_L(0);
    }
    BAR();
    {
        const u32x4 a = *(LAS u32x4*)(misc + 16), c = *(LAS u32x4*)(misc + 20);
        const u32 m = umax(umax(umax(a[0], a[1]), umax(a[2], a[3])), umax(umax(c[0], c[1]), umax(c[2], c[3])));
        float amax = __uint_as_float(m);
        if (rfl(m) < 0x0D800000u) {
            amax *= 0x1p64f;
#pragma unroll
            for (int mt = 0; mt < 5; ++mt)
#pragma unroll
                for (int j = 0; j < 4; ++j) acc[mt][j] *= 0x1p64f;
        }
        const float inv = 448.0f / amax;
        const f32x4 inv4 = {inv, inv, inv, inv};
#pragma unroll
        for (int mt = 0; mt < 5; ++mt) {
            u32x4 o;
#pragma unroll
            for (int j = 0; j < 4; ++j) {
                const f32x4 y = acc[mt][j] * inv4;
                int t = __builtin_amdgcn_cvt_pk_fp8_f32(y[0], y[1], __float_as_int(y[0]), false);
                t = __builtin_amdgcn_cvt_pk_fp8_f32(y[2], y[3], t, true);
                o[j] = (u32)t;
            }
            __builtin_amdgcn_raw_buffer_store_b128(o, rsO, oo + mt * 16 * NL, 0, AUX_AGT);
            HOLD(o);
        }
    }
}

}

extern "C" void kernel_launch(int rank, int nranks, ncclComm_t comm,
                              hipStream_t stream,
                              void* const* d_in, const int* in_sizes, int n_in,
                              void* d_out, int out_size,
                              const int* in_dtypes, int out_dtype,
                              void* const* d_out_all,
                              void* const* d_ws_all, size_t ws_bytes) {
    using namespace agk;
    if (nranks != NR || (unsigned)rank >= (unsigned)NR || ws_bytes < WS_USED || n_in < 2 ||
        in_sizes[0] != 512 * KD || in_sizes[1] != KD * NL || out_size != 4096 * NL) {
        fprintf(stderr, "unexpected problem shape\n");
        return;
    }
    static bool attr_done = false;
    if (!attr_done) {
        HIP_CHECK(hipFuncSetAttribute((const void*)ag_gemm, hipFuncAttributeMaxDynamicSharedMemorySize, LDS_BYTES));
        attr_done = true;
    }
    Params P;
    P.x = d_in[0]; P.w = d_in[1]; P.out = d_out;
    P.self = (char*)d_ws_all[rank];
    for (int p = 0; p < NR; ++p) P.peer[p] = (char*)d_ws_all[(rank + 1 + p) % NR];
    P.rank = rank; P.pad = 0;
    hipLaunchKernelGGL(ag_gemm, dim3(GRID), dim3(THREADS), LDS_BYTES, stream, P);
    HIP_CHECK(hipGetLastError());
}
```

```cpp
#include <hip/hip_runtime.h>
#include <rccl/rccl.h>
#include <cstdio>

#define HIP_CHECK(expr) do { hipError_t _e = (expr); if (_e != hipSuccess) { \
    fprintf(stderr, "HIP error %s at %s:%d\n", hipGetErrorString(_e), __FILE__, __LINE__); \
    return; } } while (0)

namespace agk {

typedef unsigned u32;
typedef unsigned long long u64;
typedef u32 u32x4 __attribute__((ext_vector_type(4)));
typedef u32 u32x2 __attribute__((ext_vector_type(2)));
typedef float f32x2 __attribute__((ext_vector_type(2)));
typedef short s16x2 __attribute__((ext_vector_type(2)));
typedef float f32x4 __attribute__((ext_vector_type(4)));
typedef __bf16 bf16x8 __attribute__((ext_vector_type(8)));
typedef short s16x4 __attribute__((ext_vector_type(4)));
typedef unsigned short u16x2 __attribute__((ext_vector_type(2)));
#define LAS __attribute__((address_space(3)))
#define GAS __attribute__((address_space(1)))

constexpr int NR = 8;
constexpr int KD = 4096;
constexpr int NL = 2048;
constexpr int GRID = 256, THREADS = 768;
constexpr int NPC = 22, PWAVES = 2, NPUSH = NPC * PWAVES;
constexpr int NDC = 48 - NPC, NDEC = NDC * 12;
constexpr int VRAW0 = 16, VRAW = 1024 - 32;
constexpr int STAGE = 52 * 1024;
constexpr int LDS_MISC = 3 * STAGE;
constexpr int LDS_BYTES = LDS_MISC + 1024;

constexpr u32 OFF_SEQ = 0;
constexpr u32 OFF_TMO = 16384;
constexpr u32 OFF_TKT = 32768;
constexpr u32 OFF_TMAX = 512u << 10;
constexpr u32 OFF_DONE = 1u << 20;
constexpr u32 OFF_HDR = 2u << 20;
constexpr u32 OFF_XD = 4u << 20;
constexpr u32 OFF_PK = 36u << 20;
constexpr u32 WS_USED = 60u << 20;

constexpr u64 DEADLINE_TICKS = 40000000ull;
constexpr int AUX_SYS = 17, AUX_AGT = 16;

struct Params {
    const void* x;
    const void* w;
    void* out;
    char* self;
    char* peer[NR];
    int rank;
    int pad;
};

#define WAIT_V(n) asm volatile("s_waitcnt vmcnt(" #n ")" ::: "memory")
#define WAIT_L(n) asm volatile("s_waitcnt lgkmcnt(" #n ")" ::: "memory")
#define BAR() do { asm volatile("" ::: "memory"); __builtin_amdgcn_s_barrier(); asm volatile("" ::: "memory"); } while (0)
#define SCHED() __builtin_amdgcn_sched_barrier(0)

__device__ __forceinline__ u32 rfl(u32 v) { return __builtin_amdgcn_readfirstlane(v); }
__device__ __forceinline__ __amdgpu_buffer_rsrc_t mk_rsrc(const void* p, u32 bytes) {
    return __builtin_amdgcn_make_buffer_rsrc((void*)p, 0, (int)bytes, 0x00020000);
}
#define HOLD(x) asm volatile("s_nop 1" :: "v"(x) : "memory")
#define HOLD4(x) asm volatile("s_nop 1" :: "v"(x[0]), "v"(x[1]), "v"(x[2]), "v"(x[3]) : "memory")
__device__ __forceinline__ int imax(int a, int b) { return a > b ? a : b; }
__device__ __forceinline__ u32 umax(u32 a, u32 b) { return a > b ? a : b; }
__device__ __forceinline__ u32 wave_max_u32(u32 v) {
    v = umax(v, (u32)__builtin_amdgcn_update_dpp(0, (int)v, 0x111, 0xf, 0xf, true));
    v = umax(v, (u32)__builtin_amdgcn_update_dpp(0, (int)v, 0x112, 0xf, 0xf, true));
    v = umax(v, (u32)__builtin_amdgcn_update_dpp(0, (int)v, 0x114, 0xf, 0xf, true));
    v = umax(v, (u32)__builtin_amdgcn_update_dpp(0, (int)v, 0x118, 0xf, 0xf, true));
    return umax(umax((u32)__builtin_amdgcn_readlane((int)v, 15), (u32)__builtin_amdgcn_readlane((int)v, 31)),
                umax((u32)__builtin_amdgcn_readlane((int)v, 47), (u32)__builtin_amdgcn_readlane((int)v, 63)));
}
__device__ __forceinline__ bool give_up(u32* tmo, u32 call, u64 t0) {
    const u64 dt = __builtin_amdgcn_s_memrealtime() - t0;
    if (dt < 20000ull) return false;
    const u32 t = rfl(__hip_atomic_load(tmo, __ATOMIC_RELAXED, __HIP_MEMORY_SCOPE_AGENT));
    if (t == call) return true;
    if (dt > DEADLINE_TICKS) {
        __hip_atomic_store(tmo, call, __ATOMIC_RELAXED, __HIP_MEMORY_SCOPE_AGENT);
        return true;
    }
    return false;
}

#define XL0 ((u32)((lane >> 3) * (KD * 2) + (((lane & 7) ^ (lane >> 4)) << 4)))
#define XL1 ((u32)((8 + (lane >> 3)) * (KD * 2) + (((lane & 7) ^ (4 + (lane >> 4))) << 4)))
#define DIRECT(v) ((v) < VRAW0 || (v) >= VRAW)
#define U_SRC(v) ((((v) & 15) * 32 * KD + ((v) >> 4) * 64) * 2)
#define U_XD(src, v) (OFF_XD + (u32)((v) >> 4) * 524288u + (u32)((src) * 16 + ((v) & 15)) * 4096u)
#define U_DONE(src, v) (OFF_DONE + (u32)(((v) >> 4) * 128 + (src) * 16 + ((v) & 15)) * 64u)

__device__ constexpr int mord(int i) { return (((i / 5) & 1) ? 4 - i % 5 : i % 5) * 4 + i / 5; }

__global__ __launch_bounds__(THREADS, 3) void ag_gemm(Params P) {
    extern __shared__ __attribute__((aligned(16))) char smem_[];
    LAS char* smem = (LAS char*)smem_;
    const int lane = threadIdx.x & 63;
    const int wid = (int)rfl(threadIdx.x >> 6);
    const int b = blockIdx.x;
    const int rank = P.rank & 7;
    char* ws = P.self;
    const void* px = P.x;
    asm volatile("" : "+s"(px));
    u32* tmo = (u32*)(ws + OFF_TMO);
    const __amdgpu_buffer_rsrc_t rsW = mk_rsrc(ws, WS_USED);
    u32* seq = (u32*)(ws + OFF_SEQ) + b * 16 + wid;

    auto peer_of = [&](int i) {
        u64 a = (u64)P.peer[7];
#pragma unroll
        for (int p = 0; p < 7; ++p) { u64 t = (u64)P.peer[p]; asm("" : "+s"(t)); a = i == p ? t : a; }
        return (GAS char*)a;
    };

    if (__builtin_expect(b < 48, 1)) {
        if (__builtin_expect(b < NPC, 1)) {
            if (__builtin_expect(wid >= PWAVES, 0)) return;
            const int pi = wid * NPC + b;
            const __amdgpu_buffer_rsrc_t rsX = mk_rsrc(px, 512u * KD * 2u);
            const u32 xl0 = XL0, xl1 = XL1;
            u32x4 d[4];
            u32 sq;
            asm volatile("buffer_load_dwordx4 %0, %5, %7, %8 offen\n\tbuffer_load_dwordx4 %1, %6, %7, %8 offen\n\t"
                         "buffer_load_dwordx4 %2, %5, %7, %9 offen\n\tbuffer_load_dwordx4 %3, %6, %7, %9 offen\n\tglobal_load_dword %4, %10, %11 sc1"
                         : "=&v"(d[0]), "=&v"(d[1]), "=&v"(d[2]), "=&v"(d[3]), "=&v"(sq)
                         : "v"(xl0), "v"(xl1), "s"(rsX), "s"(U_SRC(pi)), "s"(U_SRC(pi) + 16 * KD * 2), "v"(0u), "s"(seq) : "memory");
            SCHED();
            GAS char* hp = peer_of(lane >> 3) + (lane & 7) * 8;
            __amdgpu_buffer_rsrc_t rsP[7];
#pragma unroll
            for (int p = 0; p < 7; ++p) rsP[p] = mk_rsrc(P.peer[p], WS_USED);
            u32* tkt = (u32*)(ws + OFF_TKT);
            u32 call = 0, tbase = 0;
            int done = -1, cur = -1, nxt = pi;
            u32x4 pk[4] = {};
            u64 hw = 0, hw_done = 0;
            bool more = false;
            u32 low = 256u;
            bool raw = false;
            for (;;) {
                asm volatile("" ::: "memory");
                u32 tk;
                asm volatile("s_cmp_lt_u32 %4, 1024\n\ts_cbranch_scc0 1f\n\ts_mov_b64 exec, 1\n\tglobal_atomic_add %0, %1, %2, %3 sc0\n\ts_mov_b64 exec, -1\n1:"
                             : "=&v"(tk) : "v"(0u), "v"(1u), "s"(tkt), "s"(cur) : "memory", "scc");
                asm volatile("s_cmp_lt_u32 %8, 1024\n\ts_cbranch_scc0 1f\n\t"
                             "buffer_load_dwordx4 %0, %4, %6, %7 offen\n\tbuffer_load_dwordx4 %1, %5, %6, %7 offen\n\t"
                             "buffer_load_dwordx4 %2, %4, %6, %9 offen\n\tbuffer_load_dwordx4 %3, %5, %6, %9 offen\n1:"
                             : "+v"(d[0]), "+v"(d[1]), "+v"(d[2]), "+v"(d[3])
                             : "v"(xl0), "v"(xl1), "s"(rsX), "s"(U_SRC(nxt)), "s"(cur < 0 ? 1024 : nxt), "s"(U_SRC(nxt) + 16 * KD * 2) : "memory", "scc");
                if (done >= 0 && lane < 56 && (!DIRECT(done) || (lane & 7) == 0)) {
                    const u32 o = !DIRECT(done) ? OFF_HDR + (u32)(rank * 1024 + done) * 64u : U_DONE(rank, done);
                    __hip_atomic_store((GAS u64*)(hp + o), DIRECT(done) ? ((u64)call << 32) | call : hw_done | (u64)call << ((lane & 7) ? 55 : 32), __ATOMIC_RELAXED, __HIP_MEMORY_SCOPE_SYSTEM);
                }
                if (cur >= 1024) break;
                if (cur >= 0) {
                    const u32 o01 = raw ? U_XD(rank, cur) : OFF_PK + (u32)(rank * 1024 + cur) * 3072u;
                    const u32 o23 = o01 + 2048u;
                    const u32 v2 = raw || lane < (more ? 52 : 48) ? lane * 16u : 0x80000000u;
#pragma unroll
                    for (int p = 0; p < 7; ++p) {
                        __builtin_amdgcn_raw_buffer_store_b128(pk[0], rsP[p], lane * 16u, o01, AUX_SYS);
                        __builtin_amdgcn_raw_buffer_store_b128(pk[1], rsP[p], lane * 16u + 1024u, o01, AUX_SYS);
                        __builtin_amdgcn_raw_buffer_store_b128(pk[2], rsP[p], v2, o23, AUX_SYS);
                    }
                    if (raw) {
#pragma unroll
                        for (int p = 0; p < 7; ++p) __builtin_amdgcn_raw_buffer_store_b128(pk[3], rsP[p], lane * 16u + 1024u, o23, AUX_SYS);
                    }
                }
                HOLD4(pk);
                const int had = done;
                done = cur; hw_done = hw;
                asm volatile("s_waitcnt vmcnt(22)\n\ts_cmp_lt_i32 %5, 0\n\ts_cbranch_scc0 1f\n\ts_waitcnt vmcnt(21)\n\ts_cmp_lt_i32 %4, 0\n\ts_cbranch_scc0 1f\n\ts_waitcnt vmcnt(0)\n1:"
                             : "+v"(d[0]), "+v"(d[1]), "+v"(d[2]), "+v"(d[3]) : "s"(cur), "s"(had) : "memory", "scc");
                raw = true;
                if (!DIRECT(nxt)) {
                    LAS char* L = smem + wid * 4096;
                    u32 ns;
                    for (bool fresh = low > 255u;; fresh = true) {
                        if (fresh) {
                            u16x2 mx = {0, 0};
#pragma unroll
                            for (int i = 0; i < 4; ++i)
#pragma unroll
                                for (int e = 0; e < 4; ++e) mx = __builtin_elementwise_max(mx, __builtin_bit_cast(u16x2, d[i][e] & 0x7FFF7FFFu));
                            const u32 el = (u32)(mx.x > mx.y ? mx.x : mx.y) >> 7;
                            u32 top = wave_max_u32(el);
                            while (top > 7u && __builtin_popcountll(__builtin_amdgcn_ballot_w64(el >= top)) < 16) --top;
                            low = rfl(top < 7u ? 0u : top - 7u);
                        }
                        const u16x2 bias = __builtin_bit_cast(u16x2, low * 0x00800080u);
                        u32 sg = 0, c2[2] = {0, 0}, em = 0, mix = 0;
                        u32x4 hi[2];
#pragma unroll
                        for (int k = 0; k < 8; ++k) {
                            const u32 d0 = d[k >> 1][(k & 1) * 2], d1 = d[k >> 1][(k & 1) * 2 + 1];
                            const u32 x0 = __builtin_bit_cast(u32, (u16x2)(__builtin_bit_cast(u16x2, d0) - bias));
                            const u32 x1 = __builtin_bit_cast(u32, (u16x2)(__builtin_bit_cast(u16x2, d1) - bias));
                            const u32 h = __builtin_amdgcn_perm(x1, x0, 0x07050301u);
                            pk[k >> 2][k & 3] = __builtin_amdgcn_perm(x1, x0, 0x06040200u);
                            hi[k >> 2][k & 3] = h;
                            sg |= (h >> k) & (0x80808080u >> k);
                            c2[k >> 2] |= (h & 0x03030303u) << (2 * (k & 3));
                            em |= (k ? h >> (k - 1) : h << 1) & (0x80808080u >> k);
                            mix |= h ^ (h >> 1);
                        }
                        *(LAS u32*)(L + lane * 4) = sg;
                        *(LAS u32*)(L + 256 + lane * 4) = c2[0];
                        *(LAS u32*)(L + 512 + lane * 4) = c2[1];
                        more = __builtin_amdgcn_ballot_w64((mix & 0x3C3C3C3Cu) != 0u) != 0ull;
                        if (more) {
                            em = 0;
#pragma unroll
                            for (int k = 0; k < 8; ++k) em |= (((hi[k >> 2][k & 3] & 0x7C7C7C7Cu) + 0x7C7C7C7Cu) >> k) & (0x80808080u >> k);
                        }
                        u32 n = 0;
                        u64 m = __builtin_amdgcn_ballot_w64(em != 0u);
                        while (m != 0ull && n <= 64u) {
                            const u32 idx = n + __builtin_amdgcn_mbcnt_hi((u32)(m >> 32), __builtin_amdgcn_mbcnt_lo((u32)m, 0u));
                            const u32 bit = (u32)__builtin_ctz(em | 0x80000000u);
                            if (em != 0u) *(LAS unsigned short*)(L + 832 + idx * 2u) = (unsigned short)((u32)lane * 32u + (7u - (bit & 7u)) * 4u + (bit >> 3));
                            em &= em - 1u;
                            n += (u32)__builtin_popcountll(m);
                            m = __builtin_amdgcn_ballot_w64(em != 0u);
                        }
                        asm volatile("" ::: "memory");
                        ns = n;
                        raw = false;
                        more = more || n > 36u;
                        if (!more) break;
                        const u32 pos = *(LAS unsigned short*)(L + 832 + lane * 2) & 2047u;
                        *(LAS u32x4*)(L + 2048 + lane * 32) = hi[0];
                        *(LAS u32x4*)(L + 2064 + lane * 32) = hi[1];
                        if (lane < 16) *(LAS u32*)(L + 768 + lane * 4) = 0u;
                        asm volatile("" ::: "memory");
                        const u32 hb = *(LAS unsigned char*)(L + 2048 + pos) & 0x7Cu;
                        const u64 ms = __builtin_amdgcn_ballot_w64((u32)lane < n && hb == 0x7Cu);
                        const u32 is = __builtin_amdgcn_mbcnt_hi((u32)(ms >> 32), __builtin_amdgcn_mbcnt_lo((u32)ms, 0u));
                        const bool sh = hb == 0x7Cu && is < 36u;
                        const u32 il = (u32)lane - (is < 36u ? is : 36u);
                        const u32 nsa = (u32)__builtin_popcountll(ms);
                        ns = nsa < 36u ? nsa : 36u;
                        if ((u32)lane < n && (sh || il < 32u)) *(LAS unsigned short*)(L + (sh ? 832u + is * 2u : 768u + il * 2u)) = (unsigned short)(sh ? pos : pos | hb << 9);
                        asm volatile("" ::: "memory");
                        raw = n > 64u || n - ns > 32u;
                        if (!raw || fresh) break;
                    }
                    pk[2] = *(LAS u32x4*)(L + lane * 16);
                    u32 e[5];
#pragma unroll
                    for (int i = 0; i < 5; ++i) e[i] = *(LAS unsigned short*)(L + 824 + (lane & 7) * 10 + i * 2) & 2047u;
                    hw = (lane & 7) ? (u64)(e[0] | e[1] << 11 | e[2] << 22) | (u64)(e[2] >> 10 | e[3] << 1 | e[4] << 12) << 32
                                    : (u64)(low | (raw ? 256u : 0u) | (more ? 512u : 0u) | ns << 10 | e[4] << 16);
                    asm volatile("" ::: "memory");
                }
                if (raw) {
#pragma unroll
                    for (int i = 0; i < 4; ++i) pk[i] = d[i];
                }
                asm volatile("s_waitcnt vmcnt(0)" : "+v"(tk), "+v"(sq) :: "memory");
                const bool first = cur < 0;
                cur = nxt;
                if (first) {
                    call = rfl(sq) + 1u;
                    tbase = (call - 1u) * 1024u - 2u * NPUSH;
                    if (lane == 0) __hip_atomic_store(seq, call, __ATOMIC_RELAXED, __HIP_MEMORY_SCOPE_AGENT);
                    nxt = pi + NPUSH;
                } else {
                    const u32 u = rfl(tk) - tbase;
                    nxt = (int)(u < 1024u ? u : 1024u);
                }
            }
            return;
        }
        const u32 call = rfl(__hip_atomic_load(seq, __ATOMIC_RELAXED, __HIP_MEMORY_SCOPE_AGENT)) + 1u;
        if (lane == 0) __hip_atomic_store(seq, call, __ATOMIC_RELAXED, __HIP_MEMORY_SCOPE_AGENT);
        {
            u32* pend = nullptr;
            LAS char* L = smem + wid * 2048;
            *(LAS u32x4*)(L + lane * 32) = (u32x4){0u, 0u, 0u, 0u};
            *(LAS u32x4*)(L + lane * 32 + 16) = (u32x4){0u, 0u, 0u, 0u};
            const int hg = lane < 36 ? (lane + 4) / 5 : 0;
            const int hs = hg ? 11 * (lane + 4 - 5 * hg) : 16;
            const u32 tsh = hg ? 23u : 0u, tag = hg ? call & 511u : call;
            for (int q = VRAW0 * 7 + wid * NDC + b - NPC; q < VRAW * 7; q += NDEC) {
                const int v = q / 7, src = (rank + 1 + q - v * 7) & 7;
                const u32 xdb = U_XD(src, v);
                const u64* hp = (const u64*)(ws + OFF_HDR + (src * 1024 + v) * 64u) + hg;
                u32 hw, h0;
                bool dead = false;
                {
                    const u64 t0 = __builtin_amdgcn_s_memrealtime();
                    for (;;) {
                        const u64 g = __hip_atomic_load(hp, __ATOMIC_RELAXED, __HIP_MEMORY_SCOPE_SYSTEM);
                        hw = (u32)(g >> hs) & 2047u; h0 = rfl((u32)g);
                        u64 ok = __builtin_amdgcn_ballot_w64((u32)(g >> 32) >> tsh == tag);
                        asm volatile("s_waitcnt vmcnt(0)" : "+s"(ok), "+s"(h0), "+v"(hw) :: "memory");
                        if (pend) {
                            if (lane == 0) __hip_atomic_store(pend, call, __ATOMIC_RELAXED, __HIP_MEMORY_SCOPE_AGENT);
                            pend = nullptr;
                        }
                        if (ok == ~0ull) break;
                        if (give_up(tmo, call, t0)) { dead = true; break; }
                        __builtin_amdgcn_s_sleep(1);
                    }
                }
                if (dead) break;
                asm volatile("" ::: "memory");
                if (!(h0 & 256u)) {
                    const u32 pk = OFF_PK + (u32)(src * 1024 + v) * 3072u;
                    const u32 sg = __builtin_amdgcn_raw_buffer_load_b32(rsW, 2048u + lane * 4u, pk, AUX_SYS);
                    u32 c2[2];
                    c2[0] = __builtin_amdgcn_raw_buffer_load_b32(rsW, 2304u + lane * 4u, pk, AUX_SYS);
                    c2[1] = __builtin_amdgcn_raw_buffer_load_b32(rsW, 2560u + lane * 4u, pk, AUX_SYS);
                    u32x4 lo[2];
                    lo[0] = __builtin_amdgcn_raw_buffer_load_b128(rsW, lane * 16u, pk, AUX_SYS);
                    lo[1] = __builtin_amdgcn_raw_buffer_load_b128(rsW, lane * 16u + 1024u, pk, AUX_SYS);
                    const bool sh = (u32)lane < ((h0 >> 10) & 63u);
                    if (sh) *(LAS unsigned char*)(L + hw) = 0x7C;
                    u32 ent = 0;
                    if (h0 & 512u) {
                        ent = __builtin_amdgcn_raw_buffer_load_b16(rsW, 2816u + (lane & 31) * 2u, pk, AUX_SYS);
                        if (ent >= 2048u) *(LAS unsigned char*)(L + (ent & 2047u)) = (unsigned char)((ent >> 11) << 2);
                    }
                    asm volatile("" ::: "memory");
                    u32x4 ex[2];
                    ex[0] = *(LAS u32x4*)(L + lane * 32);
                    ex[1] = *(LAS u32x4*)(L + lane * 32 + 16);
                    asm volatile("" ::: "memory");
                    if (sh) *(LAS unsigned char*)(L + hw) = 0;
                    if (ent >= 2048u) *(LAS unsigned char*)(L + (ent & 2047u)) = 0;
                    const u16x2 bias = __builtin_bit_cast(u16x2, (h0 & 255u) * 0x00800080u);
#pragma unroll
                    for (int i = 0; i < 4; ++i) {
                        u32x4 o;
#pragma unroll
                        for (int j = 0; j < 2; ++j) {
                            const int k = 2 * i + j;
                            const u32 h = ((sg << k) & 0x80808080u) | ((c2[k >> 2] >> (2 * (k & 3))) & 0x03030303u) | ex[k >> 2][k & 3];
                            const u32 l = lo[k >> 2][k & 3];
                            o[2 * j] = __builtin_bit_cast(u32, (u16x2)(__builtin_bit_cast(u16x2, __builtin_amdgcn_perm(h, l, 0x05010400u)) + bias));
                            o[2 * j + 1] = __builtin_bit_cast(u32, (u16x2)(__builtin_bit_cast(u16x2, __builtin_amdgcn_perm(h, l, 0x07030602u)) + bias));
                        }
                        __builtin_amdgcn_raw_buffer_store_b128(o, rsW, lane * 16u + i * 1024u, xdb, AUX_AGT);
                        HOLD(o);
                    }
                }
                asm volatile("" ::: "memory");
                pend = (u32*)(ws + U_DONE(src, v));
            }
            WAIT_V(0);
            if (pend && lane == 0) __hip_atomic_store(pend, call, __ATOMIC_RELAXED, __HIP_MEMORY_SCOPE_AGENT);
        }
        return;
    }

    const int T = (b & 7) * 26 + (b >> 3) - 6;
    const int ct = (T / 104) * 4 + (T % 104 & 3), rt = (T % 104) >> 2;
    const int rg0 = rt * 10;
    const int nblk = rt == 25 ? 3 : 5;
    const u32 lds0 = (u32)(size_t)smem;
#define DMA(ldsaddr, voff, rs, soff, mods) asm volatile("s_mov_b32 m0, %0\n\ts_nop 0\n\tbuffer_load_dwordx4 %1, %2, %3 offen" mods " lds" \
        :: "s"(ldsaddr), "v"(voff), "s"(rs), "s"(soff) : "memory")
#define DMA_SAME_M0(voff, rs, soff, mods) asm volatile("buffer_load_dwordx4 %0, %1, %2 offen" mods " lds" :: "v"(voff), "s"(rs), "s"(soff) : "memory")

    if (wid == 11) return;
    if (wid >= 9) {
        __builtin_amdgcn_s_setprio(2);
        const int h = wid - 9;
        const __amdgpu_buffer_rsrc_t rsB = mk_rsrc(P.w, (u32)KD * NL * 2u);
        u32 vB[16];
#pragma unroll
        for (int i = 0; i < 16; ++i) {
            const int row = i * 2 + (lane >> 5);
            vB[i] = (u32)((h * 32 + row) * (NL * 2) + ct * 512 + (((lane & 31) ^ ((row & 3) | (row & 8))) << 4));
        }
        const u32 lB = lds0 + 20480 + h * 16384;
#define LOAD_W(kt, so) do { _Pragma("unroll") for (int i = 0; i < 16; ++i) DMA(lB + (so) + i * 1024, vB[i], rsB, (kt) * 262144, ""); } while (0)
        LOAD_W(0, 0);
        LOAD_W(1, STAGE);
        WAIT_V(16);
        BAR();
        int so = 2 * STAGE;
        for (int t = 0; t < 62; ++t) {
            LOAD_W(t + 2, so);
            so = so == 2 * STAGE ? 0 : so + STAGE;
            WAIT_V(16);
            BAR();
        }
        WAIT_V(0);
        BAR();
        return;
    }

    if (wid == 8) {
        __builtin_amdgcn_s_setprio(3);
        const u32 call = rfl(__hip_atomic_load(seq, __ATOMIC_RELAXED, __HIP_MEMORY_SCOPE_AGENT)) + 1u;
        if (lane == 0) __hip_atomic_store(seq, call, __ATOMIC_RELAXED, __HIP_MEMORY_SCOPE_AGENT);
        const u32* cp = (const u32*)(ws + OFF_DONE + ((rg0 >> 1) + ((lane & 7) < nblk ? (lane & 7) : 0)) * 64u);
        const u32 vA = (u32)lane * 16u;
        const u32 gA = OFF_XD + (u32)rg0 * 2048u;
        int ready = 0;
        auto look = [&]() {
            const u32 f = __hip_atomic_load(cp + ready * 2048, __ATOMIC_RELAXED, __HIP_MEMORY_SCOPE_SYSTEM);
            if (__builtin_amdgcn_ballot_w64(f < call) == 0ull) ++ready;
        };
        auto need = [&](int kt) {
            if (kt < ready) return;
            const u64 t0 = __builtin_amdgcn_s_memrealtime();
            for (;;) {
                look();
                if (kt < ready) break;
                if (give_up(tmo, call, t0)) { ready = 64; break; }
                __builtin_amdgcn_s_sleep(1);
            }
        };
#define LOAD_X4(g, l) do { \
            DMA((l), vA, rsW, (g), " sc0 sc1"); DMA_SAME_M0(vA, rsW, (g), " offset:1024 sc0 sc1"); DMA_SAME_M0(vA, rsW, (g), " offset:2048 sc0 sc1"); DMA_SAME_M0(vA, rsW, (g), " offset:3072 sc0 sc1"); } while (0)
#define LOAD_X(kt, so) do { _Pragma("unroll") for (int i = 0; i < 5; ++i) \
            LOAD_X4(gA + (kt) * 524288 + (i < nblk ? i : 0) * 4096, lds0 + (so) + (i < nblk ? i : 0) * 4096); } while (0)
        int xi = 0, so = 0;
        for (int t = -1; t < 63; ++t) {
            if (xi == t + 1) {
                need(xi);
                LOAD_X(xi, so);
                ++xi; so = so == 2 * STAGE ? 0 : so + STAGE;
            }
            if (xi < 64 && xi >= ready) look();
            if (xi < 64 && xi < ready) {
                LOAD_X(xi, so);
                ++xi; so = so == 2 * STAGE ? 0 : so + STAGE;
                WAIT_V(20);
            } else {
                WAIT_V(0);
            }
            BAR();
        }
        return;
    }

    const int wm = wid >> 2, wn = wid & 3;
    const u32 call = rfl(__hip_atomic_load(seq, __ATOMIC_RELAXED, __HIP_MEMORY_SCOPE_AGENT)) + 1u;
    if (lane == 0) __hip_atomic_store(seq, call, __ATOMIC_RELAXED, __HIP_MEMORY_SCOPE_AGENT);
    if (wid * 208 + T < 1024) {
        const int v = wid * 208 + T;
        const __amdgpu_buffer_rsrc_t rsX = mk_rsrc(px, 512u * KD * 2u);
        u32x4 d[4];
#pragma unroll
        for (int i = 0; i < 4; ++i) d[i] = __builtin_amdgcn_raw_buffer_load_b128(rsX, (i & 1) ? XL1 : XL0, U_SRC(v) + (i >> 1) * 16 * KD * 2, 0);
#pragma unroll
        for (int i = 0; i < 4; ++i) __builtin_amdgcn_raw_buffer_store_b128(d[i], rsW, lane * 16u + i * 1024u, U_XD(rank, v), AUX_AGT);
        HOLD4(d);
        WAIT_V(0);
        if (lane == 0) __hip_atomic_store((u32*)(ws + U_DONE(rank, v)), call, __ATOMIC_RELAXED, __HIP_MEMORY_SCOPE_AGENT);
    }

    {
        __builtin_amdgcn_s_sleep(8);
        const __amdgpu_buffer_rsrc_t rsB = mk_rsrc(P.w, (u32)KD * NL * 2u);
        u32 t[3];
#pragma unroll
        for (int i = 0; i < 3; ++i) {
            const u32 r = (u32)(i * 1664 + T * 8 + wid);
            t[i] = __builtin_amdgcn_raw_buffer_load_b32(rsB, lane * 64u, (r < 4096u ? r : 0u) * 4096u, 0);
        }
        asm volatile("" :: "v"(t[0]), "v"(t[1]), "v"(t[2]));
    }
    if (nblk < 5) {
#pragma unroll
        for (int i = 0; i < 3; ++i) *(LAS u32x4*)(smem + i * STAGE + 12288 + wid * 1024 + lane * 16) = (u32x4){0u, 0u, 0u, 0u};
        WAIT_L(0);
    }
    u32 pa[2], pb[4];
#pragma unroll
    for (int h = 0; h < 2; ++h) pa[h] = lds0 + wm * 10240 + (lane & 15) * 128 + (((4 * h + (lane >> 4)) ^ ((lane >> 1) & 7)) << 4);
    {
        const int g = lane >> 4, q = (lane >> 2) & 3, p = lane & 3;
#pragma unroll
        for (int j = 0; j < 4; ++j) {
            const int jj = j ^ (p & 1);
            pb[j] = lds0 + 20480 + (8 * g + q) * 512 + (((wn * 8 + 2 * p + (jj >> 1)) ^ (q | ((g & 1) << 3))) << 4) + (jj & 1) * 8;
        }
    }
    f32x4 acc[5][4];
#pragma unroll
    for (int m = 0; m < 5; ++m)
#pragma unroll
        for (int j = 0; j < 4; ++j) acc[m][j] = (f32x4){0.f, 0.f, 0.f, 0.f};

    s16x4 f0b[8], f1b[8];
    bf16x8 f0a[5], f1a[5];
#define RD(i, fa, fb, h) do { \
        if ((i) < 8) fb[(i) & 7] = __builtin_amdgcn_ds_read_tr16_b64_v4i16((LAS s16x4*)(size_t)(pb[((i) & 7) >> 1] + (h) * 16384 + ((i) & 1) * 2048)); \
        else fa[(i) < 8 ? 0 : (i) - 8] = *(LAS bf16x8*)(size_t)(pa[h] + ((i) - 8) * 2048); } while (0)
#define MM(i, fa, fb) acc[(i) >> 2][(i) & 3] = __builtin_amdgcn_mfma_f32_16x16x32_bf16( \
        __builtin_bit_cast(bf16x8, __builtin_shufflevector(fb[((i) & 3) * 2], fb[((i) & 3) * 2 + 1], 0, 1, 2, 3, 4, 5, 6, 7)), fa[(i) >> 2], acc[(i) >> 2][(i) & 3], 0, 0, 0)
#define VADD(v, s) asm volatile("v_add_u32 %0, %1, %0" : "+v"(v) : "s"(s))
    BAR();
    if (wm == 0) __builtin_amdgcn_s_setprio(1);
    int ph = 0;
    for (int t = 0; t < 63; ++t) {
        SCHED();
#pragma unroll
        for (int i = 0; i < 13; ++i) RD(i, f0a, f0b, 0);
#pragma unroll
        for (int i = 0; i < 13; ++i) RD(i, f1a, f1b, 1);
        SCHED();
        {
            const int dlt = ph == 2 ? -2 * STAGE : STAGE;
            ph = ph == 2 ? 0 : ph + 1;
            VADD(pa[0], dlt); VADD(pa[1], dlt);
#pragma unroll
            for (int j = 0; j < 4; ++j) VADD(pb[j], dlt);
        }
        asm volatile("s_cmp_lg_u32 %0, 0\n\ts_cbranch_scc1 1f\n\ts_waitcnt lgkmcnt(0)\n\ts_barrier\n1:" :: "s"(wm) : "memory", "scc");
        SCHED();
#pragma unroll
        for (int i = 0; i < 20; ++i) {
            if (i & 1) { MM(mord(i), f1a, f1b); SCHED(); MM(mord(i), f0a, f0b); SCHED(); }
            else { MM(mord(i), f0a, f0b); SCHED(); MM(mord(i), f1a, f1b); SCHED(); }
        }
        SCHED();
        asm volatile("s_cmp_lg_u32 %0, 1\n\ts_cbranch_scc1 1f\n\ts_barrier\n1:" :: "s"(wm) : "memory", "scc");
    }
    u32 mxi = 0;
    {
        SCHED();
#pragma unroll
        for (int i = 0; i < 13; ++i) RD(i, f0a, f0b, 0);
#pragma unroll
        for (int i = 0; i < 13; ++i) RD(i, f1a, f1b, 1);
        SCHED();
        int m0 = 0, m1 = 0;
#define MX(i) do { const f32x4 v = acc[mord(i) >> 2][mord(i) & 3]; \
            m0 = imax(imax(m0, __float_as_int(v[0])), __float_as_int(v[1])); m1 = imax(imax(m1, __float_as_int(v[2])), __float_as_int(v[3])); } while (0)
#pragma unroll
        for (int i = 0; i < 20; ++i) {
            if (i & 1) { MM(mord(i), f1a, f1b); SCHED(); MM(mord(i), f0a, f0b); SCHED(); }
            else { MM(mord(i), f0a, f0b); SCHED(); MM(mord(i), f1a, f1b); SCHED(); }
            if (i >= 2) { MX(i - 2); asm volatile("" : "+v"(m0), "+v"(m1)); SCHED(); }
        }
        MX(18); MX(19);
        mxi = (u32)imax(m0, m1);
    }
    __builtin_amdgcn_s_setprio(0);

    const u32 tmx = OFF_TMAX + (call & 1u) * 65536u;
    LAS u32* misc = (LAS u32*)(smem + LDS_MISC + 512);
    {
        const u32 wmx = wave_max_u32(mxi);
        if (lane == 0) misc[wid] = wmx;
    }
    if (wid == 0) {
        GAS u64* tdst = (GAS u64*)(peer_of(lane & 7) + tmx + (rank * 256 + T) * 32u);
        asm volatile("s_waitcnt lgkmcnt(0)" : "+v"(tdst) :: "memory");
        BAR();
        const u32x4 a = *(LAS u32x4*)misc, c = *(LAS u32x4*)(misc + 4);
        const u32 m = umax(umax(umax(a[0], a[1]), umax(a[2], a[3])), umax(umax(c[0], c[1]), umax(c[2], c[3])));
        if (lane < NR) __hip_atomic_store(tdst, ((u64)call << 32) | m, __ATOMIC_RELAXED, __HIP_MEMORY_SCOPE_SYSTEM);
    } else {
        WAIT_L(0);
        BAR();
    }
    {
        const bool send = ((lane >> 3) ^ wn) & 1;
        const u32 sl = lds0 + ((lane >> 4) * 8 + (lane & 7)) * 16;
        if (send) {
#pragma unroll
            for (int i = 0; i < 20; ++i) *(LAS f32x4*)(size_t)(sl + wid * 10240 + i * 512) = acc[i >> 2][i & 3];
        }
        WAIT_L(0);
        BAR();
        if (send) {
#pragma unroll
            for (int i = 0; i < 20; ++i) acc[i >> 2][i & 3] = *(LAS f32x4*)(size_t)(sl + (wid ^ 1) * 10240 + i * 512);
        }
#pragma unroll
        for (int m = 0; m < 5; ++m) asm volatile("" : "+v"(acc[m][0]), "+v"(acc[m][1]), "+v"(acc[m][2]), "+v"(acc[m][3]));
    }
    const __amdgpu_buffer_rsrc_t rsO = mk_rsrc(P.out, 4096u * NL);
    u32 oo = (u32)((rg0 * 16 + wm * 80 + (lane & 7) + (wn & 1) * 8) * NL + ct * 256 + (wn >> 1) * 128 + ((lane >> 3) & 1) * 64 + (lane >> 4) * 16);
    asm volatile("" : "+v"(oo));
    {
        const u64 t0 = __builtin_amdgcn_s_memrealtime();
        u32 m = 0;
        bool bad;
        u32 o[4];
#pragma unroll
        for (int i = 0; i < 4; ++i) o[i] = i * 64 + lane < 208 ? tmx + (u32)(wid * 256 + i * 64 + lane) * 32u : lane == 63 ? OFF_TMO : 0x80000000u;
        for (;;) {
            asm volatile("" ::: "memory");
            u32x2 a[4];
#pragma unroll
            for (int i = 0; i < 4; ++i) a[i] = __builtin_amdgcn_raw_buffer_load_b64(rsW, o[i], 0, AUX_SYS);
            bad = (u32)__builtin_amdgcn_readlane((int)a[3][0], 63) == call;
#pragma unroll
            for (int i = 0; i < 4; ++i)
                if (a[i][1] == call) { m = umax(m, a[i][0]); o[i] = 0x80000000u; }
            const u64 miss = __builtin_amdgcn_ballot_w64((o[0] & o[1] & o[2] & (lane == 63 ? ~0u : o[3])) >> 31 == 0u);
            if (miss == 0ull) break;
            if (give_up(tmo, call, t0)) { bad = true; break; }
            if (__builtin_popcountll(miss) > 48) __builtin_amdgcn_s_sleep(16);
            __builtin_amdgcn_s_sleep(1);
        }
        m = bad ? 0x7FC00000u : wave_max_u32(m);
        if (lane == 0) misc[16 + wid] = m;
        WAIT_L(0);
    }
    BAR();
    {
        const u32x4 a = *(LAS u32x4*)(misc + 16), c = *(LAS u32x4*)(misc + 20);
        const u32 m = umax(umax(umax(a[0], a[1]), umax(a[2], a[3])), umax(umax(c[0], c[1]), umax(c[2], c[3])));
        float amax = __uint_as_float(m);
        if (rfl(m) - 0x0D800000u >= 0x64000000u) {
            const float f = m < 0x0D800000u ? 0x1p64f : 0x1p-64f;
            amax *= f;
#pragma unroll
            for (int mt = 0; mt < 5; ++mt)
#pragma unroll
                for (int j = 0; j < 4; ++j) acc[mt][j] *= f;
        }
        const float inv = 0.875f / amax;
        const f32x2 inv2 = {inv, inv};
        const bool sw = (lane >> 4) & 1;
        const u32 nan = inv != inv || amax == 0.f ? 0x7F7F7F7Fu : 0u;
#pragma unroll
        for (int mt = 0; mt < 5; ++mt) {
            u32 q[4];
#pragma unroll
            for (int j = 0; j < 4; ++j) {
                f32x2 y0, y1;
                asm("v_pk_mul_f32 %0, %1, %2 clamp" : "=v"(y0) : "v"(__builtin_shufflevector(acc[mt][j], acc[mt][j], 0, 1)), "v"(inv2));
                asm("v_pk_mul_f32 %0, %1, %2 clamp" : "=v"(y1) : "v"(__builtin_shufflevector(acc[mt][j], acc[mt][j], 2, 3)), "v"(inv2));
                const float y00 = y0[0];
                s16x2 t = __builtin_bit_cast(s16x2, y00);
                t = __builtin_amdgcn_cvt_scalef32_pk_fp8_f32(t, y0[0], y0[1], 0x1p-9f, false);
                t = __builtin_amdgcn_cvt_scalef32_pk_fp8_f32(t, y1[0], y1[1], 0x1p-9f, true);
                q[j] = __builtin_bit_cast(u32, t);
            }
            u32x4 o;
            o[0] = sw ? q[1] : q[0]; o[1] = sw ? q[0] : q[1]; o[2] = sw ? q[3] : q[2]; o[3] = sw ? q[2] : q[3];
            if (rfl(nan)) o = (u32x4){nan, nan, nan, nan};
            __builtin_amdgcn_raw_buffer_store_b128(o, rsO, oo + mt * 16 * NL, 0, AUX_AGT);
            HOLD(o);
            SCHED();
        }
    }
}

}

extern "C" void kernel_launch(int rank, int nranks, ncclComm_t comm,
                              hipStream_t stream,
                              void* const* d_in, const int* in_sizes, int n_in,
                              void* d_out, int out_size,
                              const int* in_dtypes, int out_dtype,
                              void* const* d_out_all,
                              void* const* d_ws_all, size_t ws_bytes) {
    using namespace agk;
    if (nranks != NR || (unsigned)rank >= (unsigned)NR || ws_bytes < WS_USED || n_in < 2 ||
        in_sizes[0] != 512 * KD || in_sizes[1] != KD * NL || out_size != 4096 * NL) {
        fprintf(stderr, "unexpected problem shape\n");
        return;
    }
    static bool attr_done = false;
    if (!attr_done) {
        HIP_CHECK(hipFuncSetAttribute((const void*)ag_gemm, hipFuncAttributeMaxDynamicSharedMemorySize, LDS_BYTES));
        attr_done = true;
    }
    Params P;
    P.x = d_in[0]; P.w = d_in[1]; P.out = d_out;
    P.self = (char*)d_ws_all[rank];
    for (int p = 0; p < NR; ++p) P.peer[p] = (char*)d_ws_all[(rank + 1 + p) % NR];
    P.rank = rank; P.pad = 0;
    hipLaunchKernelGGL(ag_gemm, dim3(GRID), dim3(THREADS), LDS_BYTES, stream, P);
    HIP_CHECK(hipGetLastError());
}
```
